```python
import numpy as np

kernel_name = "gpu_gemm_rs_m4096_k4096_n4096_e4m3e5m2_dequant"


ABSMAX_RTOL = 0.05

COMPUTE_DTYPE = "float32"
EPILOGUE = "dequant"
IN_QUANT = None
OUT_QUANT = None
IN_FP8 = "e4m3e5m2"
IN_DTYPES = {"x": "e4m3", "w_mat": "e5m2",
             "scale_x": "float32", "scale_w": "float32"}
WORLD_SIZE = 8

def _snap_fp8(x, mant_bits, max_abs, min_normal):
    a = np.abs(x.astype(np.float64))
    _m, _e = np.frexp(a)
    q = np.where(a >= min_normal, np.ldexp(1.0, _e - 1 - mant_bits),
                 min_normal * 2.0 ** -mant_bits)
    snapped = np.minimum(np.round(a / q) * q, max_abs)
    return (np.sign(x) * snapped).astype(np.float32)

def setup_inputs():

    x = _snap_fp8(np.random.randn(4096, 4096).astype(np.float32), 3, 448.0, 2.0 ** -6)
    w_mat = _snap_fp8(np.random.randn(4096, 4096).astype(np.float32), 2, 57344.0, 2.0 ** -14)
    scale_x = (0.02 * (1.0 + np.random.rand(1))).astype(np.float32)
    scale_w = (0.02 * (1.0 + np.random.rand(1))).astype(np.float32)
    return {"x": x, "w_mat": w_mat, "scale_x": scale_x, "scale_w": scale_w}

def reference(x, w_mat, scale_x, scale_w):
    acc = (x @ w_mat).astype(np.float32)
    y = (acc * (scale_x * scale_w)).astype(np.float32)
    return y
```

```cpp
#include <hip/hip_runtime.h>
#include <rccl/rccl.h>
#include <cstdio>
#include <cstdint>

#define HIP_CHECK(expr) do { hipError_t _e = (expr); if (_e != hipSuccess) { \
    fprintf(stderr, "HIP error %s at %s:%d\n", hipGetErrorString(_e), __FILE__, __LINE__); \
    return; } } while (0)

namespace {

typedef unsigned u32;
typedef unsigned long long u64;
typedef unsigned u32x4 __attribute__((ext_vector_type(4)));
typedef int i32x8 __attribute__((ext_vector_type(8)));
typedef float f32x4 __attribute__((ext_vector_type(4)));

constexpr int NR = 8;
constexpr int MS = 512;
constexpr int KS = 512;
constexpr int NN = 4096;
constexpr int NJ = 8;
constexpr int NC = 64;
constexpr int NU = NJ * NC;
constexpr int NXU = NR * 16;

#define NSEND 128
#define SWAVES 1
constexpr int NS = NSEND;
constexpr int SW = SWAVES;
constexpr int NW = NS * SW;
constexpr int NCONS = 256;
constexpr int GRID = NS + NCONS;

constexpr size_t CNT_OFF   = 0;
constexpr size_t TMO_OFF   = 64 << 10;
constexpr size_t XFLAG_OFF = 128 << 10;
constexpr size_t WFLAG_OFF = 256 << 10;
constexpr size_t DATA_OFF  = 1 << 20;
constexpr size_t PAR_BYTES = 32u << 20;
constexpr size_t X_BYTES   = 2u << 20;
constexpr u32    SLOT      = 4224;
constexpr size_t W_BYTES   = (size_t)NU * NR * SLOT;
constexpr size_t WS_NEED   = DATA_OFF + PAR_BYTES + X_BYTES + W_BYTES;
static_assert(X_BYTES + W_BYTES <= PAR_BYTES, "parity region too small");
static_assert((size_t)GRID * 4 * 32 <= TMO_OFF, "counter region too small");
static_assert(WFLAG_OFF + (size_t)NU * NR * 128 <= DATA_OFF, "flag region too small");

constexpr u64 DEADLINE = 40000000ull;
constexpr int AUX_SYS = 17;
constexpr u32 MAX_EXC = 511;

struct Params {
    const unsigned char* x;
    const unsigned char* w;
    const float* sx;
    const float* sw;
    float* out;
    char* ws[NR];
    int rank;
    int pad;
};

__device__ __forceinline__ u32 perm(u32 hi, u32 lo, u32 sel) { return __builtin_amdgcn_perm(hi, lo, sel); }

template <int IMM>
__device__ __forceinline__ void st16_sys(char* base, u32 off, u32x4 v) {
    asm volatile("global_store_dwordx4 %0, %1, %2 offset:%3 sc0 sc1\n\ts_nop 1"
                 :: "v"(off), "v"(v), "s"(base), "i"(IMM) : "memory");
}
__device__ __forceinline__ char* uni(char* p) {
    const u64 v = (u64)p;
    const u32 lo = __builtin_amdgcn_readfirstlane((u32)v);
    const u32 hi = __builtin_amdgcn_readfirstlane((u32)(v >> 32));
    char* r = (char*)(((u64)hi << 32) | lo);
    asm volatile("s_nop 4" : "+s"(r));
    return r;
}
__device__ __forceinline__ void wait_vm0() { asm volatile("s_waitcnt vmcnt(0)" ::: "memory"); }

#define STAMP(slot) do {} while (0)

__device__ __forceinline__ void wait_flags(const u32* flag, bool active, u32 call, u32* tmo) {
    const u64 t0 = __builtin_amdgcn_s_memrealtime();
    for (u32 it = 0;; ++it) {
        u32 v = call;
        if (active) v = __hip_atomic_load(flag, __ATOMIC_RELAXED, __HIP_MEMORY_SCOPE_SYSTEM);
        if (__all(v >= call)) break;
        if ((it & 15) == 15) {
            if (__hip_atomic_load(tmo, __ATOMIC_RELAXED, __HIP_MEMORY_SCOPE_AGENT) == call ||
                __builtin_amdgcn_s_memrealtime() - t0 > DEADLINE) {
                __hip_atomic_store(tmo, call, __ATOMIC_RELAXED, __HIP_MEMORY_SCOPE_AGENT);
                break;
            }
        }
        __builtin_amdgcn_s_sleep(2);
    }
    asm volatile("" ::: "memory");
}

template <int CTRL, int RMASK>
__device__ __forceinline__ u32 dpp0(u32 v) {
    return __builtin_amdgcn_update_dpp(0u, v, CTRL, RMASK, 0xf, false);
}
__device__ __forceinline__ u32 wave_incl_scan(u32 v) {
    v += dpp0<0x111, 0xf>(v);
    v += dpp0<0x112, 0xf>(v);
    v += dpp0<0x114, 0xf>(v);
    v += dpp0<0x118, 0xf>(v);
    v += dpp0<0x142, 0xa>(v);
    v += dpp0<0x143, 0xc>(v);
    return v;
}
__device__ __forceinline__ u32 wave_max(u32 v) {
    v = max(v, dpp0<0x111, 0xf>(v));
    v = max(v, dpp0<0x112, 0xf>(v));
    v = max(v, dpp0<0x114, 0xf>(v));
    v = max(v, dpp0<0x118, 0xf>(v));
    v = max(v, dpp0<0x142, 0xa>(v));
    v = max(v, dpp0<0x143, 0xc>(v));
    return __builtin_amdgcn_readlane(v, 63);
}

constexpr int LDS_BYTES = 65536;
constexpr int SND_LDS = 2048;

__device__ __forceinline__ void sender(const Params& p, char* lds, int wave) {
    const int lane = threadIdx.x & 63;
    const int wid = __builtin_amdgcn_readfirstlane(blockIdx.x * SW + wave);
    const int rank = p.rank;
    char* const myws = p.ws[rank];
    u32* const cnt = (u32*)(myws + CNT_OFF + (size_t)(blockIdx.x * 4 + wave) * 32);
    const u32 call = __builtin_amdgcn_readfirstlane(*cnt + 1);
    const size_t par = DATA_OFF + (size_t)(call & 1) * PAR_BYTES;
    STAMP(blockIdx.x * 64 + 0);

    for (int xi = wid; xi < NXU; xi += NW) {
        const int dest = (rank + 1 + xi) & (NR - 1);
        const int rb = xi >> 3;
        const unsigned char* src = p.x + (size_t)(dest * MS + rb * 32) * KS
                                 + (lane & 15) * KS + ((lane >> 4) & 1) * 32 + (lane >> 5) * 16;
        u32x4 v[16];
#pragma unroll
        for (int s = 0; s < 16; ++s)
            v[s] = *(const u32x4*)(src + (s & 1) * 16 * KS + (s >> 1) * 64);
        char* base = uni(p.ws[dest] + par + (size_t)(rank * 16 + rb) * 16384);
        const u32 off = lane * 16;
#pragma unroll
        for (int s = 0; s < 16; s += 4) {
            st16_sys<0>(base + s * 1024, off, v[s]);
            st16_sys<1024>(base + s * 1024, off, v[s + 1]);
            st16_sys<2048>(base + s * 1024, off, v[s + 2]);
            st16_sys<3072>(base + s * 1024, off, v[s + 3]);
        }
        wait_vm0();
        if (lane == 0)
            __hip_atomic_store((u32*)(p.ws[dest] + XFLAG_OFF + (size_t)(rank * 16 + rb) * 128), call,
                               __ATOMIC_RELAXED, __HIP_MEMORY_SCOPE_SYSTEM);
    }
    STAMP(blockIdx.x * 64 + 1);

    char* dst[NR];
    const int rot = wid & (NR - 1);
#pragma unroll
    for (int d = 0; d < NR; ++d) dst[d] = uni(p.ws[(rank + 1 + d + rot) & (NR - 1)] + par);
    char* const flagbase = p.ws[lane & (NR - 1)] + WFLAG_OFF + (size_t)rank * 128;

    unsigned short* const list = (unsigned short*)(lds + wave * SND_LDS);
    u32* const hist = (u32*)(lds + wave * SND_LDS + 1280);
    const int q = lane & 15, g = lane >> 4;

    for (int u = wid; u < NU; u += NW) {
        const int j = u / NC, c = u % NC;
        const unsigned char* src = p.w + (size_t)(j * 64 + g * 16) * NN + c * 64 + q * 4;
        u32 r[16];
#pragma unroll
        for (int i = 0; i < 16; ++i) r[i] = *(const u32*)(src + (size_t)i * NN);

        if (lane < 40) hist[lane] = 0;
        if (lane < 32) ((u32*)list)[lane] = 0;
        asm volatile("" ::: "memory");
        __hip_atomic_fetch_add(&hist[(r[0] >> 2) & 31], 1u, __ATOMIC_RELAXED, __HIP_MEMORY_SCOPE_WORKGROUP);
        __hip_atomic_fetch_add(&hist[(r[5] >> 10) & 31], 1u, __ATOMIC_RELAXED, __HIP_MEMORY_SCOPE_WORKGROUP);
        __hip_atomic_fetch_add(&hist[(r[10] >> 18) & 31], 1u, __ATOMIC_RELAXED, __HIP_MEMORY_SCOPE_WORKGROUP);
        __hip_atomic_fetch_add(&hist[(r[15] >> 26) & 31], 1u, __ATOMIC_RELAXED, __HIP_MEMORY_SCOPE_WORKGROUP);
        asm volatile("s_waitcnt lgkmcnt(0)" ::: "memory");
        u32 wsum = 0;
        {
            const int hl = lane & 31;
#pragma unroll
            for (int i = 0; i < 8; ++i) wsum += hist[hl + i];
        }
        const u32 key = (lane <= 24) ? (wsum * 32 + (31 - lane)) : 0u;
        const u32 b = 31 - (wave_max(key) & 31);

        u32 T[16];
#pragma unroll
        for (int t = 0; t < 4; ++t) {
            const u32 t0 = perm(r[4 * t + 1], r[4 * t + 0], 0x05010400u);
            const u32 t1 = perm(r[4 * t + 1], r[4 * t + 0], 0x07030602u);
            const u32 t2 = perm(r[4 * t + 3], r[4 * t + 2], 0x05010400u);
            const u32 t3 = perm(r[4 * t + 3], r[4 * t + 2], 0x07030602u);
            T[0 + t]  = perm(t2, t0, 0x05040100u);
            T[4 + t]  = perm(t2, t0, 0x07060302u);
            T[8 + t]  = perm(t3, t1, 0x05040100u);
            T[12 + t] = perm(t3, t1, 0x07060302u);
        }
        const u32 B4 = (b << 2) * 0x01010101u;
        const u32 NB4 = (((8 - b) & 7) << 2) * 0x01010101u;
        u32 code[16], z[16];
        u32 n = 0;
#pragma unroll
        for (int i = 0; i < 16; ++i) {
            const u32 d4 = ((T[i] & 0x1C1C1C1Cu) + NB4) & 0x1C1C1C1Cu;
            code[i] = (T[i] & 0x83838383u) | d4;
            z[i] = T[i] ^ (code[i] + B4);
            n += __builtin_popcount((z[i] | (z[i] >> 1)) & 0x20202020u);
        }
        const u32 incl = wave_incl_scan(n);
        const u32 total = __builtin_amdgcn_readlane(incl, 63);
        const u32 slotoff = (u32)(X_BYTES + (size_t)(u * NR + rank) * SLOT);
        const u32 off = slotoff + 128 + lane * 16;

        if (total <= MAX_EXC) {
            u32 idx = 1 + incl - n;
            const u32 posb = (g & 1) * 512 + (g >> 1) * 256 + q * 16;
#pragma unroll
            for (int i = 0; i < 16; ++i) {
                u32 zz = z[i];
                while (zz) {
                    const u32 sh = __builtin_ctz(zz) & ~7u;
                    const u32 pos = posb + (i >> 2) * 1024 + (i & 3) * 4 + (sh >> 3);
                    list[idx++] = (unsigned short)((pos << 2) | ((zz >> (sh + 5)) & 3));
                    zz &= ~(0xFFu << sh);
                }
            }
            if (lane == 0) list[0] = (unsigned short)(b | (total << 6));
            u32 cw[12];
#pragma unroll
            for (int h = 0; h < 2; ++h) {
                const u32* cd = code + 8 * h;
                cw[6 * h + 0] = cd[0] | ((cd[6] & 0x03030303u) << 5);
                cw[6 * h + 1] = cd[1] | ((cd[6] & 0x0C0C0C0Cu) << 3);
                cw[6 * h + 2] = cd[2] | ((cd[6] & 0x10101010u) << 1) | ((cd[6] & 0x80808080u) >> 1);
                cw[6 * h + 3] = cd[3] | ((cd[7] & 0x03030303u) << 5);
                cw[6 * h + 4] = cd[4] | ((cd[7] & 0x0C0C0C0Cu) << 3);
                cw[6 * h + 5] = cd[5] | ((cd[7] & 0x10101010u) << 1) | ((cd[7] & 0x80808080u) >> 1);
            }
            asm volatile("s_waitcnt lgkmcnt(0)" ::: "memory");
            const u32x4 hv = *(const u32x4*)((const char*)list + (lane & 7) * 16);
            const u32x4 c0 = {cw[0], cw[1], cw[2], cw[3]};
            const u32x4 c1 = {cw[4], cw[5], cw[6], cw[7]};
            const u32x4 c2 = {cw[8], cw[9], cw[10], cw[11]};
#pragma unroll
            for (int d = 0; d < NR; ++d) st16_sys<0>(dst[d], off, c0);
#pragma unroll
            for (int d = 0; d < NR; ++d) st16_sys<1024>(dst[d], off, c1);
#pragma unroll
            for (int d = 0; d < NR; ++d) st16_sys<2048>(dst[d], off, c2);
            if (lane < 8) {
#pragma unroll
                for (int d = 0; d < NR; ++d) st16_sys<0>(dst[d], slotoff + lane * 16, hv);
            }
            if (total > 63) {
                const u32x4 ov = *(const u32x4*)((const char*)list + 128 + (lane < 56 ? lane : 0) * 16);
                if ((u32)lane * 8 < total - 63) {
#pragma unroll
                    for (int d = 0; d < NR; ++d) st16_sys<3072>(dst[d], off, ov);
                }
            }
        } else {
            const u32x4 hv = {(lane == 0) ? 32u : 0u, 0u, 0u, 0u};
#pragma unroll
            for (int f = 0; f < 4; ++f) {
                const u32x4 v = {T[4 * f], T[4 * f + 1], T[4 * f + 2], T[4 * f + 3]};
#pragma unroll
                for (int d = 0; d < NR; ++d) st16_sys<0>(dst[d] + f * 1024, off, v);
            }
            if (lane < 8) {
#pragma unroll
                for (int d = 0; d < NR; ++d) st16_sys<0>(dst[d], slotoff + lane * 16, hv);
            }
        }
        wait_vm0();
        if (lane < NR)
            __hip_atomic_store((u32*)(flagbase + (size_t)u * NR * 128), call,
                               __ATOMIC_RELAXED, __HIP_MEMORY_SCOPE_SYSTEM);
        STAMP(blockIdx.x * 64 + 2 + (u / NW));
    }
    if (lane == 0) *cnt = call;
}

__device__ __forceinline__ void decode_unit(char* img, __amdgpu_buffer_rsrc_t rs, u32 slot, int lane,
                                            const u32x4 (&cw)[3], u32 ent) {
    const u32 hdr = __builtin_amdgcn_readfirstlane(ent) & 0xFFFFu;
    const int q = lane & 15, g = lane >> 4;
    char* const wp = img + (g & 1) * 512 + (g >> 1) * 256 + q * 16;
    if (!(hdr & 32)) {
        const u32 B4 = ((hdr & 31) << 2) * 0x01010101u;
        const u32 c[12] = {cw[0].x, cw[0].y, cw[0].z, cw[0].w, cw[1].x, cw[1].y, cw[1].z, cw[1].w,
                           cw[2].x, cw[2].y, cw[2].z, cw[2].w};
#pragma unroll
        for (int h = 0; h < 2; ++h) {
            const u32* cc = c + 6 * h;
            u32x4 a, bq;
            a.x = (cc[0] & 0x9F9F9F9Fu) + B4;
            a.y = (cc[1] & 0x9F9F9F9Fu) + B4;
            a.z = (cc[2] & 0x9F9F9F9Fu) + B4;
            a.w = (cc[3] & 0x9F9F9F9Fu) + B4;
            bq.x = (cc[4] & 0x9F9F9F9Fu) + B4;
            bq.y = (cc[5] & 0x9F9F9F9Fu) + B4;
            bq.z = (((cc[0] >> 5) & 0x03030303u) | ((cc[1] >> 3) & 0x0C0C0C0Cu) |
                    ((cc[2] >> 1) & 0x10101010u) | ((cc[2] << 1) & 0x80808080u)) + B4;
            bq.w = (((cc[3] >> 5) & 0x03030303u) | ((cc[4] >> 3) & 0x0C0C0C0Cu) |
                    ((cc[5] >> 1) & 0x10101010u) | ((cc[5] << 1) & 0x80808080u)) + B4;
            *(u32x4*)(wp + (2 * h) * 1024) = a;
            *(u32x4*)(wp + (2 * h + 1) * 1024) = bq;
        }
        const u32 total = hdr >> 6;
        {
            const u32 e = (lane == 0) ? 0u : (ent & 0xFFFFu);
            const u32 pos = e >> 2;
            const u32 x = (e & 3) << (5 + 8 * (pos & 3));
            __hip_atomic_fetch_xor((u32*)(img + (pos & ~3u)), x, __ATOMIC_RELAXED, __HIP_MEMORY_SCOPE_WORKGROUP);
        }
        for (u32 i0 = 63; i0 < total; i0 += 64) {
            const u32 i = i0 + lane;
            u32 e = __builtin_amdgcn_raw_buffer_load_b16(rs, slot + 128 + 3072 + (i - 63) * 2, 0, AUX_SYS);
            e = (i < total) ? (e & 0xFFFFu) : 0u;
            const u32 pos = e >> 2;
            const u32 x = (e & 3) << (5 + 8 * (pos & 3));
            __hip_atomic_fetch_xor((u32*)(img + (pos & ~3u)), x, __ATOMIC_RELAXED, __HIP_MEMORY_SCOPE_WORKGROUP);
        }
    } else {
        const u32x4 v3 = __builtin_amdgcn_raw_buffer_load_b128(rs, slot + 128 + 3072 + lane * 16, 0, AUX_SYS);
        *(u32x4*)(wp + 0) = cw[0];
        *(u32x4*)(wp + 1024) = cw[1];
        *(u32x4*)(wp + 2048) = cw[2];
        *(u32x4*)(wp + 3072) = v3;
    }
}

__device__ __forceinline__ void consumer(const Params& p, char* lds) {
    const int lane = threadIdx.x & 63;
    const int wave = __builtin_amdgcn_readfirstlane(threadIdx.x >> 6);
    const int t = blockIdx.x - NS;
    const int c = (t & 7) * 8 + ((t >> 3) & 7);
    const int m = t >> 6;
    const int rb = m * 4 + wave;
    char* const myws = p.ws[p.rank];
    u32* const cnt = (u32*)(myws + CNT_OFF + (size_t)(blockIdx.x * 4 + wave) * 32);
    u32* const tmo = (u32*)(myws + TMO_OFF);
    const u32 call = __builtin_amdgcn_readfirstlane(*cnt + 1);
    const size_t par = DATA_OFF + (size_t)(call & 1) * PAR_BYTES;
    const __amdgpu_buffer_rsrc_t rs =
        __builtin_amdgcn_make_buffer_rsrc(myws + par, 0, (int)(X_BYTES + W_BYTES), 0x00020000);
    STAMP(blockIdx.x * 64 + wave * 16 + 0);

    f32x4 acc[2][4];
#pragma unroll
    for (int a = 0; a < 2; ++a)
#pragma unroll
        for (int f = 0; f < 4; ++f) acc[a][f] = f32x4{0.f, 0.f, 0.f, 0.f};

    wait_flags((const u32*)(myws + XFLAG_OFF + (size_t)((lane & 7) * 16 + rb) * 128), lane < NR, call, tmo);
    STAMP(blockIdx.x * 64 + wave * 16 + 1);

    const u32 xoff = (u32)(((lane >> 5) * 16 + rb) * 16384 + (lane & 31) * 16);
    const int L16 = lane & 15, G = lane >> 4;
    const u32 bread = (G >> 1) * 4096 + (G & 1) * 256 + L16 * 16;

    for (int j = 0; j < NJ; ++j) {
        u32x4 A[4][2][2];
#pragma unroll
        for (int P = 0; P < 4; ++P)
#pragma unroll
            for (int a = 0; a < 2; ++a)
#pragma unroll
                for (int i = 0; i < 2; ++i)
                    A[P][a][i] = __builtin_amdgcn_raw_buffer_load_b128(
                        rs, xoff + j * 2048 + a * 1024 + i * 512, P * 2 * 16 * 16384, AUX_SYS);

        const int u = j * NC + c;
        wait_flags((const u32*)(myws + WFLAG_OFF + (size_t)(u * NR + 2 * wave + (lane & 1)) * 128),
                   lane < 2, call, tmo);
        char* const buf = lds + (j & 1) * 32768;
        u32x4 cw[2][3];
        u32 ent[2];
#pragma unroll
        for (int s = 0; s < 2; ++s) {
            const u32 slot = (u32)(X_BYTES + (size_t)(u * NR + 2 * wave + s) * SLOT);
#pragma unroll
            for (int i = 0; i < 3; ++i)
                cw[s][i] = __builtin_amdgcn_raw_buffer_load_b128(rs, slot + 128 + i * 1024 + lane * 16, 0, AUX_SYS);
            ent[s] = __builtin_amdgcn_raw_buffer_load_b16(rs, slot + lane * 2, 0, AUX_SYS);
        }
#pragma unroll
        for (int s = 0; s < 2; ++s) {
            const u32 slot = (u32)(X_BYTES + (size_t)(u * NR + 2 * wave + s) * SLOT);
            decode_unit(buf + (2 * wave + s) * 4096, rs, slot, lane, cw[s], ent[s]);
        }
        __syncthreads();
#pragma unroll
        for (int P = 0; P < 4; ++P) {
#pragma unroll
            for (int f = 0; f < 4; ++f) {
                const char* bp = buf + P * 8192 + f * 1024 + bread;
                const u32x4 b0 = *(const u32x4*)(bp);
                const u32x4 b1 = *(const u32x4*)(bp + 512);
                const i32x8 B = {(int)b0.x, (int)b0.y, (int)b0.z, (int)b0.w, (int)b1.x, (int)b1.y, (int)b1.z, (int)b1.w};
#pragma unroll
                for (int a = 0; a < 2; ++a) {
                    const u32x4 a0 = A[P][a][0], a1 = A[P][a][1];
                    const i32x8 Av = {(int)a0.x, (int)a0.y, (int)a0.z, (int)a0.w, (int)a1.x, (int)a1.y, (int)a1.z, (int)a1.w};
                    acc[a][f] = __builtin_amdgcn_mfma_scale_f32_16x16x128_f8f6f4(Av, B, acc[a][f], 0, 1, 0, 0, 0, 0);
                }
            }
        }
        STAMP(blockIdx.x * 64 + wave * 16 + 2 + j);
    }

    if (lane == 0) *cnt = call;
    const bool bad = __hip_atomic_load(tmo, __ATOMIC_RELAXED, __HIP_MEMORY_SCOPE_AGENT) == call;
    const float s = bad ? __builtin_nanf("") : (p.sx[0] * p.sw[0]);
    float* const orow = p.out + (size_t)(m * 128 + wave * 32 + G * 4) * NN + c * 64 + L16 * 4;
#pragma unroll
    for (int a = 0; a < 2; ++a)
#pragma unroll
        for (int r = 0; r < 4; ++r) {
            f32x4 v = {acc[a][0][r] * s, acc[a][1][r] * s, acc[a][2][r] * s, acc[a][3][r] * s};
            *(f32x4*)(orow + (size_t)(a * 16 + r) * NN) = v;
        }
    STAMP(blockIdx.x * 64 + wave * 16 + 10);
}

__global__ void __launch_bounds__(256, 2) gemm_rs_kernel(Params p) {
    __shared__ __attribute__((aligned(16))) char lds[LDS_BYTES];
    if (blockIdx.x < NS) {
        const int wave = threadIdx.x >> 6;
        if (wave < SW) sender(p, lds, wave);
    } else {
        consumer(p, lds);
    }
}

}

extern "C" void kernel_launch(int rank, int nranks, ncclComm_t comm, hipStream_t stream,
                              void* const* d_in, const int* in_sizes, int n_in,
                              void* d_out, int out_size, const int* in_dtypes, int out_dtype,
                              void* const* d_out_all, void* const* d_ws_all, size_t ws_bytes) {
    (void)comm; (void)in_dtypes; (void)out_dtype; (void)d_out_all;
    if (nranks != NR || n_in != 4 || rank < 0 || rank >= NR || ws_bytes < WS_NEED ||
        in_sizes[0] != 4096 * KS || in_sizes[1] != KS * NN || out_size != MS * NN) {
        fprintf(stderr, "gemm_rs: unexpected problem shape\n");
        return;
    }
    Params p;
    p.x = (const unsigned char*)d_in[0];
    p.w = (const unsigned char*)d_in[1];
    p.sx = (const float*)d_in[2];
    p.sw = (const float*)d_in[3];
    p.out = (float*)d_out;
    for (int r = 0; r < NR; ++r) p.ws[r] = (char*)d_ws_all[r];
    p.rank = rank;
    p.pad = 0;
    hipLaunchKernelGGL(gemm_rs_kernel, dim3(GRID), dim3(256), 0, stream, p);
    HIP_CHECK(hipGetLastError());
}
```

```cpp
#include <hip/hip_runtime.h>
#include <rccl/rccl.h>
#include <cstdio>
#include <cstdint>

#define HIP_CHECK(expr) do { hipError_t _e = (expr); if (_e != hipSuccess) { \
    fprintf(stderr, "HIP error %s at %s:%d\n", hipGetErrorString(_e), __FILE__, __LINE__); \
    return; } } while (0)

namespace {

typedef unsigned u32;
typedef unsigned long long u64;
typedef __attribute__((address_space(1))) unsigned gu32;
typedef unsigned u32x4 __attribute__((ext_vector_type(4)));
typedef int i32x8 __attribute__((ext_vector_type(8)));
typedef float f32x4 __attribute__((ext_vector_type(4)));

constexpr int NR = 8;
constexpr int MS = 512;
constexpr int KS = 512;
constexpr int NN = 4096;
constexpr int NJ = 8;
constexpr int NC = 64;
constexpr int NU = NJ * NC;

#define NSEND 64
#define SWAVES 1
constexpr int NS = NSEND;
constexpr int SW = SWAVES;
#define NWAVES 32
constexpr int NW = NWAVES;
constexpr int NCONS = 192;
constexpr int GRID = NS + NCONS;
static_assert(GRID == 256 && SW <= 3, "one workgroup per CU, 256 x half-units");
static_assert(NW <= NU, "every sender wave takes at least one w unit");

constexpr size_t CNT_OFF   = 0;
constexpr size_t TMO_OFF   = 64 << 10;
constexpr size_t TICK_OFF  = 96 << 10;
constexpr size_t XFLAG_OFF = 128 << 10;
constexpr size_t WFLAG_OFF = 256 << 10;
constexpr size_t DATA_OFF  = 1 << 20;
constexpr size_t PAR_BYTES = 32u << 20;
constexpr size_t X_BYTES   = 2u << 20;
constexpr u32    SLOT      = 4224;
constexpr size_t W_BYTES   = (size_t)NU * NR * SLOT;
constexpr size_t WS_NEED   = DATA_OFF + PAR_BYTES + X_BYTES + W_BYTES;
static_assert(X_BYTES + W_BYTES <= PAR_BYTES, "parity region too small");
static_assert((size_t)GRID * 4 * 32 <= TMO_OFF, "counter region too small");
static_assert(WFLAG_OFF + (size_t)NU * NR * 128 <= DATA_OFF, "flag region too small");

constexpr u64 DEADLINE = 40000000ull;
constexpr int AUX_SYS = 17;
#define A_DB 1
#define OUT_AUX 16
constexpr u32 MAX_EXC = 511;
constexpr u32 REPICK = 127;

struct Params {
    const unsigned char* x;
    const unsigned char* w;
    const float* sx;
    const float* sw;
    float* out;
    char* ws[NR];
    int rank;
    int pad;
};

__device__ __forceinline__ u32 perm(u32 hi, u32 lo, u32 sel) { return __builtin_amdgcn_perm(hi, lo, sel); }

template <int IMM>
__device__ __forceinline__ void st16_sys(char* base, u32 off, u32x4 v) {
    asm volatile("global_store_dwordx4 %0, %1, %2 offset:%3 sc0 sc1\n\ts_nop 1"
                 :: "v"(off), "v"(v), "s"(base), "i"(IMM) : "memory");
}
__device__ __forceinline__ char* uni(char* p) {
    const u64 v = (u64)p;
    const u32 lo = __builtin_amdgcn_readfirstlane((u32)v);
    const u32 hi = __builtin_amdgcn_readfirstlane((u32)(v >> 32));
    char* r = (char*)(((u64)hi << 32) | lo);
    asm volatile("s_nop 4" : "+s"(r));
    return r;
}
__device__ __forceinline__ void wait_vm0() { asm volatile("s_waitcnt vmcnt(0)" ::: "memory"); }

#define STAMP(slot) do {} while (0)
#define STAMP_HW(slot) do {} while (0)

__device__ __forceinline__ bool wait_flags(const u32* flag, bool active, u32 call, u32* tmo) {
    const u64 t0 = __builtin_amdgcn_s_memrealtime();
    bool bad = false;
    for (u32 it = 0;; ++it) {
        u32 v = call;
        if (active) v = __hip_atomic_load(flag, __ATOMIC_RELAXED, __HIP_MEMORY_SCOPE_SYSTEM);
        if (__all(v >= call)) break;
        if ((it & 15) == 15) {
            if (__hip_atomic_load(tmo, __ATOMIC_RELAXED, __HIP_MEMORY_SCOPE_AGENT) == call ||
                __builtin_amdgcn_s_memrealtime() - t0 > DEADLINE) {
                __hip_atomic_store(tmo, call, __ATOMIC_RELAXED, __HIP_MEMORY_SCOPE_AGENT);
                bad = true;
                break;
            }
        }
        __builtin_amdgcn_s_sleep(2);
    }
    asm volatile("" ::: "memory");
    return bad;
}

template <int CTRL, int RMASK>
__device__ __forceinline__ u32 dpp0(u32 v) {
    return __builtin_amdgcn_update_dpp(0u, v, CTRL, RMASK, 0xf, false);
}
__device__ __forceinline__ u32 wave_incl_scan(u32 v) {
    v += dpp0<0x111, 0xf>(v);
    v += dpp0<0x112, 0xf>(v);
    v += dpp0<0x114, 0xf>(v);
    v += dpp0<0x118, 0xf>(v);
    v += dpp0<0x142, 0xa>(v);
    v += dpp0<0x143, 0xc>(v);
    return v;
}
__device__ __forceinline__ u32 wave_max(u32 v) {
    v = max(v, dpp0<0x111, 0xf>(v));
    v = max(v, dpp0<0x112, 0xf>(v));
    v = max(v, dpp0<0x114, 0xf>(v));
    v = max(v, dpp0<0x118, 0xf>(v));
    v = max(v, dpp0<0x142, 0xa>(v));
    v = max(v, dpp0<0x143, 0xc>(v));
    return __builtin_amdgcn_readlane(v, 63);
}

constexpr int LDS_BYTES = 96 << 10;
constexpr int SND_LDS = 10240;

struct Enc {
    u32x4 c0, c1, c2, c3;
    u32x4 hv;
    u32 total;
};

typedef __attribute__((address_space(3))) char lchar;
__device__ __forceinline__ u32 lds_addr(char* p) { return (u32)(u64)(lchar*)p; }

__device__ __forceinline__ void dma_raw(const unsigned char* src, u32 ldsdst) {
    u32 keep;
    const unsigned char* s1 = src + 16 * NN;
    const unsigned char* s2 = src + 32 * NN;
    const unsigned char* s3 = src + 48 * NN;
    asm volatile("s_mov_b32 %0, m0\n\t"
                 "s_mov_b32 m0, %5\n\ts_nop 0\n\tglobal_load_lds_dwordx4 %1, off\n\t"
                 "s_add_u32 m0, m0, 0x400\n\ts_nop 0\n\tglobal_load_lds_dwordx4 %2, off\n\t"
                 "s_add_u32 m0, m0, 0x400\n\ts_nop 0\n\tglobal_load_lds_dwordx4 %3, off\n\t"
                 "s_add_u32 m0, m0, 0x400\n\ts_nop 0\n\tglobal_load_lds_dwordx4 %4, off\n\t"
                 "s_mov_b32 m0, %0"
                 : "=&s"(keep) : "v"(src), "v"(s1), "v"(s2), "v"(s3), "s"(ldsdst) : "memory", "scc");
}
__device__ __forceinline__ void draw_ticket(u32& t, u32* ctr) {
    u64 save;
    asm volatile("s_mov_b64 %1, exec\n\ts_mov_b64 exec, 1\n\tglobal_atomic_add %0, %2, %3, %4 sc0\n\ts_mov_b64 exec, %1"
                 : "=&v"(t), "=&s"(save) : "v"(0u), "v"(1u), "s"(ctr) : "memory");
}
template <int N>
__device__ __forceinline__ void wait_tk(u32& t) {
    asm volatile("s_waitcnt vmcnt(%1)" : "+v"(t) : "n"(N) : "memory");
}

__device__ __forceinline__ u32 pick_base(const u32 (&r)[16], u32* hist, int lane) {
    if (lane < 40) hist[lane] = 0;
    asm volatile("" ::: "memory");
    __hip_atomic_fetch_add(&hist[(r[0] >> 2) & 31], 1u, __ATOMIC_RELAXED, __HIP_MEMORY_SCOPE_WORKGROUP);
    __hip_atomic_fetch_add(&hist[(r[5] >> 10) & 31], 1u, __ATOMIC_RELAXED, __HIP_MEMORY_SCOPE_WORKGROUP);
    __hip_atomic_fetch_add(&hist[(r[10] >> 18) & 31], 1u, __ATOMIC_RELAXED, __HIP_MEMORY_SCOPE_WORKGROUP);
    __hip_atomic_fetch_add(&hist[(r[15] >> 26) & 31], 1u, __ATOMIC_RELAXED, __HIP_MEMORY_SCOPE_WORKGROUP);
    asm volatile("" ::: "memory");
    u32 wsum = 0;
    const int hl = lane & 31;
#pragma unroll
    for (int i = 0; i < 8; ++i) wsum += hist[hl + i];
    const u32 key = (lane <= 24) ? (wsum * 32 + (31 - lane)) : 0u;
    return 31 - (wave_max(key) & 31);
}

__device__ __forceinline__ void encode(Enc& e, const char* raw, unsigned short* list, u32* hist, int lane, u32& b) {
    const int q = lane & 15, g = lane >> 4;
    u32 r[16];
#pragma unroll
    for (int i = 0; i < 16; ++i) r[i] = *(const u32*)(raw + (g * 16 + i) * 64 + q * 4);
    if (lane < 32) ((u32*)list)[lane] = 0;
    u32 T[16];
#pragma unroll
    for (int t = 0; t < 4; ++t) {
        const u32 t0 = perm(r[4 * t + 1], r[4 * t + 0], 0x05010400u);
        const u32 t1 = perm(r[4 * t + 1], r[4 * t + 0], 0x07030602u);
        const u32 t2 = perm(r[4 * t + 3], r[4 * t + 2], 0x05010400u);
        const u32 t3 = perm(r[4 * t + 3], r[4 * t + 2], 0x07030602u);
        T[0 + t]  = perm(t2, t0, 0x05040100u);
        T[4 + t]  = perm(t2, t0, 0x07060302u);
        T[8 + t]  = perm(t3, t1, 0x05040100u);
        T[12 + t] = perm(t3, t1, 0x07060302u);
    }
    bool picked = false;
    if (b >= 32) { b = pick_base(r, hist, lane); picked = true; }
    u32 code[16], D[4], n, incl, total;
    while (true) {
        const u32 B4 = (b << 2) * 0x01010101u;
        const u32 NB4 = (((8 - b) & 7) << 2) * 0x01010101u;
        n = 0;
#pragma unroll
        for (int k = 0; k < 4; ++k) {
            u32 v[4];
#pragma unroll
            for (int t = 0; t < 4; ++t) {
                const int i = 4 * k + t;
                const u32 d4 = ((T[i] & 0x1C1C1C1Cu) + NB4) & 0x1C1C1C1Cu;
                code[i] = (T[i] & 0x83838383u) | d4;
                const u32 z = T[i] ^ (code[i] + B4);
                v[t] = __builtin_amdgcn_udot4(z, 0x40100401u, 0u, false);
            }
            D[k] = (v[0] >> 5) | (v[1] << 3) | (v[2] << 11) | (v[3] << 19);
            n += __builtin_popcount((D[k] | (D[k] >> 1)) & 0x55555555u);
        }
        incl = wave_incl_scan(n);
        total = __builtin_amdgcn_readlane(incl, 63);
        if (total <= REPICK || picked) break;
        picked = true;
        const u32 b2 = pick_base(r, hist, lane);
        if (b2 == b) break;
        b = b2;
    }
    e.total = total;
    if (total <= MAX_EXC) {
        u32 idx = 1 + incl - n;
        const u32 posb = ((g & 1) * 512 + (g >> 1) * 256 + q * 16) << 2;
#pragma unroll
        for (int k = 0; k < 4; ++k) {
            u32 dd = D[k];
            while (dd) {
                const u32 bit = __builtin_ctz(dd) & ~1u;
                list[idx++] = (unsigned short)(posb + k * 4096 + (bit << 1) + ((dd >> bit) & 3));
                dd &= ~(3u << bit);
            }
        }
        if (lane == 0) list[0] = (unsigned short)(b | (total << 6));
        u32 cw[12];
#pragma unroll
        for (int h = 0; h < 2; ++h) {
            const u32* cd = code + 8 * h;
            cw[6 * h + 0] = cd[0] | ((cd[6] & 0x03030303u) << 5);
            cw[6 * h + 1] = cd[1] | ((cd[6] & 0x0C0C0C0Cu) << 3);
            cw[6 * h + 2] = cd[2] | ((cd[6] & 0x10101010u) << 1) | ((cd[6] & 0x80808080u) >> 1);
            cw[6 * h + 3] = cd[3] | ((cd[7] & 0x03030303u) << 5);
            cw[6 * h + 4] = cd[4] | ((cd[7] & 0x0C0C0C0Cu) << 3);
            cw[6 * h + 5] = cd[5] | ((cd[7] & 0x10101010u) << 1) | ((cd[7] & 0x80808080u) >> 1);
        }
        asm volatile("" ::: "memory");
        e.hv = *(const u32x4*)((const char*)list + (lane & 7) * 16);
        e.c3 = *(const u32x4*)((const char*)list + 128 + (lane < 56 ? lane : 0) * 16);
        e.c0 = u32x4{cw[0], cw[1], cw[2], cw[3]};
        e.c1 = u32x4{cw[4], cw[5], cw[6], cw[7]};
        e.c2 = u32x4{cw[8], cw[9], cw[10], cw[11]};
        asm volatile("" ::: "memory");
    } else {
        e.hv = u32x4{(lane == 0) ? 32u : 0u, 0u, 0u, 0u};
        e.c0 = u32x4{T[0], T[1], T[2], T[3]};
        e.c1 = u32x4{T[4], T[5], T[6], T[7]};
        e.c2 = u32x4{T[8], T[9], T[10], T[11]};
        e.c3 = u32x4{T[12], T[13], T[14], T[15]};
    }
}

__device__ __forceinline__ void store_unit(const Enc& e, char* const (&dst)[NR], u32 slotoff, int lane) {
    const u32 off = slotoff + 128 + lane * 16;
#pragma unroll
    for (int d = 0; d < NR; ++d) st16_sys<0>(dst[d], off, e.c0);
#pragma unroll
    for (int d = 0; d < NR; ++d) st16_sys<1024>(dst[d], off, e.c1);
#pragma unroll
    for (int d = 0; d < NR; ++d) st16_sys<2048>(dst[d], off, e.c2);
    if (lane < 8) {
#pragma unroll
        for (int d = 0; d < NR; ++d) st16_sys<0>(dst[d], slotoff + lane * 16, e.hv);
    }
    if (e.total > 63) {
        if (e.total > MAX_EXC || (u32)lane * 8 < e.total - 63) {
#pragma unroll
            for (int d = 0; d < NR; ++d) st16_sys<3072>(dst[d], off, e.c3);
        }
    }
}

__device__ __forceinline__ void flag_store(char* a, u32 call) {
    __hip_atomic_store((gu32*)(u64)a, call, __ATOMIC_RELAXED, __HIP_MEMORY_SCOPE_SYSTEM);
}

__device__ __forceinline__ void sender(const Params& p, char* lds, int wave) {
    const int lane = threadIdx.x & 63;
    const int wid = __builtin_amdgcn_readfirstlane(wave * NS + blockIdx.x);
    const int rank = p.rank;
    char* const myws = p.ws[rank];
    u32* const cnt = (u32*)(myws + CNT_OFF + (size_t)(blockIdx.x * 4 + wave) * 32);
    STAMP(blockIdx.x * 64 + 0);
    STAMP_HW(blockIdx.x * 64 + 15);
    char* const mylds = lds + wave * SND_LDS;
    char* const raw0 = mylds;
    unsigned short* const list = (unsigned short*)(mylds + 8192);
    u32* const hist = (u32*)(mylds + 8192 + 1280);
    const u32 rawaddr = __builtin_amdgcn_readfirstlane(lds_addr(raw0));
    const unsigned char* const wsrc = p.w + (size_t)(lane >> 2) * NN + (lane & 3) * 16;
    auto unit_src = [&](int u) { return wsrc + (size_t)(u / NC) * 64 * NN + (u % NC) * 64; };

    u32* const tick = (u32*)uni(myws + TICK_OFF);
    u32 tk;
    dma_raw(unit_src(wid), rawaddr);
    draw_ticket(tk, tick);

    const u32 call = __builtin_amdgcn_readfirstlane(*cnt + 1);
    const size_t par = DATA_OFF + (size_t)(call & 1) * PAR_BYTES;
    char* flagbase = p.ws[lane & (NR - 1)] + WFLAG_OFF + (size_t)rank * 128;
    char* dst[NR];
    const int rot = wid & (NR - 1);
#pragma unroll
    for (int d = 0; d < NR; ++d) dst[d] = uni(p.ws[(rank + 1 + d + rot) & (NR - 1)] + par);
    asm volatile("" : "+v"(flagbase));
    wait_tk<0>(tk);

    const u32 tbase = (call - 1) * (NU + NW) - NW;
    int u = wid;
    int un = (int)(__builtin_amdgcn_readfirstlane(tk) - tbase);
    int kb = 0;
    if ((u32)un < (u32)NU) dma_raw(unit_src(un), rawaddr + 4096);
    Enc e;
    u32 base = 32;
    encode(e, raw0, list, hist, lane, base);
    STAMP(blockIdx.x * 64 + 1);

    int uprev = -1;
    while (true) {
        draw_ticket(tk, tick);
        store_unit(e, dst, (u32)(X_BYTES + (size_t)(u * NR + rank) * SLOT), lane);
        wait_tk<32>(tk);
        if (uprev >= 0) {
            if (lane < NR) flag_store(flagbase + (size_t)uprev * NR * 128, call);
        }
        if ((u32)un >= (u32)NU) break;
        const int unn = (int)(__builtin_amdgcn_readfirstlane(tk) - tbase);
        if ((u32)unn < (u32)NU)
            dma_raw(unit_src(unn), __builtin_amdgcn_readfirstlane(rawaddr + kb * 4096));
        kb ^= 1;
        encode(e, raw0 + kb * 4096, list, hist, lane, base);
        uprev = u;
        u = un;
        un = unn;
    }
    wait_vm0();
    if (lane < NR) flag_store(flagbase + (size_t)u * NR * 128, call);
    if (lane == 0) *cnt = call;
}

__device__ __forceinline__ void decode_unit(char* img, __amdgpu_buffer_rsrc_t rs, u32 slot, int lane,
                                            const u32x4 (&cw)[3], u32 ent) {
    const u32 hdr = __builtin_amdgcn_readfirstlane(ent) & 0xFFFFu;
    const int q = lane & 15, g = lane >> 4;
    char* const wp = img + (g & 1) * 512 + (g >> 1) * 256 + q * 16;
    if (!(hdr & 32)) {
        const u32 B4 = ((hdr & 31) << 2) * 0x01010101u;
        const u32 c[12] = {cw[0].x, cw[0].y, cw[0].z, cw[0].w, cw[1].x, cw[1].y, cw[1].z, cw[1].w,
                           cw[2].x, cw[2].y, cw[2].z, cw[2].w};
#pragma unroll
        for (int h = 0; h < 2; ++h) {
            const u32* cc = c + 6 * h;
            u32x4 a, bq;
            a.x = (cc[0] & 0x9F9F9F9Fu) + B4;
            a.y = (cc[1] & 0x9F9F9F9Fu) + B4;
            a.z = (cc[2] & 0x9F9F9F9Fu) + B4;
            a.w = (cc[3] & 0x9F9F9F9Fu) + B4;
            bq.x = (cc[4] & 0x9F9F9F9Fu) + B4;
            bq.y = (cc[5] & 0x9F9F9F9Fu) + B4;
            bq.z = (((cc[0] >> 5) & 0x03030303u) | ((cc[1] >> 3) & 0x0C0C0C0Cu) |
                    ((cc[2] >> 1) & 0x10101010u) | ((cc[2] << 1) & 0x80808080u)) + B4;
            bq.w = (((cc[3] >> 5) & 0x03030303u) | ((cc[4] >> 3) & 0x0C0C0C0Cu) |
                    ((cc[5] >> 1) & 0x10101010u) | ((cc[5] << 1) & 0x80808080u)) + B4;
            *(u32x4*)(wp + (2 * h) * 1024) = a;
            *(u32x4*)(wp + (2 * h + 1) * 1024) = bq;
        }
        const u32 total = hdr >> 6;
        {
            const u32 e = (lane == 0) ? 0u : (ent & 0xFFFFu);
            const u32 pos = e >> 2;
            const u32 x = (e & 3) << (5 + 8 * (pos & 3));
            __hip_atomic_fetch_xor((u32*)(img + (pos & ~3u)), x, __ATOMIC_RELAXED, __HIP_MEMORY_SCOPE_WORKGROUP);
        }
        for (u32 i0 = 63; i0 < total; i0 += 64) {
            const u32 i = i0 + lane;
            u32 e = __builtin_amdgcn_raw_buffer_load_b16(rs, slot + 128 + 3072 + (i - 63) * 2, 0, AUX_SYS);
            e = (i < total) ? (e & 0xFFFFu) : 0u;
            const u32 pos = e >> 2;
            const u32 x = (e & 3) << (5 + 8 * (pos & 3));
            __hip_atomic_fetch_xor((u32*)(img + (pos & ~3u)), x, __ATOMIC_RELAXED, __HIP_MEMORY_SCOPE_WORKGROUP);
        }
    } else {
        const u32x4 v3 = __builtin_amdgcn_raw_buffer_load_b128(rs, slot + 128 + 3072 + lane * 16, 0, AUX_SYS);
        *(u32x4*)(wp + 0) = cw[0];
        *(u32x4*)(wp + 1024) = cw[1];
        *(u32x4*)(wp + 2048) = cw[2];
        *(u32x4*)(wp + 3072) = v3;
    }
}

__device__ __forceinline__ void send_x(const Params& p, u32 call, size_t par, int b, int lane) {
    const int dest = __builtin_amdgcn_readfirstlane((p.rank + 1 + b) & (NR - 1));
    const int rf = __builtin_amdgcn_readfirstlane((b >> 3) & 31);
    const unsigned char* src = p.x + (size_t)(dest * MS + rf * 16 + (lane & 15)) * KS
                             + ((lane >> 4) & 1) * 32 + (lane >> 5) * 16;
    u32x4 v[8];
#pragma unroll
    for (int j = 0; j < 8; ++j) v[j] = *(const u32x4*)(src + j * 64);
    char* base = uni(p.ws[dest] + par + (size_t)(p.rank * 32 + rf) * 8192);
    char* xf = uni(p.ws[dest] + XFLAG_OFF + (size_t)(p.rank * 32 + rf) * 128);
#pragma unroll
    for (int j = 0; j < 8; ++j) asm volatile("" :: "v"(v[j]));
#pragma unroll
    for (int j = 0; j < 8; j += 4) {
        st16_sys<0>(base + j * 1024, lane * 16, v[j]);
        st16_sys<1024>(base + j * 1024, lane * 16, v[j + 1]);
        st16_sys<2048>(base + j * 1024, lane * 16, v[j + 2]);
        st16_sys<3072>(base + j * 1024, lane * 16, v[j + 3]);
    }
    wait_vm0();
    if (lane == 0) __hip_atomic_store((gu32*)(u64)xf, call, __ATOMIC_RELAXED, __HIP_MEMORY_SCOPE_SYSTEM);
}

constexpr int NA = 3;

__device__ __forceinline__ void consumer(const Params& p, char* lds) {
    const int lane = threadIdx.x & 63;
    const int wave = __builtin_amdgcn_readfirstlane(threadIdx.x >> 6);
    const int t = blockIdx.x - NS;
    const int c = (t & 7) * 8 + ((t >> 3) & 7);
    const int m = t >> 6;
    const int rf0 = m * 11 + ((m == 2 && wave == 3) ? 8 : wave * 3);
    const int nf = (wave == 3 || (m == 2 && wave == 2)) ? 2 : 3;
    char* const myws = p.ws[p.rank];
    u32* const cnt = (u32*)(myws + CNT_OFF + (size_t)(blockIdx.x * 4 + wave) * 32);
    u32* const tmo = (u32*)(myws + TMO_OFF);
    const u32 call = __builtin_amdgcn_readfirstlane(*cnt + 1);
    const float scale = p.sx[0] * p.sw[0];
    const size_t par = DATA_OFF + (size_t)(call & 1) * PAR_BYTES;
    const __amdgpu_buffer_rsrc_t rs =
        __builtin_amdgcn_make_buffer_rsrc(myws + par, 0, (int)(X_BYTES + W_BYTES), 0x00020000);
    u32* const badw = (u32*)(lds + 65536);
    STAMP(blockIdx.x * 64 + wave * 16 + 0);
    STAMP_HW(blockIdx.x * 64 + wave * 16 + 15);

    f32x4 acc[NA][4];
#pragma unroll
    for (int a = 0; a < NA; ++a)
#pragma unroll
        for (int f = 0; f < 4; ++f) acc[a][f] = f32x4{0.f, 0.f, 0.f, 0.f};

    if (wave == (t & 3)) send_x(p, call, par, blockIdx.x, lane);

    bool bad;
    {
        const int a = lane >> 3;
        const int rf = rf0 + (a < nf ? a : nf - 1);
        bad = wait_flags((const u32*)(myws + XFLAG_OFF + (size_t)((lane & 7) * 32 + rf) * 128), lane < NA * NR, call, tmo);
    }
    STAMP(blockIdx.x * 64 + wave * 16 + 1);

    const u32 xoff = (u32)(((lane >> 5) * 32 + rf0) * 8192 + (lane & 31) * 16);
    const u32 xlast = (nf - 1) * 8192;
    const int L16 = lane & 15, G = lane >> 4;
    const u32 bread = (G >> 1) * 4096 + (G & 1) * 256 + L16 * 16;

    u32x4 A0[4 * NA * 2], A1[4 * NA * 2];
    auto load_a = [&](u32x4 (&A)[4 * NA * 2], int j) {
#pragma unroll
        for (int P = 0; P < 4; ++P)
#pragma unroll
            for (int a = 0; a < NA; ++a)
#pragma unroll
                for (int i = 0; i < 2; ++i)
                    A[(P * NA + a) * 2 + i] = __builtin_amdgcn_raw_buffer_load_b128(
                        rs, xoff + (a == NA - 1 ? xlast : a * 8192) + j * 1024 + i * 512, P * 2 * 32 * 8192, AUX_SYS);
    };
    auto round = [&](int j, const u32x4 (&A)[4 * NA * 2], u32x4 (&An)[4 * NA * 2]) {
        const int u = j * NC + c;
        bad |= wait_flags((const u32*)(myws + WFLAG_OFF + (size_t)(u * NR + 2 * wave + (lane & 1)) * 128),
                          lane < 2, call, tmo);
        char* const buf = lds + (j & 1) * 32768;
        u32x4 cw[2][3];
        u32 ent[2];
#pragma unroll
        for (int s = 0; s < 2; ++s) {
            const u32 slot = (u32)(X_BYTES + (size_t)(u * NR + 2 * wave + s) * SLOT);
            ent[s] = __builtin_amdgcn_raw_buffer_load_b16(rs, slot + lane * 2, 0, AUX_SYS);
#pragma unroll
            for (int i = 0; i < 3; ++i)
                cw[s][i] = __builtin_amdgcn_raw_buffer_load_b128(rs, slot + 128 + i * 1024 + lane * 16, 0, AUX_SYS);
        }
#pragma unroll
        for (int s = 0; s < 2; ++s) {
            const u32 slot = (u32)(X_BYTES + (size_t)(u * NR + 2 * wave + s) * SLOT);
            decode_unit(buf + (2 * wave + s) * 4096, rs, slot, lane, cw[s], ent[s]);
        }
        if (lane == 0) badw[wave] = bad;
        __syncthreads();
        if (j + 1 < NJ) load_a(An, j + 1);
        u32x4 Bf[2][2][2];
#pragma unroll
        for (int f = 0; f < 2; ++f) {
            Bf[0][f][0] = *(const u32x4*)(buf + f * 1024 + bread);
            Bf[0][f][1] = *(const u32x4*)(buf + f * 1024 + bread + 512);
        }
#pragma unroll
        for (int st = 0; st < 8; ++st) {
            if (st < 7) {
                const int Pn = (st + 1) >> 1, fn = ((st + 1) & 1) * 2;
#pragma unroll
                for (int f = 0; f < 2; ++f) {
                    Bf[(st + 1) & 1][f][0] = *(const u32x4*)(buf + Pn * 8192 + (fn + f) * 1024 + bread);
                    Bf[(st + 1) & 1][f][1] = *(const u32x4*)(buf + Pn * 8192 + (fn + f) * 1024 + bread + 512);
                }
            }
            __builtin_amdgcn_sched_barrier(0);
            const int P = st >> 1, f0 = (st & 1) * 2;
#pragma unroll
            for (int f = 0; f < 2; ++f) {
                const u32x4 b0 = Bf[st & 1][f][0], b1 = Bf[st & 1][f][1];
                const i32x8 B = {(int)b0.x, (int)b0.y, (int)b0.z, (int)b0.w, (int)b1.x, (int)b1.y, (int)b1.z, (int)b1.w};
#pragma unroll
                for (int a = 0; a < NA; ++a) {
                    const u32x4 a0 = A[(P * NA + a) * 2], a1 = A[(P * NA + a) * 2 + 1];
                    const i32x8 Av = {(int)a0.x, (int)a0.y, (int)a0.z, (int)a0.w, (int)a1.x, (int)a1.y, (int)a1.z, (int)a1.w};
                    acc[a][f0 + f] = __builtin_amdgcn_mfma_scale_f32_16x16x128_f8f6f4(Av, B, acc[a][f0 + f], 0, 1, 0, 0, 0, 0);
                }
            }
            __builtin_amdgcn_sched_barrier(0);
        }
        STAMP(blockIdx.x * 64 + wave * 16 + 2 + j);
    };
    load_a(A0, 0);
    for (int j = 0; j < NJ; j += 2) {
        round(j, A0, A1);
        round(j + 1, A1, A0);
    }

    const u32x4 bw = *(const u32x4*)badw;
    const float s = (bw.x | bw.y | bw.z | bw.w) ? __builtin_nanf("") : scale;
    const __amdgpu_buffer_rsrc_t ro = __builtin_amdgcn_make_buffer_rsrc(p.out, 0, MS * NN * 4, 0x00020000);
    const u32 obase = (u32)(((rf0 * 16 + G * 4) * NN + c * 64 + L16 * 4) * 4);
#pragma unroll
    for (int a = 0; a < NA; ++a) {
        if (a < nf) {
#pragma unroll
            for (int r = 0; r < 4; ++r) {
                const f32x4 v = {acc[a][0][r] * s, acc[a][1][r] * s, acc[a][2][r] * s, acc[a][3][r] * s};
                __builtin_amdgcn_raw_buffer_store_b128(__builtin_bit_cast(u32x4, v), ro, obase + (a * 16 + r) * NN * 4, 0, OUT_AUX);
            }
        }
    }
    if (lane == 0) *cnt = call;
    STAMP(blockIdx.x * 64 + wave * 16 + 10);
}

__global__ void __launch_bounds__(256, 1) gemm_rs_kernel(Params p) {
    __shared__ __attribute__((aligned(16))) char lds[LDS_BYTES];
    if (blockIdx.x < NS) {
        const int wave = threadIdx.x >> 6;
        if (wave < SW) {
            if (wave * NS + (int)blockIdx.x < NW) sender(p, lds, wave);
        } else if (wave == 3) {
            const int lane = threadIdx.x & 63;
            u32* const cnt = (u32*)(p.ws[p.rank] + CNT_OFF + (size_t)(blockIdx.x * 4 + wave) * 32);
            const u32 call = __builtin_amdgcn_readfirstlane(*cnt + 1);
            send_x(p, call, DATA_OFF + (size_t)(call & 1) * PAR_BYTES, blockIdx.x, lane);
            if (lane == 0) *cnt = call;
        }
    } else {
        consumer(p, lds);
    }
}

}

extern "C" void kernel_launch(int rank, int nranks, ncclComm_t comm, hipStream_t stream,
                              void* const* d_in, const int* in_sizes, int n_in,
                              void* d_out, int out_size, const int* in_dtypes, int out_dtype,
                              void* const* d_out_all, void* const* d_ws_all, size_t ws_bytes) {
    (void)comm; (void)in_dtypes; (void)out_dtype; (void)d_out_all;
    if (nranks != NR || n_in != 4 || rank < 0 || rank >= NR || ws_bytes < WS_NEED ||
        in_sizes[0] != 4096 * KS || in_sizes[1] != KS * NN || out_size != MS * NN) {
        fprintf(stderr, "gemm_rs: unexpected problem shape\n");
        return;
    }
    Params p;
    p.x = (const unsigned char*)d_in[0];
    p.w = (const unsigned char*)d_in[1];
    p.sx = (const float*)d_in[2];
    p.sw = (const float*)d_in[3];
    p.out = (float*)d_out;
    for (int r = 0; r < NR; ++r) p.ws[r] = (char*)d_ws_all[r];
    p.rank = rank;
    p.pad = 0;
    hipLaunchKernelGGL(gemm_rs_kernel, dim3(GRID), dim3(256), 0, stream, p);
    HIP_CHECK(hipGetLastError());
}
```

```cpp
#include <hip/hip_runtime.h>
#include <rccl/rccl.h>
#include <cstdio>
#include <cstdint>

#define HIP_CHECK(expr) do { hipError_t _e = (expr); if (_e != hipSuccess) { \
    fprintf(stderr, "HIP error %s at %s:%d\n", hipGetErrorString(_e), __FILE__, __LINE__); \
    return; } } while (0)

namespace {

typedef unsigned u32;
typedef unsigned long long u64;
typedef __attribute__((address_space(1))) unsigned gu32;
typedef unsigned u32x4 __attribute__((ext_vector_type(4)));
typedef int i32x8 __attribute__((ext_vector_type(8)));
typedef float f32x4 __attribute__((ext_vector_type(4)));

constexpr int NR = 8;
constexpr int MS = 512;
constexpr int KS = 512;
constexpr int NN = 4096;
constexpr int NJ = 8;
constexpr int NC = 64;
constexpr int NU = NJ * NC;

#define NSEND 64
#define SWAVES 1
constexpr int NS = NSEND;
constexpr int SW = SWAVES;
#define NWAVES 32
constexpr int NW = NWAVES;
constexpr int NCONS = 192;
constexpr int GRID = NS + NCONS;
constexpr int XW = 256 / NS;
static_assert(XW * NS == 256 && SW + XW <= 8, "x half-units");
static_assert(GRID == 256 && SW <= 7, "one workgroup per CU, 256 x half-units");
static_assert(NW <= NU, "every sender wave takes at least one w unit");

constexpr size_t CNT_OFF   = 0;
constexpr size_t TMO_OFF   = 64 << 10;
constexpr size_t TICK_OFF  = 96 << 10;
constexpr size_t XFLAG_OFF = 128 << 10;
constexpr size_t WFLAG_OFF = 256 << 10;
constexpr size_t DATA_OFF  = 1 << 20;
constexpr size_t PAR_BYTES = 32u << 20;
constexpr size_t X_BYTES   = 2u << 20;
constexpr u32    SLOT      = 4224;
constexpr size_t W_BYTES   = (size_t)NU * NR * SLOT;
constexpr size_t WS_NEED   = DATA_OFF + PAR_BYTES + X_BYTES + W_BYTES;
static_assert(X_BYTES + W_BYTES <= PAR_BYTES, "parity region too small");
static_assert((size_t)GRID * 8 * 16 <= TMO_OFF, "counter region too small");
static_assert(WFLAG_OFF + (size_t)NU * NR * 128 <= DATA_OFF, "flag region too small");

constexpr u64 DEADLINE = 40000000ull;
constexpr int AUX_SYS = 17;
#define DEFER_FLAG 1
#define EARLY_ST 2
#define EARLY_ST2 7
#define OUT_AUX 16
constexpr u32 MAX_EXC = 511;
constexpr u32 REPICK = 127;

struct Params {
    const unsigned char* x;
    const unsigned char* w;
    const float* sx;
    const float* sw;
    float* out;
    char* ws[NR];
    int rank;
    int pad;
};

__device__ __forceinline__ u32 perm(u32 hi, u32 lo, u32 sel) { return __builtin_amdgcn_perm(hi, lo, sel); }

template <int IMM>
__device__ __forceinline__ void st16_sys(char* base, u32 off, u32x4 v) {
    asm volatile("global_store_dwordx4 %0, %1, %2 offset:%3 sc0 sc1\n\ts_nop 1"
                 :: "v"(off), "v"(v), "s"(base), "i"(IMM) : "memory");
}
__device__ __forceinline__ char* uni(char* p) {
    const u64 v = (u64)p;
    const u32 lo = __builtin_amdgcn_readfirstlane((u32)v);
    const u32 hi = __builtin_amdgcn_readfirstlane((u32)(v >> 32));
    char* r = (char*)(((u64)hi << 32) | lo);
    asm volatile("s_nop 4" : "+s"(r));
    return r;
}
__device__ __forceinline__ void wait_vm0() { asm volatile("s_waitcnt vmcnt(0)" ::: "memory"); }

#define STAMP(slot) do {} while (0)
#define STAMP_HW(slot) do {} while (0)

__device__ __forceinline__ bool wait_flags(const gu32* flag, bool active, u32 call, gu32* tmo) {
    const u64 t0 = __builtin_amdgcn_s_memrealtime();
    bool bad = false;
    for (u32 it = 0;; ++it) {
        u32 v = call;
        if (active) v = __hip_atomic_load(flag, __ATOMIC_RELAXED, __HIP_MEMORY_SCOPE_SYSTEM);
        if (__all(v >= call)) break;
        if ((it & 15) == 15) {
            if (__hip_atomic_load(tmo, __ATOMIC_RELAXED, __HIP_MEMORY_SCOPE_AGENT) == call ||
                __builtin_amdgcn_s_memrealtime() - t0 > DEADLINE) {
                __hip_atomic_store(tmo, call, __ATOMIC_RELAXED, __HIP_MEMORY_SCOPE_AGENT);
                bad = true;
                break;
            }
        }
        __builtin_amdgcn_s_sleep(2);
    }
    asm volatile("" ::: "memory");
    return bad;
}

template <int CTRL, int RMASK>
__device__ __forceinline__ u32 dpp0(u32 v) {
    return __builtin_amdgcn_update_dpp(0u, v, CTRL, RMASK, 0xf, false);
}
__device__ __forceinline__ u32 wave_incl_scan(u32 v) {
    v += dpp0<0x111, 0xf>(v);
    v += dpp0<0x112, 0xf>(v);
    v += dpp0<0x114, 0xf>(v);
    v += dpp0<0x118, 0xf>(v);
    v += dpp0<0x142, 0xa>(v);
    v += dpp0<0x143, 0xc>(v);
    return v;
}
__device__ __forceinline__ u32 wave_max(u32 v) {
    v = max(v, dpp0<0x111, 0xf>(v));
    v = max(v, dpp0<0x112, 0xf>(v));
    v = max(v, dpp0<0x114, 0xf>(v));
    v = max(v, dpp0<0x118, 0xf>(v));
    v = max(v, dpp0<0x142, 0xa>(v));
    v = max(v, dpp0<0x143, 0xc>(v));
    return __builtin_amdgcn_readlane(v, 63);
}

constexpr int A7_OFF = 65536 + 256;
constexpr int LDS_BYTES = A7_OFF + 11 * 8192;
constexpr int SND_LDS = 10240;

struct Enc {
    u32x4 c0, c1, c2, c3;
    u32x4 hv;
    u32 total;
};

typedef __attribute__((address_space(3))) char lchar;
__device__ __forceinline__ u32 lds_addr(char* p) { return (u32)(u64)(lchar*)p; }

__device__ __forceinline__ void dma_raw(const unsigned char* src, u32 ldsdst) {
    u32 keep;
    const unsigned char* s1 = src + 16 * NN;
    const unsigned char* s2 = src + 32 * NN;
    const unsigned char* s3 = src + 48 * NN;
    asm volatile("s_mov_b32 %0, m0\n\t"
                 "s_mov_b32 m0, %5\n\ts_nop 0\n\tglobal_load_lds_dwordx4 %1, off\n\t"
                 "s_add_u32 m0, m0, 0x400\n\ts_nop 0\n\tglobal_load_lds_dwordx4 %2, off\n\t"
                 "s_add_u32 m0, m0, 0x400\n\ts_nop 0\n\tglobal_load_lds_dwordx4 %3, off\n\t"
                 "s_add_u32 m0, m0, 0x400\n\ts_nop 0\n\tglobal_load_lds_dwordx4 %4, off\n\t"
                 "s_mov_b32 m0, %0"
                 : "=&s"(keep) : "v"(src), "v"(s1), "v"(s2), "v"(s3), "s"(ldsdst) : "memory", "scc");
}
__device__ __forceinline__ void draw_ticket(u32& t, u32* ctr) {
    u64 save;
    asm volatile("s_mov_b64 %1, exec\n\ts_mov_b64 exec, 1\n\tglobal_atomic_add %0, %2, %3, %4 sc0\n\ts_mov_b64 exec, %1"
                 : "=&v"(t), "=&s"(save) : "v"(0u), "v"(1u), "s"(ctr) : "memory");
}
template <int N>
__device__ __forceinline__ void wait_tk(u32& t) {
    asm volatile("s_waitcnt vmcnt(%1)" : "+v"(t) : "n"(N) : "memory");
}

__device__ __forceinline__ u32 pick_base(const u32 (&r)[16], u32* hist, int lane) {
    if (lane < 40) hist[lane] = 0;
    asm volatile("" ::: "memory");
    __hip_atomic_fetch_add(&hist[(r[0] >> 2) & 31], 1u, __ATOMIC_RELAXED, __HIP_MEMORY_SCOPE_WORKGROUP);
    __hip_atomic_fetch_add(&hist[(r[5] >> 10) & 31], 1u, __ATOMIC_RELAXED, __HIP_MEMORY_SCOPE_WORKGROUP);
    __hip_atomic_fetch_add(&hist[(r[10] >> 18) & 31], 1u, __ATOMIC_RELAXED, __HIP_MEMORY_SCOPE_WORKGROUP);
    __hip_atomic_fetch_add(&hist[(r[15] >> 26) & 31], 1u, __ATOMIC_RELAXED, __HIP_MEMORY_SCOPE_WORKGROUP);
    asm volatile("" ::: "memory");
    u32 wsum = 0;
    const int hl = lane & 31;
#pragma unroll
    for (int i = 0; i < 8; ++i) wsum += hist[hl + i];
    const u32 key = (lane <= 24) ? (wsum * 32 + (31 - lane)) : 0u;
    return 31 - (wave_max(key) & 31);
}

__device__ __forceinline__ void encode(Enc& e, const char* raw, unsigned short* list, u32* hist, int lane, u32& b) {
    const int q = lane & 15, g = lane >> 4;
    u32 r[16];
#pragma unroll
    for (int i = 0; i < 16; ++i) r[i] = *(const u32*)(raw + (g * 16 + i) * 64 + q * 4);
    if (lane < 32) ((u32*)list)[lane] = 0;
    u32 T[16];
#pragma unroll
    for (int t = 0; t < 4; ++t) {
        const u32 t0 = perm(r[4 * t + 1], r[4 * t + 0], 0x05010400u);
        const u32 t1 = perm(r[4 * t + 1], r[4 * t + 0], 0x07030602u);
        const u32 t2 = perm(r[4 * t + 3], r[4 * t + 2], 0x05010400u);
        const u32 t3 = perm(r[4 * t + 3], r[4 * t + 2], 0x07030602u);
        T[0 + t]  = perm(t2, t0, 0x05040100u);
        T[4 + t]  = perm(t2, t0, 0x07060302u);
        T[8 + t]  = perm(t3, t1, 0x05040100u);
        T[12 + t] = perm(t3, t1, 0x07060302u);
    }
    bool picked = false;
    if (b >= 32) { b = pick_base(r, hist, lane); picked = true; }
    u32 code[16], D[4], n, incl, total;
    while (true) {
        const u32 B4 = (b << 2) * 0x01010101u;
        const u32 NB4 = (((8 - b) & 7) << 2) * 0x01010101u;
        n = 0;
#pragma unroll
        for (int k = 0; k < 4; ++k) {
            u32 v[4];
#pragma unroll
            for (int t = 0; t < 4; ++t) {
                const int i = 4 * k + t;
                const u32 d4 = ((T[i] & 0x1C1C1C1Cu) + NB4) & 0x1C1C1C1Cu;
                code[i] = (T[i] & 0x83838383u) | d4;
                const u32 z = T[i] ^ (code[i] + B4);
                v[t] = __builtin_amdgcn_udot4(z, 0x40100401u, 0u, false);
            }
            D[k] = (v[0] >> 5) | (v[1] << 3) | (v[2] << 11) | (v[3] << 19);
            n += __builtin_popcount((D[k] | (D[k] >> 1)) & 0x55555555u);
        }
        incl = wave_incl_scan(n);
        total = __builtin_amdgcn_readlane(incl, 63);
        if (total <= REPICK || picked) break;
        picked = true;
        const u32 b2 = pick_base(r, hist, lane);
        if (b2 == b) break;
        b = b2;
    }
    e.total = total;
    if (total <= MAX_EXC) {
        u32 idx = 1 + incl - n;
        const u32 posb = ((g & 1) * 512 + (g >> 1) * 256 + q * 16) << 2;
#pragma unroll
        for (int k = 0; k < 4; ++k) {
            u32 dd = D[k];
            while (dd) {
                const u32 bit = __builtin_ctz(dd) & ~1u;
                list[idx++] = (unsigned short)(posb + k * 4096 + (bit << 1) + ((dd >> bit) & 3));
                dd &= ~(3u << bit);
            }
        }
        if (lane == 0) list[0] = (unsigned short)(b | (total << 6));
        u32 cw[12];
#pragma unroll
        for (int h = 0; h < 2; ++h) {
            const u32* cd = code + 8 * h;
            cw[6 * h + 0] = cd[0] | ((cd[6] & 0x03030303u) << 5);
            cw[6 * h + 1] = cd[1] | ((cd[6] & 0x0C0C0C0Cu) << 3);
            cw[6 * h + 2] = cd[2] | ((cd[6] & 0x10101010u) << 1) | ((cd[6] & 0x80808080u) >> 1);
            cw[6 * h + 3] = cd[3] | ((cd[7] & 0x03030303u) << 5);
            cw[6 * h + 4] = cd[4] | ((cd[7] & 0x0C0C0C0Cu) << 3);
            cw[6 * h + 5] = cd[5] | ((cd[7] & 0x10101010u) << 1) | ((cd[7] & 0x80808080u) >> 1);
        }
        asm volatile("" ::: "memory");
        e.hv = *(const u32x4*)((const char*)list + (lane & 7) * 16);
        e.c3 = *(const u32x4*)((const char*)list + 128 + (lane < 56 ? lane : 0) * 16);
        e.c0 = u32x4{cw[0], cw[1], cw[2], cw[3]};
        e.c1 = u32x4{cw[4], cw[5], cw[6], cw[7]};
        e.c2 = u32x4{cw[8], cw[9], cw[10], cw[11]};
        asm volatile("" ::: "memory");
    } else {
        e.hv = u32x4{(lane == 0) ? 32u : 0u, 0u, 0u, 0u};
        e.c0 = u32x4{T[0], T[1], T[2], T[3]};
        e.c1 = u32x4{T[4], T[5], T[6], T[7]};
        e.c2 = u32x4{T[8], T[9], T[10], T[11]};
        e.c3 = u32x4{T[12], T[13], T[14], T[15]};
    }
}

__device__ __forceinline__ void store_unit(const Enc& e, char* const (&dst)[NR], u32 slotoff, int lane) {
    const u32 off = slotoff + 128 + lane * 16;
#pragma unroll
    for (int d = 0; d < NR; ++d) st16_sys<0>(dst[d], off, e.c0);
#pragma unroll
    for (int d = 0; d < NR; ++d) st16_sys<1024>(dst[d], off, e.c1);
#pragma unroll
    for (int d = 0; d < NR; ++d) st16_sys<2048>(dst[d], off, e.c2);
    if (lane < 8) {
#pragma unroll
        for (int d = 0; d < NR; ++d) st16_sys<0>(dst[d], slotoff + lane * 16, e.hv);
    }
    if (e.total > 63) {
        if (e.total > MAX_EXC || (u32)lane * 8 < e.total - 63) {
#pragma unroll
            for (int d = 0; d < NR; ++d) st16_sys<3072>(dst[d], off, e.c3);
        }
    }
}

__device__ __forceinline__ void flag_store(char* a, u32 call) {
    __hip_atomic_store((gu32*)(u64)a, call, __ATOMIC_RELAXED, __HIP_MEMORY_SCOPE_SYSTEM);
}

__device__ __forceinline__ void sender(const Params& p, char* lds, int wave) {
    const int lane = threadIdx.x & 63;
    const int wid = __builtin_amdgcn_readfirstlane(wave * NS + blockIdx.x);
    const int rank = p.rank;
    char* const myws = p.ws[rank];
    u32* const cnt = (u32*)(myws + CNT_OFF + (size_t)(blockIdx.x * 8 + wave) * 16);
    STAMP(blockIdx.x * 128 + 0);
    STAMP_HW(blockIdx.x * 128 + 15);
    char* const mylds = lds + wave * SND_LDS;
    char* const raw0 = mylds;
    unsigned short* const list = (unsigned short*)(mylds + 8192);
    u32* const hist = (u32*)(mylds + 8192 + 1280);
    const u32 rawaddr = __builtin_amdgcn_readfirstlane(lds_addr(raw0));
    const unsigned char* const wsrc = p.w + (size_t)(lane >> 2) * NN + (lane & 3) * 16;
    auto unit_src = [&](int u) { return wsrc + (size_t)(u / NC) * 64 * NN + (u % NC) * 64; };

    u32* const tick = (u32*)uni(myws + TICK_OFF);
    u32 tk;
    dma_raw(unit_src(wid), rawaddr);
    draw_ticket(tk, tick);

    const u32 call = __builtin_amdgcn_readfirstlane(*cnt + 1);
    const size_t par = DATA_OFF + (size_t)(call & 1) * PAR_BYTES;
    char* flagbase = p.ws[lane & (NR - 1)] + WFLAG_OFF + (size_t)rank * 128;
    char* dst[NR];
    const int rot = wid & (NR - 1);
#pragma unroll
    for (int d = 0; d < NR; ++d) dst[d] = uni(p.ws[(rank + 1 + d + rot) & (NR - 1)] + par);
    asm volatile("" : "+v"(flagbase));
    wait_tk<0>(tk);

    const u32 tbase = (call - 1) * (NU + NW) - NW;
    int u = wid;
    int un = (int)(__builtin_amdgcn_readfirstlane(tk) - tbase);
    int kb = 0;
    if ((u32)un < (u32)NU) dma_raw(unit_src(un), rawaddr + 4096);
    Enc e;
    u32 base = 32;
    encode(e, raw0, list, hist, lane, base);
    STAMP(blockIdx.x * 128 + 1);

    int uprev = -1;
    while (true) {
        draw_ticket(tk, tick);
        store_unit(e, dst, (u32)(X_BYTES + (size_t)(u * NR + rank) * SLOT), lane);
        wait_tk<32>(tk);
        if (uprev >= 0) {
            if (lane < NR) flag_store(flagbase + (size_t)uprev * NR * 128, call);
        }
        if ((u32)un >= (u32)NU) break;
        const int unn = (int)(__builtin_amdgcn_readfirstlane(tk) - tbase);
        if ((u32)unn < (u32)NU)
            dma_raw(unit_src(unn), __builtin_amdgcn_readfirstlane(rawaddr + kb * 4096));
        kb ^= 1;
        encode(e, raw0 + kb * 4096, list, hist, lane, base);
        uprev = u;
        u = un;
        un = unn;
    }
    wait_vm0();
    if (lane < NR) flag_store(flagbase + (size_t)u * NR * 128, call);
    if (lane == 0) *cnt = call;
}

__device__ __forceinline__ void decode_unit(char* img, __amdgpu_buffer_rsrc_t rs, u32 slot, int lane,
                                            const u32x4 (&cw)[3], u32 ent) {
    const u32 hdr = __builtin_amdgcn_readfirstlane(ent) & 0xFFFFu;
    const int q = lane & 15, g = lane >> 4;
    char* const wp = img + (g & 1) * 512 + (g >> 1) * 256 + q * 16;
    if (!(hdr & 32)) {
        const u32 B4 = ((hdr & 31) << 2) * 0x01010101u;
        const u32 c[12] = {cw[0].x, cw[0].y, cw[0].z, cw[0].w, cw[1].x, cw[1].y, cw[1].z, cw[1].w,
                           cw[2].x, cw[2].y, cw[2].z, cw[2].w};
#pragma unroll
        for (int h = 0; h < 2; ++h) {
            const u32* cc = c + 6 * h;
            u32x4 a, bq;
            a.x = (cc[0] & 0x9F9F9F9Fu) + B4;
            a.y = (cc[1] & 0x9F9F9F9Fu) + B4;
            a.z = (cc[2] & 0x9F9F9F9Fu) + B4;
            a.w = (cc[3] & 0x9F9F9F9Fu) + B4;
            bq.x = (cc[4] & 0x9F9F9F9Fu) + B4;
            bq.y = (cc[5] & 0x9F9F9F9Fu) + B4;
            bq.z = (((cc[0] >> 5) & 0x03030303u) | ((cc[1] >> 3) & 0x0C0C0C0Cu) |
                    ((cc[2] >> 1) & 0x10101010u) | ((cc[2] << 1) & 0x80808080u)) + B4;
            bq.w = (((cc[3] >> 5) & 0x03030303u) | ((cc[4] >> 3) & 0x0C0C0C0Cu) |
                    ((cc[5] >> 1) & 0x10101010u) | ((cc[5] << 1) & 0x80808080u)) + B4;
            *(u32x4*)(wp + (2 * h) * 1024) = a;
            *(u32x4*)(wp + (2 * h + 1) * 1024) = bq;
        }
        const u32 total = hdr >> 6;
        {
            const u32 e = (lane == 0) ? 0u : (ent & 0xFFFFu);
            const u32 pos = e >> 2;
            const u32 x = (e & 3) << (5 + 8 * (pos & 3));
            __hip_atomic_fetch_xor((u32*)(img + (pos & ~3u)), x, __ATOMIC_RELAXED, __HIP_MEMORY_SCOPE_WORKGROUP);
        }
#pragma clang loop unroll(disable)
        for (u32 i0 = 63; i0 < total; i0 += 64) {
            const u32 i = i0 + lane;
            u32 e = __builtin_amdgcn_raw_buffer_load_b16(rs, slot + 128 + 3072 + (i - 63) * 2, 0, AUX_SYS);
            e = (i < total) ? (e & 0xFFFFu) : 0u;
            const u32 pos = e >> 2;
            const u32 x = (e & 3) << (5 + 8 * (pos & 3));
            __hip_atomic_fetch_xor((u32*)(img + (pos & ~3u)), x, __ATOMIC_RELAXED, __HIP_MEMORY_SCOPE_WORKGROUP);
        }
    } else {
        const u32x4 v3 = __builtin_amdgcn_raw_buffer_load_b128(rs, slot + 128 + 3072 + lane * 16, 0, AUX_SYS);
        *(u32x4*)(wp + 0) = cw[0];
        *(u32x4*)(wp + 1024) = cw[1];
        *(u32x4*)(wp + 2048) = cw[2];
        *(u32x4*)(wp + 3072) = v3;
    }
}

__device__ __forceinline__ void send_x(const Params& p, u32 call, size_t par, int b, int lane) {
    const int dest = __builtin_amdgcn_readfirstlane((p.rank + 1 + b) & (NR - 1));
    const int rf = __builtin_amdgcn_readfirstlane((b >> 3) & 31);
    const unsigned char* src = p.x + (size_t)(dest * MS + rf * 16 + (lane & 15)) * KS
                             + ((lane >> 4) & 1) * 32 + (lane >> 5) * 16;
    u32x4 v[8];
#pragma unroll
    for (int j = 0; j < 8; ++j) v[j] = *(const u32x4*)(src + j * 64);
    char* base = uni(p.ws[dest] + par + (size_t)(p.rank * 32 + rf) * 8192);
    char* xf = uni(p.ws[dest] + XFLAG_OFF + (size_t)(p.rank * 32 + rf) * 128);
#pragma unroll
    for (int j = 0; j < 8; ++j) asm volatile("" :: "v"(v[j]));
#pragma unroll
    for (int j = 0; j < 8; j += 4) {
        st16_sys<0>(base + j * 1024, lane * 16, v[j]);
        st16_sys<1024>(base + j * 1024, lane * 16, v[j + 1]);
        st16_sys<2048>(base + j * 1024, lane * 16, v[j + 2]);
        st16_sys<3072>(base + j * 1024, lane * 16, v[j + 3]);
    }
    wait_vm0();
    if (lane == 0) __hip_atomic_store((gu32*)(u64)xf, call, __ATOMIC_RELAXED, __HIP_MEMORY_SCOPE_SYSTEM);
}

template <int NF>
__device__ __forceinline__ void consumer_wave(const Params& p, char* lds, int t, int wave, int rf0, u32 call, size_t par) {
    const int lane = threadIdx.x & 63;
    const int c = (t & 7) * 8 + ((t >> 3) & 7);
    char* const myws = uni(p.ws[p.rank]);
    gu32* const tmo = (gu32*)(u64)(myws + TMO_OFF);
    const float scale = p.sx[0] * p.sw[0];
    const __amdgpu_buffer_rsrc_t rs =
        __builtin_amdgcn_make_buffer_rsrc(uni(myws + par), 0, (int)(X_BYTES + W_BYTES), 0x00020000);
    u32* const badw = (u32*)(lds + 65536);
    u32* const rdy = (u32*)(lds + 65536 + 32);
    const __amdgpu_buffer_rsrc_t ro = __builtin_amdgcn_make_buffer_rsrc(uni((char*)p.out), 0, MS * NN * 4, 0x00020000);

    f32x4 acc[NF][4];

    const u32 xoff = (u32)(((lane >> 5) * 32 + rf0) * 8192 + (lane & 31) * 16);
    const int L16 = lane & 15, G = lane >> 4;
    const u32 bread = (G >> 1) * 4096 + (G & 1) * 256 + L16 * 16;

    u32x4 A[4 * NF * 2];
    auto load_a = [&](int P, int j) {
#pragma unroll
        for (int a = 0; a < NF; ++a)
#pragma unroll
            for (int i = 0; i < 2; ++i)
                A[(P * NF + a) * 2 + i] = __builtin_amdgcn_raw_buffer_load_b128(
                    rs, xoff + j * 1024 + i * 512, P * 2 * 32 * 8192 + a * 8192, AUX_SYS);
    };
    char* const a7 = lds + A7_OFF + (rf0 - (t >> 6) * 11) * 8192;
    auto dma_a7 = [&]() {
        const char* src = myws + par + xoff + (NJ - 1) * 1024;
        asm volatile("" : "+v"(src));
        const u32 dst = __builtin_amdgcn_readfirstlane(lds_addr(a7));
#pragma unroll
        for (int a = 0; a < NF; ++a)
#pragma unroll
            for (int P = 0; P < 4; ++P) {
                u32 keep;
                const char* s0 = src + a * 8192 + (size_t)P * 2 * 32 * 8192;
                asm volatile("s_mov_b32 %0, m0\n\ts_mov_b32 m0, %2\n\ts_nop 0\n\tglobal_load_lds_dwordx4 %1, off sc0 sc1\n\t"
                             "s_add_u32 m0, m0, 0x400\n\ts_nop 0\n\tglobal_load_lds_dwordx4 %3, off sc0 sc1\n\ts_mov_b32 m0, %0"
                             : "=&s"(keep) : "v"(s0), "s"(dst + a * 8192 + P * 2048), "v"(s0 + 512) : "memory", "scc");
            }
    };
    auto mfma = [&](int P, int f, const u32x4& b0, const u32x4& b1) {
        const i32x8 B = {(int)b0.x, (int)b0.y, (int)b0.z, (int)b0.w, (int)b1.x, (int)b1.y, (int)b1.z, (int)b1.w};
#pragma unroll
        for (int a = 0; a < NF; ++a) {
            const u32x4 a0 = A[(P * NF + a) * 2], a1 = A[(P * NF + a) * 2 + 1];
            const i32x8 Av = {(int)a0.x, (int)a0.y, (int)a0.z, (int)a0.w, (int)a1.x, (int)a1.y, (int)a1.z, (int)a1.w};
            acc[a][f] = __builtin_amdgcn_mfma_scale_f32_16x16x128_f8f6f4(Av, B, acc[a][f], 0, 1, 0, 0, 0, 0);
        }
    };
    u32x4 cw[3];
    u32 ent;
    auto unit_slot = [&](int j) { return (u32)(X_BYTES + (size_t)((j * NC + c) * NR + wave) * SLOT); };
    auto issue = [&](int j, u32 oob) {
        const u32 slot = unit_slot(j) | oob;
        ent = __builtin_amdgcn_raw_buffer_load_b16(rs, slot + lane * 2, 0, AUX_SYS);
#pragma unroll
        for (int i = 0; i < 3; ++i)
            cw[i] = __builtin_amdgcn_raw_buffer_load_b128(rs, slot + 128 + i * 1024 + lane * 16, 0, AUX_SYS);
    };
    auto decode = [&](int j, u32 oob) {
        decode_unit(lds + (j & 1) * 32768 + wave * 4096, rs, unit_slot(j) | oob, lane, cw, ent);
    };
    auto wflag = [&](int j) { return (const gu32*)(u64)(myws + WFLAG_OFF + (size_t)((j * NC + c) * NR + wave) * 128); };
    bool bad = false;

    bool have = false;
    auto round = [&](int j) {
        if (!have) {
            bad |= wait_flags(wflag(j), lane == 0, call, tmo);
            issue(j, 0);
        }
        decode(j, 0);
        __syncthreads();
        u32 pv = __hip_atomic_load(wflag(j + 1), __ATOMIC_RELAXED, __HIP_MEMORY_SCOPE_SYSTEM);
        const char* const buf = lds + (j & 1) * 32768;
        u32x4 Bf[2][2][2];
#pragma unroll
        for (int f = 0; f < 2; ++f) {
            Bf[0][f][0] = *(const u32x4*)(buf + f * 1024 + bread);
            Bf[0][f][1] = *(const u32x4*)(buf + f * 1024 + bread + 512);
        }
#pragma unroll
        for (int st = 0; st < 8; ++st) {
            if (st < 7) {
                const int Pn = (st + 1) >> 1, fn = ((st + 1) & 1) * 2;
#pragma unroll
                for (int f = 0; f < 2; ++f) {
                    Bf[(st + 1) & 1][f][0] = *(const u32x4*)(buf + Pn * 8192 + (fn + f) * 1024 + bread);
                    Bf[(st + 1) & 1][f][1] = *(const u32x4*)(buf + Pn * 8192 + (fn + f) * 1024 + bread + 512);
                }
            }
            __builtin_amdgcn_sched_barrier(0);
#pragma unroll
            for (int f = 0; f < 2; ++f) mfma(st >> 1, (st & 1) * 2 + f, Bf[st & 1][f][0], Bf[st & 1][f][1]);
            if ((st & 1) && j + 2 < NJ) load_a(st >> 1, j + 1);
            if (st == EARLY_ST) {
                have = __builtin_amdgcn_readfirstlane(pv) >= call;
                asm volatile("" ::: "memory");
                if (have) issue(j + 1, 0);
                else pv = __hip_atomic_load(wflag(j + 1), __ATOMIC_RELAXED, __HIP_MEMORY_SCOPE_SYSTEM);
            }
            if (st == EARLY_ST2 && !have) {
                have = __builtin_amdgcn_readfirstlane(pv) >= call;
                asm volatile("" ::: "memory");
                if (have) issue(j + 1, 0);
            }
            __builtin_amdgcn_sched_barrier(0);
        }
        STAMP(blockIdx.x * 128 + wave * 16 + 2 + j);
    };

#pragma clang loop unroll(disable)
    for (u32 pass = 0; pass < 2; ++pass) {
        asm volatile("" : "+s"(pass));
        const u32 oob = pass ? 0u : 0x80000000u;
#pragma unroll
        for (int a = 0; a < NF; ++a)
#pragma unroll
            for (int f = 0; f < 4; ++f) acc[a][f] = f32x4{0.f, 0.f, 0.f, 0.f};
        if (pass) {
            bad = wait_flags((const gu32*)(u64)(myws + XFLAG_OFF + (size_t)((lane & 7) * 32 + rf0 + ((lane >> 3) & (NF - 1))) * 128),
                             lane < NF * NR, call, tmo);
            STAMP(blockIdx.x * 128 + wave * 16 + 1);
            dma_a7();
#pragma unroll
            for (int P = 0; P < 4; ++P) load_a(P, 0);
#pragma clang loop unroll(disable)
            for (int j = 0; j < NJ - 1; ++j) round(j);
            asm volatile("s_waitcnt vmcnt(0)" ::: "memory");
        }

        {
            constexpr int j = NJ - 1;
            const char* const bbuf = lds + (j & 1) * 32768 + bread;
            const char* const abuf = a7 + lane * 16;
            const gu32* const fl = wflag(j);
            const u64 t0 = __builtin_amdgcn_s_memrealtime();
            asm volatile("" :: "s"(t0));
            bool own = false;
            u32 todo = 15, avail = 0, it = 0;
            auto pick = [&]() {
                if (!pass) { own = false; avail = 0; todo = 15; }
                while (!avail) {
                    u32 v = call;
                    if (!own && pass && !have) v = __hip_atomic_load(fl, __ATOMIC_RELAXED, __HIP_MEMORY_SCOPE_SYSTEM);
                    const u32x4 r0 = *(const u32x4*)rdy, r1 = *(const u32x4*)(rdy + 4);
                    const u32 m = (min(r0.x, r0.y) > pass ? 1u : 0u) | (min(r0.z, r0.w) > pass ? 2u : 0u) |
                                  (min(r1.x, r1.y) > pass ? 4u : 0u) | (min(r1.z, r1.w) > pass ? 8u : 0u);
                    avail = __builtin_amdgcn_readfirstlane(m & todo);
                    todo &= ~avail;
                    if (!own) {
                        bool go = __builtin_amdgcn_readfirstlane(v) >= call;
                        if (!go && (++it & 15) == 15 &&
                            (__hip_atomic_load(tmo, __ATOMIC_RELAXED, __HIP_MEMORY_SCOPE_AGENT) == call ||
                             __builtin_amdgcn_s_memrealtime() - t0 > DEADLINE)) {
                            __hip_atomic_store(tmo, call, __ATOMIC_RELAXED, __HIP_MEMORY_SCOPE_AGENT);
                            bad = true;
                            go = true;
                        }
                        if (go) {
                            asm volatile("" ::: "memory");
                            STAMP(blockIdx.x * 128 + wave * 16 + 11);
                            if (!have) issue(j, oob);
                            decode(j, oob);
                            if (lane == 0) {
                                badw[wave] = bad;
                                __hip_atomic_store(&rdy[wave], pass + 1, __ATOMIC_RELEASE, __HIP_MEMORY_SCOPE_WORKGROUP);
                            }
                            STAMP(blockIdx.x * 128 + wave * 16 + 13);
                            own = true;
                        }
                    } else if (!avail) {
                        __builtin_amdgcn_s_sleep(1);
                    }
                }
                asm volatile("" ::: "memory");
                const int P = __builtin_ctz(avail);
                avail &= avail - 1;
                return P;
            };
            struct Frags { u32x4 a[NF][2], b[4][2]; };
            auto rd_a = [&](Frags& F, int P, int a) {
                F.a[a][0] = *(const u32x4*)(abuf + a * 8192 + P * 2048);
                F.a[a][1] = *(const u32x4*)(abuf + a * 8192 + P * 2048 + 1024);
            };
            auto rd_b = [&](Frags& F, int P, int f) {
                F.b[f][0] = *(const u32x4*)(bbuf + P * 8192 + f * 1024);
                F.b[f][1] = *(const u32x4*)(bbuf + P * 8192 + f * 1024 + 512);
            };
            auto mm_f = [&](const Frags& F, int f) {
                const i32x8 B = {(int)F.b[f][0].x, (int)F.b[f][0].y, (int)F.b[f][0].z, (int)F.b[f][0].w,
                                 (int)F.b[f][1].x, (int)F.b[f][1].y, (int)F.b[f][1].z, (int)F.b[f][1].w};
#pragma unroll
                for (int a = 0; a < NF; ++a) {
                    const i32x8 Av = {(int)F.a[a][0].x, (int)F.a[a][0].y, (int)F.a[a][0].z, (int)F.a[a][0].w,
                                      (int)F.a[a][1].x, (int)F.a[a][1].y, (int)F.a[a][1].z, (int)F.a[a][1].w};
                    acc[a][f] = __builtin_amdgcn_mfma_scale_f32_16x16x128_f8f6f4(Av, B, acc[a][f], 0, 1, 0, 0, 0, 0);
                }
            };
            auto mmrd = [&](const Frags& Fc, Frags& Fn, int P) {
#pragma unroll
                for (int f = 0; f < 4; ++f) {
                    mm_f(Fc, f);
                    if (f < NF) rd_a(Fn, P, f);
                    rd_b(Fn, P, f);
                    __builtin_amdgcn_sched_barrier(0);
                }
            };
            Frags F0, F1;
            {
                const int P = pick();
#pragma unroll
                for (int a = 0; a < NF; ++a) rd_a(F0, P, a);
#pragma unroll
                for (int f = 0; f < 4; ++f) rd_b(F0, P, f);
            }
            mmrd(F0, F1, pick());
            mmrd(F1, F0, pick());
            mmrd(F0, F1, pick());
#pragma unroll
            for (int f = 0; f < 4; ++f) mm_f(F1, f);
        }
        STAMP(blockIdx.x * 128 + wave * 16 + 9);

        const u32x4 b0 = *(const u32x4*)badw, b1 = *(const u32x4*)(badw + 4);
        const float s = (b0.x | b0.y | b0.z | b0.w | b1.x | b1.y | b1.z | b1.w) ? __builtin_nanf("") : scale;
        const u32 obase = (u32)(((rf0 * 16 + G * 4) * NN + c * 64 + L16 * 4) * 4) | oob;
#pragma unroll
        for (int a = 0; a < NF; ++a)
#pragma unroll
            for (int r = 0; r < 4; ++r) {
                const f32x4 v = {acc[a][0][r] * s, acc[a][1][r] * s, acc[a][2][r] * s, acc[a][3][r] * s};
                __builtin_amdgcn_raw_buffer_store_b128(__builtin_bit_cast(u32x4, v), ro, obase + (a * 16 + r) * NN * 4, 0, OUT_AUX);
            }
        STAMP(blockIdx.x * 128 + wave * 16 + 10);
    }
}

__device__ __forceinline__ void consumer(const Params& p, char* lds) {
    const int lane = threadIdx.x & 63;
    const int wave = __builtin_amdgcn_readfirstlane(threadIdx.x >> 6);
    const int t = blockIdx.x - NS;
    const int m = t >> 6;
    const int n2 = (m == 2) ? 2 : 3;
    const int rf0 = m * 11 + (wave < n2 ? 2 * wave : n2 + wave);
    char* const myws = p.ws[p.rank];
    u32* const cnt = (u32*)(myws + CNT_OFF + (size_t)(blockIdx.x * 8 + wave) * 16);
    const u32 call = __builtin_amdgcn_readfirstlane(*cnt + 1);
    const size_t par = DATA_OFF + (size_t)(call & 1) * PAR_BYTES;
    STAMP(blockIdx.x * 128 + wave * 16 + 0);
    STAMP_HW(blockIdx.x * 128 + wave * 16 + 15);
    if (lane == 0) ((u32*)(lds + 65536 + 32))[wave] = 0;
    if (wave < n2) consumer_wave<2>(p, lds, t, wave, rf0, call, par);
    else consumer_wave<1>(p, lds, t, wave, rf0, call, par);
    if (lane == 0) *cnt = call;
}

__global__ void __launch_bounds__(512, 2) gemm_rs_kernel(Params p) {
    __shared__ __attribute__((aligned(16))) char lds[LDS_BYTES];
    if (blockIdx.x < NS) {
        const int wave = threadIdx.x >> 6;
        if (wave < SW) {
            if (wave * NS + (int)blockIdx.x < NW) sender(p, lds, wave);
        } else if (wave >= 8 - XW) {
            const int lane = threadIdx.x & 63;
            u32* const cnt = (u32*)(p.ws[p.rank] + CNT_OFF + (size_t)(blockIdx.x * 8 + wave) * 16);
            const u32 call = __builtin_amdgcn_readfirstlane(*cnt + 1);
            send_x(p, call, DATA_OFF + (size_t)(call & 1) * PAR_BYTES, (wave - (8 - XW)) * NS + blockIdx.x, lane);
            if (lane == 0) *cnt = call;
        }
    } else {
        consumer(p, lds);
    }
}

}

extern "C" void kernel_launch(int rank, int nranks, ncclComm_t comm, hipStream_t stream,
                              void* const* d_in, const int* in_sizes, int n_in,
                              void* d_out, int out_size, const int* in_dtypes, int out_dtype,
                              void* const* d_out_all, void* const* d_ws_all, size_t ws_bytes) {
    (void)comm; (void)in_dtypes; (void)out_dtype; (void)d_out_all;
    if (nranks != NR || n_in != 4 || rank < 0 || rank >= NR || ws_bytes < WS_NEED ||
        in_sizes[0] != 4096 * KS || in_sizes[1] != KS * NN || out_size != MS * NN) {
        fprintf(stderr, "gemm_rs: unexpected problem shape\n");
        return;
    }
    Params p;
    p.x = (const unsigned char*)d_in[0];
    p.w = (const unsigned char*)d_in[1];
    p.sx = (const float*)d_in[2];
    p.sw = (const float*)d_in[3];
    p.out = (float*)d_out;
    for (int r = 0; r < NR; ++r) p.ws[r] = (char*)d_ws_all[r];
    p.rank = rank;
    p.pad = 0;
    hipLaunchKernelGGL(gemm_rs_kernel, dim3(GRID), dim3(512), 0, stream, p);
    HIP_CHECK(hipGetLastError());
}
```

```cpp
#include <hip/hip_runtime.h>
#include <rccl/rccl.h>
#include <cstdio>
#include <cstdint>

#define HIP_CHECK(expr) do { hipError_t _e = (expr); if (_e != hipSuccess) { \
    fprintf(stderr, "HIP error %s at %s:%d\n", hipGetErrorString(_e), __FILE__, __LINE__); \
    return; } } while (0)

namespace {

typedef unsigned u32;
typedef unsigned long long u64;
typedef __attribute__((address_space(1))) unsigned gu32;
typedef unsigned u32x4 __attribute__((ext_vector_type(4)));
typedef int i32x8 __attribute__((ext_vector_type(8)));
typedef float f32x4 __attribute__((ext_vector_type(4)));

constexpr int NR = 8;
constexpr int MS = 512;
constexpr int KS = 512;
constexpr int NN = 4096;
constexpr int NJ = 8;
constexpr int NC = 64;
constexpr int NU = NJ * NC;

#define NSEND 32
#define SWAVES 1
constexpr int NS = NSEND;
constexpr int SW = SWAVES;
#define NWAVES 30
constexpr int NW = NWAVES;
constexpr int NCONS = 224;
constexpr int GRID = NS + NCONS;
constexpr int XN = 1;
static_assert(GRID == 256 && SW <= 7, "one workgroup per CU, 256 x half-units");
static_assert(NW <= NU, "every sender wave takes at least one w unit");

constexpr size_t CNT_OFF   = 0;
constexpr size_t TMO_OFF   = 64 << 10;
constexpr size_t TICK_OFF  = 96 << 10;
constexpr size_t DATA_OFF  = 1 << 20;
constexpr u32    SLOT      = 4224;
constexpr u32    WF_OFF    = 0;
constexpr u32    XF_OFF    = 64 << 10;
constexpr u32    X_OFF     = 128 << 10;
constexpr u32    W_OFF     = 384 << 10;
constexpr u32    BLK       = 2560 << 10;
constexpr size_t PAR_BYTES = (size_t)NR * BLK;
constexpr size_t WS_NEED   = DATA_OFF + 2 * PAR_BYTES;
static_assert(W_OFF + NU * SLOT <= BLK, "block too small");
static_assert((size_t)GRID * 8 * 16 <= TMO_OFF, "counter region too small");

constexpr u64 DEADLINE = 40000000ull;
constexpr int AUX_SYS = 17;
#define DEFER_FLAG 1
#define PRETOUCH 1
#define EARLY_ST 2
#define EARLY_ST2 7
#define OUT_AUX 16
constexpr u32 MAX_EXC = 511;
constexpr u32 REPICK = 127;

struct Params {
    const unsigned char* x;
    const unsigned char* w;
    const float* sx;
    const float* sw;
    float* out;
    char* ws[NR];
    int rank;
    int pad;
};

__device__ __forceinline__ u32 slot_off(int src, int u) { return (u32)src * BLK + W_OFF + (u32)u * SLOT; }
__device__ __forceinline__ u32 perm(u32 hi, u32 lo, u32 sel) { return __builtin_amdgcn_perm(hi, lo, sel); }

template <int IMM>
__device__ __forceinline__ void st16_sys(char* base, u32 off, u32x4 v) {
    asm volatile("global_store_dwordx4 %0, %1, %2 offset:%3 sc0 sc1\n\ts_nop 1"
                 :: "v"(off), "v"(v), "s"(base), "i"(IMM) : "memory");
}
__device__ __forceinline__ char* uni(char* p) {
    const u64 v = (u64)p;
    const u32 lo = __builtin_amdgcn_readfirstlane((u32)v);
    const u32 hi = __builtin_amdgcn_readfirstlane((u32)(v >> 32));
    char* r = (char*)(((u64)hi << 32) | lo);
    asm volatile("s_nop 4" : "+s"(r));
    return r;
}
__device__ __forceinline__ void wait_vm0() { asm volatile("s_waitcnt vmcnt(0)" ::: "memory"); }

#define STAMP(slot) do {} while (0)
#define STAMP_HW(slot) do {} while (0)

__device__ __forceinline__ bool wait_flags(const gu32* flag, bool active, u32 call, gu32* tmo) {
    const u64 t0 = __builtin_amdgcn_s_memrealtime();
    bool bad = false;
    for (u32 it = 0;; ++it) {
        u32 v = call;
        if (active) v = __hip_atomic_load(flag, __ATOMIC_RELAXED, __HIP_MEMORY_SCOPE_SYSTEM);
        if (__all(v >= call)) break;
        if ((it & 15) == 15) {
            if (__hip_atomic_load(tmo, __ATOMIC_RELAXED, __HIP_MEMORY_SCOPE_AGENT) == call ||
                __builtin_amdgcn_s_memrealtime() - t0 > DEADLINE) {
                __hip_atomic_store(tmo, call, __ATOMIC_RELAXED, __HIP_MEMORY_SCOPE_AGENT);
                bad = true;
                break;
            }
        }
        __builtin_amdgcn_s_sleep(2);
    }
    asm volatile("" ::: "memory");
    return bad;
}

template <int CTRL, int RMASK>
__device__ __forceinline__ u32 dpp0(u32 v) {
    return __builtin_amdgcn_update_dpp(0u, v, CTRL, RMASK, 0xf, false);
}
__device__ __forceinline__ u32 wave_incl_scan(u32 v) {
    v += dpp0<0x111, 0xf>(v);
    v += dpp0<0x112, 0xf>(v);
    v += dpp0<0x114, 0xf>(v);
    v += dpp0<0x118, 0xf>(v);
    v += dpp0<0x142, 0xa>(v);
    v += dpp0<0x143, 0xc>(v);
    return v;
}
__device__ __forceinline__ u32 wave_max(u32 v) {
    v = max(v, dpp0<0x111, 0xf>(v));
    v = max(v, dpp0<0x112, 0xf>(v));
    v = max(v, dpp0<0x114, 0xf>(v));
    v = max(v, dpp0<0x118, 0xf>(v));
    v = max(v, dpp0<0x142, 0xa>(v));
    v = max(v, dpp0<0x143, 0xc>(v));
    return __builtin_amdgcn_readlane(v, 63);
}

constexpr int A7_OFF = 65536 + 256;
constexpr int LDS_BYTES = A7_OFF + 11 * 8192;
constexpr int SND_LDS = 10240;

struct Enc {
    u32x4 c0, c1, c2, c3;
    u32x4 hv;
    u32 total;
};

typedef __attribute__((address_space(3))) char lchar;
__device__ __forceinline__ u32 lds_addr(char* p) { return (u32)(u64)(lchar*)p; }

__device__ __forceinline__ void dma_raw(const unsigned char* src, u32 ldsdst) {
    u32 keep;
    const unsigned char* s1 = src + 16 * NN;
    const unsigned char* s2 = src + 32 * NN;
    const unsigned char* s3 = src + 48 * NN;
    asm volatile("s_mov_b32 %0, m0\n\t"
                 "s_mov_b32 m0, %5\n\ts_nop 0\n\tglobal_load_lds_dwordx4 %1, off\n\t"
                 "s_add_u32 m0, m0, 0x400\n\ts_nop 0\n\tglobal_load_lds_dwordx4 %2, off\n\t"
                 "s_add_u32 m0, m0, 0x400\n\ts_nop 0\n\tglobal_load_lds_dwordx4 %3, off\n\t"
                 "s_add_u32 m0, m0, 0x400\n\ts_nop 0\n\tglobal_load_lds_dwordx4 %4, off\n\t"
                 "s_mov_b32 m0, %0"
                 : "=&s"(keep) : "v"(src), "v"(s1), "v"(s2), "v"(s3), "s"(ldsdst) : "memory", "scc");
}
__device__ __forceinline__ void draw_ticket(u32& t, u32* ctr) {
    u64 save;
    asm volatile("s_mov_b64 %1, exec\n\ts_mov_b64 exec, 1\n\tglobal_atomic_add %0, %2, %3, %4 sc0\n\ts_mov_b64 exec, %1"
                 : "=&v"(t), "=&s"(save) : "v"(0u), "v"(1u), "s"(ctr) : "memory");
}
template <int N>
__device__ __forceinline__ void wait_tk(u32& t) {
    asm volatile("s_waitcnt vmcnt(%1)" : "+v"(t) : "n"(N) : "memory");
}

__device__ __forceinline__ u32 pick_base(const u32 (&r)[16], u32* hist, int lane) {
    if (lane < 40) hist[lane] = 0;
    asm volatile("" ::: "memory");
    __hip_atomic_fetch_add(&hist[(r[0] >> 2) & 31], 1u, __ATOMIC_RELAXED, __HIP_MEMORY_SCOPE_WORKGROUP);
    __hip_atomic_fetch_add(&hist[(r[5] >> 10) & 31], 1u, __ATOMIC_RELAXED, __HIP_MEMORY_SCOPE_WORKGROUP);
    __hip_atomic_fetch_add(&hist[(r[10] >> 18) & 31], 1u, __ATOMIC_RELAXED, __HIP_MEMORY_SCOPE_WORKGROUP);
    __hip_atomic_fetch_add(&hist[(r[15] >> 26) & 31], 1u, __ATOMIC_RELAXED, __HIP_MEMORY_SCOPE_WORKGROUP);
    asm volatile("" ::: "memory");
    u32 wsum = 0;
    const int hl = lane & 31;
#pragma unroll
    for (int i = 0; i < 8; ++i) wsum += hist[hl + i];
    const u32 key = (lane <= 24) ? (wsum * 32 + (31 - lane)) : 0u;
    return 31 - (wave_max(key) & 31);
}

__device__ __forceinline__ void encode(Enc& e, const char* raw, unsigned short* list, u32* hist, int lane, u32& b) {
    const int q = lane & 15, g = lane >> 4;
    u32 r[16];
#pragma unroll
    for (int i = 0; i < 16; ++i) r[i] = *(const u32*)(raw + (g * 16 + i) * 64 + q * 4);
    u32 T[16];
#pragma unroll
    for (int t = 0; t < 4; ++t) {
        const u32 t0 = perm(r[4 * t + 1], r[4 * t + 0], 0x05010400u);
        const u32 t1 = perm(r[4 * t + 1], r[4 * t + 0], 0x07030602u);
        const u32 t2 = perm(r[4 * t + 3], r[4 * t + 2], 0x05010400u);
        const u32 t3 = perm(r[4 * t + 3], r[4 * t + 2], 0x07030602u);
        T[0 + t]  = perm(t2, t0, 0x05040100u);
        T[4 + t]  = perm(t2, t0, 0x07060302u);
        T[8 + t]  = perm(t3, t1, 0x05040100u);
        T[12 + t] = perm(t3, t1, 0x07060302u);
    }
    bool picked = false;
    if (b >= 32) { b = pick_base(r, hist, lane); picked = true; }
    u32 code[16], D[4], n, incl, total;
    while (true) {
        const u32 B4 = (b << 2) * 0x01010101u;
        const u32 NB4 = (((8 - b) & 7) << 2) * 0x01010101u;
        n = 0;
#pragma unroll
        for (int k = 0; k < 4; ++k) {
            u32 v[4];
#pragma unroll
            for (int t = 0; t < 4; ++t) {
                const int i = 4 * k + t;
                const u32 d4 = ((T[i] & 0x1C1C1C1Cu) + NB4) & 0x1C1C1C1Cu;
                code[i] = (T[i] & 0x83838383u) | d4;
                const u32 z = T[i] ^ (code[i] + B4);
                v[t] = __builtin_amdgcn_udot4(z, 0x40100401u, 0u, false);
            }
            D[k] = (v[0] >> 5) | (v[1] << 3) | (v[2] << 11) | (v[3] << 19);
            n += __builtin_popcount((D[k] | (D[k] >> 1)) & 0x55555555u);
        }
        incl = wave_incl_scan(n);
        total = __builtin_amdgcn_readlane(incl, 63);
        if (total <= REPICK || picked) break;
        picked = true;
        const u32 b2 = pick_base(r, hist, lane);
        if (b2 == b) break;
        b = b2;
    }
    e.total = total;
    if (total <= MAX_EXC) {
        u32 idx = 1 + incl - n;
        const u32 posb = ((g & 1) * 512 + (g >> 1) * 256 + q * 16) << 2;
#pragma unroll
        for (int k = 0; k < 4; ++k) {
            u32 dd = D[k];
            while (dd) {
                const u32 bit = __builtin_ctz(dd) & ~1u;
                list[idx++] = (unsigned short)(posb + k * 4096 + (bit << 1) + ((dd >> bit) & 3));
                dd &= ~(3u << bit);
            }
        }
        if (lane == 0) list[0] = (unsigned short)(b | (total << 6));
        u32 cw[12];
#pragma unroll
        for (int h = 0; h < 2; ++h) {
            const u32* cd = code + 8 * h;
            cw[6 * h + 0] = cd[0] | ((cd[6] & 0x03030303u) << 5);
            cw[6 * h + 1] = cd[1] | ((cd[6] & 0x0C0C0C0Cu) << 3);
            cw[6 * h + 2] = cd[2] | ((cd[6] & 0x10101010u) << 1) | ((cd[6] & 0x80808080u) >> 1);
            cw[6 * h + 3] = cd[3] | ((cd[7] & 0x03030303u) << 5);
            cw[6 * h + 4] = cd[4] | ((cd[7] & 0x0C0C0C0Cu) << 3);
            cw[6 * h + 5] = cd[5] | ((cd[7] & 0x10101010u) << 1) | ((cd[7] & 0x80808080u) >> 1);
        }
        asm volatile("" ::: "memory");
        e.hv = *(const u32x4*)((const char*)list + (lane & 7) * 16);
        e.c3 = *(const u32x4*)((const char*)list + 128 + (lane < 56 ? lane : 0) * 16);
        e.c0 = u32x4{cw[0], cw[1], cw[2], cw[3]};
        e.c1 = u32x4{cw[4], cw[5], cw[6], cw[7]};
        e.c2 = u32x4{cw[8], cw[9], cw[10], cw[11]};
        asm volatile("" ::: "memory");
    } else {
        e.hv = u32x4{(lane == 0) ? 32u : 0u, 0u, 0u, 0u};
        e.c0 = u32x4{T[0], T[1], T[2], T[3]};
        e.c1 = u32x4{T[4], T[5], T[6], T[7]};
        e.c2 = u32x4{T[8], T[9], T[10], T[11]};
        e.c3 = u32x4{T[12], T[13], T[14], T[15]};
    }
}

__device__ __forceinline__ void store_unit(const Enc& e, char* const (&dst)[NR], u32 slotoff, int lane) {
    const u32 off = slotoff + 128 + lane * 16;
#pragma unroll
    for (int d = 0; d < NR; ++d) st16_sys<0>(dst[d], off, e.c0);
#pragma unroll
    for (int d = 0; d < NR; ++d) st16_sys<1024>(dst[d], off, e.c1);
#pragma unroll
    for (int d = 0; d < NR; ++d) st16_sys<2048>(dst[d], off, e.c2);
    if ((u32)lane * 8 <= (e.total > MAX_EXC ? 0u : min(e.total, 63u))) {
#pragma unroll
        for (int d = 0; d < NR; ++d) st16_sys<0>(dst[d], slotoff + lane * 16, e.hv);
    }
    if (e.total > 63) {
        if (e.total > MAX_EXC || (u32)lane * 8 < e.total - 63) {
#pragma unroll
            for (int d = 0; d < NR; ++d) st16_sys<3072>(dst[d], off, e.c3);
        }
    }
}

__device__ __forceinline__ void flag_store(char* a, u32 call) {
    __hip_atomic_store((gu32*)(u64)a, call, __ATOMIC_RELAXED, __HIP_MEMORY_SCOPE_SYSTEM);
}

__device__ __forceinline__ void sender(const Params& p, char* lds, int wave) {
    const int lane = threadIdx.x & 63;
    const int wid = __builtin_amdgcn_readfirstlane(wave * NS + blockIdx.x);
    const int rank = p.rank;
    char* const myws = p.ws[rank];
    u32* const cnt = (u32*)(myws + CNT_OFF + (size_t)(blockIdx.x * 8 + wave) * 16);
    STAMP(blockIdx.x * 128 + 0);
    STAMP_HW(blockIdx.x * 128 + 15);
    char* const mylds = lds + wave * SND_LDS;
    char* const raw0 = mylds;
    unsigned short* const list = (unsigned short*)(mylds + 8192);
    u32* const hist = (u32*)(mylds + 8192 + 1280);
    const u32 rawaddr = __builtin_amdgcn_readfirstlane(lds_addr(raw0));
    const unsigned char* const wsrc = p.w + (size_t)(lane >> 2) * NN + (lane & 3) * 16;
    auto unit_src = [&](int u) { return wsrc + (size_t)(u / NC) * 64 * NN + (u % NC) * 64; };

    u32* const tick = (u32*)uni(myws + TICK_OFF);
    u32 tk;
    dma_raw(unit_src(wid), rawaddr);
    draw_ticket(tk, tick);

    const u32 call = __builtin_amdgcn_readfirstlane(*cnt + 1);
    const size_t par = DATA_OFF + (size_t)(call & 1) * PAR_BYTES;
    char* flagbase = p.ws[lane & (NR - 1)] + par + (size_t)rank * BLK + WF_OFF;
    char* dst[NR];
    const int rot = wid & (NR - 1);
#pragma unroll
    for (int d = 0; d < NR; ++d) dst[d] = uni(p.ws[(rank + 1 + d + rot) & (NR - 1)] + par);
    asm volatile("" : "+v"(flagbase));
    wait_tk<0>(tk);

    const u32 tbase = (call - 1) * (NU + NW) - NW;
    int u = wid;
    int un = (int)(__builtin_amdgcn_readfirstlane(tk) - tbase);
    int kb = 0;
    if ((u32)un < (u32)NU) dma_raw(unit_src(un), rawaddr + 4096);
    Enc e;
    u32 base = 32;
    encode(e, raw0, list, hist, lane, base);
    STAMP(blockIdx.x * 128 + 1);

    int uprev = -1;
    while (true) {
        draw_ticket(tk, tick);
        store_unit(e, dst, slot_off(rank, u), lane);
        wait_tk<32>(tk);
        if (uprev >= 0) {
            if (lane < NR) flag_store(flagbase + (size_t)uprev * 128, call);
        }
        if ((u32)un >= (u32)NU) break;
        const int unn = (int)(__builtin_amdgcn_readfirstlane(tk) - tbase);
        if ((u32)unn < (u32)NU)
            dma_raw(unit_src(unn), __builtin_amdgcn_readfirstlane(rawaddr + kb * 4096));
        kb ^= 1;
        encode(e, raw0 + kb * 4096, list, hist, lane, base);
        uprev = u;
        u = un;
        un = unn;
    }
    wait_vm0();
    if (lane < NR) flag_store(flagbase + (size_t)u * 128, call);
    if (lane == 0) *cnt = call;
}

__device__ __forceinline__ void decode_unit(char* img, __amdgpu_buffer_rsrc_t rs, u32 slot, int lane,
                                            const u32x4 (&cw)[3], u32 ent) {
    const u32 hdr = __builtin_amdgcn_readfirstlane(ent) & 0xFFFFu;
    const int q = lane & 15, g = lane >> 4;
    char* const wp = img + (g & 1) * 512 + (g >> 1) * 256 + q * 16;
    if (!(hdr & 32)) {
        const u32 B4 = ((hdr & 31) << 2) * 0x01010101u;
        const u32 c[12] = {cw[0].x, cw[0].y, cw[0].z, cw[0].w, cw[1].x, cw[1].y, cw[1].z, cw[1].w,
                           cw[2].x, cw[2].y, cw[2].z, cw[2].w};
#pragma unroll
        for (int h = 0; h < 2; ++h) {
            const u32* cc = c + 6 * h;
            u32x4 a, bq;
            a.x = (cc[0] & 0x9F9F9F9Fu) + B4;
            a.y = (cc[1] & 0x9F9F9F9Fu) + B4;
            a.z = (cc[2] & 0x9F9F9F9Fu) + B4;
            a.w = (cc[3] & 0x9F9F9F9Fu) + B4;
            bq.x = (cc[4] & 0x9F9F9F9Fu) + B4;
            bq.y = (cc[5] & 0x9F9F9F9Fu) + B4;
            bq.z = (((cc[0] >> 5) & 0x03030303u) | ((cc[1] >> 3) & 0x0C0C0C0Cu) |
                    ((cc[2] >> 1) & 0x10101010u) | ((cc[2] << 1) & 0x80808080u)) + B4;
            bq.w = (((cc[3] >> 5) & 0x03030303u) | ((cc[4] >> 3) & 0x0C0C0C0Cu) |
                    ((cc[5] >> 1) & 0x10101010u) | ((cc[5] << 1) & 0x80808080u)) + B4;
            *(u32x4*)(wp + (2 * h) * 1024) = a;
            *(u32x4*)(wp + (2 * h + 1) * 1024) = bq;
        }
        const u32 total = hdr >> 6;
        {
            const u32 e = (lane == 0 || (u32)lane > total) ? 0u : (ent & 0xFFFFu);
            const u32 pos = e >> 2;
            const u32 x = (e & 3) << (5 + 8 * (pos & 3));
            __hip_atomic_fetch_xor((u32*)(img + (pos & ~3u)), x, __ATOMIC_RELAXED, __HIP_MEMORY_SCOPE_WORKGROUP);
        }
#pragma clang loop unroll(disable)
        for (u32 i0 = 63; i0 < total; i0 += 64) {
            const u32 i = i0 + lane;
            u32 e = __builtin_amdgcn_raw_buffer_load_b16(rs, slot + 128 + 3072 + (i - 63) * 2, 0, AUX_SYS);
            e = (i < total) ? (e & 0xFFFFu) : 0u;
            const u32 pos = e >> 2;
            const u32 x = (e & 3) << (5 + 8 * (pos & 3));
            __hip_atomic_fetch_xor((u32*)(img + (pos & ~3u)), x, __ATOMIC_RELAXED, __HIP_MEMORY_SCOPE_WORKGROUP);
        }
    } else {
        const u32x4 v3 = __builtin_amdgcn_raw_buffer_load_b128(rs, slot + 128 + 3072 + lane * 16, 0, AUX_SYS);
        *(u32x4*)(wp + 0) = cw[0];
        *(u32x4*)(wp + 1024) = cw[1];
        *(u32x4*)(wp + 2048) = cw[2];
        *(u32x4*)(wp + 3072) = v3;
    }
}

__device__ __forceinline__ void send_x(const Params& p, u32 call, size_t par, int h0, int lane) {
    u32x4 v[XN][8];
    char* base[XN];
    char* xf[XN];
#pragma unroll
    for (int n = 0; n < XN; ++n) {
        const int dest = __builtin_amdgcn_readfirstlane((p.rank + 1 + h0 + n) & (NR - 1));
        const int rf = __builtin_amdgcn_readfirstlane(((h0 + n) >> 3) & 31);
        const unsigned char* src = p.x + (size_t)(dest * MS + rf * 16 + (lane & 15)) * KS
                                 + ((lane >> 4) & 1) * 32 + (lane >> 5) * 16;
#pragma unroll
        for (int j = 0; j < 8; ++j) v[n][j] = *(const u32x4*)(src + j * 64);
        base[n] = uni(p.ws[dest] + par + (size_t)p.rank * BLK + X_OFF + rf * 8192);
        xf[n] = uni(p.ws[dest] + par + (size_t)p.rank * BLK + XF_OFF + rf * 128);
    }
#pragma unroll
    for (int n = 0; n < XN; ++n)
#pragma unroll
        for (int j = 0; j < 8; ++j) asm volatile("" :: "v"(v[n][j]));
#pragma unroll
    for (int n = 0; n < XN; ++n)
#pragma unroll
        for (int j = 0; j < 8; j += 4) {
            st16_sys<0>(base[n] + j * 1024, lane * 16, v[n][j]);
            st16_sys<1024>(base[n] + j * 1024, lane * 16, v[n][j + 1]);
            st16_sys<2048>(base[n] + j * 1024, lane * 16, v[n][j + 2]);
            st16_sys<3072>(base[n] + j * 1024, lane * 16, v[n][j + 3]);
        }
    wait_vm0();
    if (lane == 0) {
#pragma unroll
        for (int n = 0; n < XN; ++n)
            __hip_atomic_store((gu32*)(u64)xf[n], call, __ATOMIC_RELAXED, __HIP_MEMORY_SCOPE_SYSTEM);
    }
}

template <int NF>
__device__ __forceinline__ void consumer_wave(const Params& p, char* lds, int c, int wave, int rf0, int lf0, u32 call, size_t par) {
    const int lane = threadIdx.x & 63;
    char* const myws = uni(p.ws[p.rank]);
    gu32* const tmo = (gu32*)(u64)(myws + TMO_OFF);
    const float scale = p.sx[0] * p.sw[0];
    const __amdgpu_buffer_rsrc_t rs =
        __builtin_amdgcn_make_buffer_rsrc(uni(myws + par), 0, (int)PAR_BYTES, 0x00020000);
    u32* const badw = (u32*)(lds + 65536);
    u32* const rdy = (u32*)(lds + 65536 + 32);
    const __amdgpu_buffer_rsrc_t ro = __builtin_amdgcn_make_buffer_rsrc(uni((char*)p.out), 0, MS * NN * 4, 0x00020000);

    f32x4 acc[NF][4];

    const u32 xoff = (u32)((lane >> 5) * BLK + X_OFF + rf0 * 8192 + (lane & 31) * 16);
    const int L16 = lane & 15, G = lane >> 4;
    const u32 bread = (G >> 1) * 4096 + (G & 1) * 256 + L16 * 16;

    u32x4 A[4 * NF * 2];
    auto load_a = [&](int P, int j) {
#pragma unroll
        for (int a = 0; a < NF; ++a)
#pragma unroll
            for (int i = 0; i < 2; ++i)
                A[(P * NF + a) * 2 + i] = __builtin_amdgcn_raw_buffer_load_b128(
                    rs, xoff + j * 1024 + i * 512, P * 2 * BLK + a * 8192, AUX_SYS);
    };
    char* const a7 = lds + A7_OFF + lf0 * 8192;
    auto dma_a7 = [&]() {
        const char* src = myws + par + xoff + (NJ - 1) * 1024;
        asm volatile("" : "+v"(src));
        const u32 dst = __builtin_amdgcn_readfirstlane(lds_addr(a7));
#pragma unroll
        for (int a = 0; a < NF; ++a)
#pragma unroll
            for (int P = 0; P < 4; ++P) {
                u32 keep;
                const char* s0 = src + a * 8192 + (size_t)P * 2 * BLK;
                asm volatile("s_mov_b32 %0, m0\n\ts_mov_b32 m0, %2\n\ts_nop 0\n\tglobal_load_lds_dwordx4 %1, off sc0 sc1\n\t"
                             "s_add_u32 m0, m0, 0x400\n\ts_nop 0\n\tglobal_load_lds_dwordx4 %3, off sc0 sc1\n\ts_mov_b32 m0, %0"
                             : "=&s"(keep) : "v"(s0), "s"(dst + a * 8192 + P * 2048), "v"(s0 + 512) : "memory", "scc");
            }
    };
    auto mfma = [&](int P, int f, const u32x4& b0, const u32x4& b1) {
        const i32x8 B = {(int)b0.x, (int)b0.y, (int)b0.z, (int)b0.w, (int)b1.x, (int)b1.y, (int)b1.z, (int)b1.w};
#pragma unroll
        for (int a = 0; a < NF; ++a) {
            const u32x4 a0 = A[(P * NF + a) * 2], a1 = A[(P * NF + a) * 2 + 1];
            const i32x8 Av = {(int)a0.x, (int)a0.y, (int)a0.z, (int)a0.w, (int)a1.x, (int)a1.y, (int)a1.z, (int)a1.w};
            acc[a][f] = __builtin_amdgcn_mfma_scale_f32_16x16x128_f8f6f4(Av, B, acc[a][f], 0, 1, 0, 0, 0, 0);
        }
    };
    u32x4 cw[3];
    u32 ent;
    auto unit_slot = [&](int j) { return slot_off(wave, j * NC + c); };
    auto issue = [&](int j, u32 oob) {
        const u32 slot = unit_slot(j) | oob;
        ent = __builtin_amdgcn_raw_buffer_load_b16(rs, slot + lane * 2, 0, AUX_SYS);
#pragma unroll
        for (int i = 0; i < 3; ++i)
            cw[i] = __builtin_amdgcn_raw_buffer_load_b128(rs, slot + 128 + i * 1024 + lane * 16, 0, AUX_SYS);
    };
    auto decode = [&](int j, u32 oob) {
        decode_unit(lds + (j & 1) * 32768 + wave * 4096, rs, unit_slot(j) | oob, lane, cw, ent);
    };
    auto wflag = [&](int j) { return (const gu32*)(u64)(myws + par + (size_t)wave * BLK + WF_OFF + (j * NC + c) * 128); };
    bool bad = false;

    bool have = false;
    auto round = [&](int j) {
        if (!have) {
            bad |= wait_flags(wflag(j), lane == 0, call, tmo);
            issue(j, 0);
        }
        decode(j, 0);
        __syncthreads();
        if (j == NJ - 3) {
            const u32 dummy = __builtin_amdgcn_raw_buffer_load_b32(ro, (u32)(((rf0 * 16 + G * 4) * NN + c * 64 + L16 * 4) * 4), 0, 16);
            asm volatile("" :: "v"(dummy));
        }
        u32 pv = __hip_atomic_load(wflag(j + 1), __ATOMIC_RELAXED, __HIP_MEMORY_SCOPE_SYSTEM);
        const char* const buf = lds + (j & 1) * 32768;
        u32x4 Bf[2][2][2];
#pragma unroll
        for (int f = 0; f < 2; ++f) {
            Bf[0][f][0] = *(const u32x4*)(buf + f * 1024 + bread);
            Bf[0][f][1] = *(const u32x4*)(buf + f * 1024 + bread + 512);
        }
#pragma unroll
        for (int st = 0; st < 8; ++st) {
            if (st < 7) {
                const int Pn = (st + 1) >> 1, fn = ((st + 1) & 1) * 2;
#pragma unroll
                for (int f = 0; f < 2; ++f) {
                    Bf[(st + 1) & 1][f][0] = *(const u32x4*)(buf + Pn * 8192 + (fn + f) * 1024 + bread);
                    Bf[(st + 1) & 1][f][1] = *(const u32x4*)(buf + Pn * 8192 + (fn + f) * 1024 + bread + 512);
                }
            }
            __builtin_amdgcn_sched_barrier(0);
#pragma unroll
            for (int f = 0; f < 2; ++f) mfma(st >> 1, (st & 1) * 2 + f, Bf[st & 1][f][0], Bf[st & 1][f][1]);
            if ((st & 1) && j + 2 < NJ) load_a(st >> 1, j + 1);
            if (st == EARLY_ST) {
                have = __builtin_amdgcn_readfirstlane(pv) >= call;
                asm volatile("" ::: "memory");
                if (have) issue(j + 1, 0);
                else pv = __hip_atomic_load(wflag(j + 1), __ATOMIC_RELAXED, __HIP_MEMORY_SCOPE_SYSTEM);
            }
            if (st == EARLY_ST2 && !have) {
                have = __builtin_amdgcn_readfirstlane(pv) >= call;
                asm volatile("" ::: "memory");
                if (have) issue(j + 1, 0);
            }
            __builtin_amdgcn_sched_barrier(0);
        }
        STAMP(blockIdx.x * 128 + wave * 16 + 2 + j);
    };

#pragma clang loop unroll(disable)
    for (u32 pass = 0; pass < 2; ++pass) {
        asm volatile("" : "+s"(pass));
        const u32 oob = pass ? 0u : 0x80000000u;
#pragma unroll
        for (int a = 0; a < NF; ++a)
#pragma unroll
            for (int f = 0; f < 4; ++f) acc[a][f] = f32x4{0.f, 0.f, 0.f, 0.f};
        if (pass) {
            bad = wait_flags((const gu32*)(u64)(myws + par + (size_t)(lane & 7) * BLK + XF_OFF + (rf0 + ((lane >> 3) & (NF - 1))) * 128),
                             lane < NF * NR, call, tmo);
            STAMP(blockIdx.x * 128 + wave * 16 + 1);
            dma_a7();
#pragma unroll
            for (int P = 0; P < 4; ++P) load_a(P, 0);
#pragma clang loop unroll(disable)
            for (int j = 0; j < NJ - 1; ++j) round(j);
            asm volatile("s_waitcnt vmcnt(0)" ::: "memory");
        }

        {
            constexpr int j = NJ - 1;
            const char* const bbuf = lds + (j & 1) * 32768 + bread;
            const char* const abuf = a7 + lane * 16;
            const gu32* const fl = wflag(j);
            const u64 t0 = __builtin_amdgcn_s_memrealtime();
            asm volatile("" :: "s"(t0));
            bool own = false;
            u32 todo = 15, avail = 0, it = 0;
            auto pick = [&]() {
                if (!pass) { own = false; avail = 0; todo = 15; }
                while (!avail) {
                    u32 v = call;
                    if (!own && pass && !have) v = __hip_atomic_load(fl, __ATOMIC_RELAXED, __HIP_MEMORY_SCOPE_SYSTEM);
                    const u32x4 r0 = *(const u32x4*)rdy, r1 = *(const u32x4*)(rdy + 4);
                    const u32 m = (min(r0.x, r0.y) > pass ? 1u : 0u) | (min(r0.z, r0.w) > pass ? 2u : 0u) |
                                  (min(r1.x, r1.y) > pass ? 4u : 0u) | (min(r1.z, r1.w) > pass ? 8u : 0u);
                    avail = __builtin_amdgcn_readfirstlane(m & todo);
                    todo &= ~avail;
                    if (!own) {
                        bool go = __builtin_amdgcn_readfirstlane(v) >= call;
                        if (!go && (++it & 15) == 15 &&
                            (__hip_atomic_load(tmo, __ATOMIC_RELAXED, __HIP_MEMORY_SCOPE_AGENT) == call ||
                             __builtin_amdgcn_s_memrealtime() - t0 > DEADLINE)) {
                            __hip_atomic_store(tmo, call, __ATOMIC_RELAXED, __HIP_MEMORY_SCOPE_AGENT);
                            bad = true;
                            go = true;
                        }
                        if (go) {
                            asm volatile("" ::: "memory");
                            STAMP(blockIdx.x * 128 + wave * 16 + 11);
                            if (!have) issue(j, oob);
                            decode(j, oob);
                            if (lane == 0) {
                                badw[wave] = bad;
                                __hip_atomic_store(&rdy[wave], pass + 1, __ATOMIC_RELEASE, __HIP_MEMORY_SCOPE_WORKGROUP);
                            }
                            STAMP(blockIdx.x * 128 + wave * 16 + 13);
                            own = true;
                        }
                    } else if (!avail) {
                        __builtin_amdgcn_s_sleep(1);
                    }
                }
                asm volatile("" ::: "memory");
                const int P = __builtin_ctz(avail);
                avail &= avail - 1;
                return P;
            };
            struct Frags { u32x4 a[NF][2], b[4][2]; };
            auto rd_a = [&](Frags& F, int P, int a) {
                F.a[a][0] = *(const u32x4*)(abuf + a * 8192 + P * 2048);
                F.a[a][1] = *(const u32x4*)(abuf + a * 8192 + P * 2048 + 1024);
            };
            auto rd_b = [&](Frags& F, int P, int f) {
                F.b[f][0] = *(const u32x4*)(bbuf + P * 8192 + f * 1024);
                F.b[f][1] = *(const u32x4*)(bbuf + P * 8192 + f * 1024 + 512);
            };
            auto mm_f = [&](const Frags& F, int f) {
                const i32x8 B = {(int)F.b[f][0].x, (int)F.b[f][0].y, (int)F.b[f][0].z, (int)F.b[f][0].w,
                                 (int)F.b[f][1].x, (int)F.b[f][1].y, (int)F.b[f][1].z, (int)F.b[f][1].w};
#pragma unroll
                for (int a = 0; a < NF; ++a) {
                    const i32x8 Av = {(int)F.a[a][0].x, (int)F.a[a][0].y, (int)F.a[a][0].z, (int)F.a[a][0].w,
                                      (int)F.a[a][1].x, (int)F.a[a][1].y, (int)F.a[a][1].z, (int)F.a[a][1].w};
                    acc[a][f] = __builtin_amdgcn_mfma_scale_f32_16x16x128_f8f6f4(Av, B, acc[a][f], 0, 1, 0, 0, 0, 0);
                }
            };
            auto mmrd = [&](const Frags& Fc, Frags& Fn, int P) {
#pragma unroll
                for (int f = 0; f < 4; ++f) {
                    mm_f(Fc, f);
                    if (f < NF) rd_a(Fn, P, f);
                    rd_b(Fn, P, f);
                    __builtin_amdgcn_sched_barrier(0);
                }
            };
            Frags F0, F1;
            {
                const int P = pick();
#pragma unroll
                for (int a = 0; a < NF; ++a) rd_a(F0, P, a);
#pragma unroll
                for (int f = 0; f < 4; ++f) rd_b(F0, P, f);
            }
            mmrd(F0, F1, pick());
            mmrd(F1, F0, pick());
            mmrd(F0, F1, pick());
#pragma unroll
            for (int f = 0; f < 4; ++f) mm_f(F1, f);
        }
        STAMP(blockIdx.x * 128 + wave * 16 + 9);

        const u32x4 b0 = *(const u32x4*)badw, b1 = *(const u32x4*)(badw + 4);
        const float s = (b0.x | b0.y | b0.z | b0.w | b1.x | b1.y | b1.z | b1.w) ? __builtin_nanf("") : scale;
        const u32 obase = (u32)(((rf0 * 16 + G * 4) * NN + c * 64 + L16 * 4) * 4) | oob;
#pragma unroll
        for (int a = 0; a < NF; ++a)
#pragma unroll
            for (int r = 0; r < 4; ++r) {
                const f32x4 v = {acc[a][0][r] * s, acc[a][1][r] * s, acc[a][2][r] * s, acc[a][3][r] * s};
                __builtin_amdgcn_raw_buffer_store_b128(__builtin_bit_cast(u32x4, v), ro, obase + (a * 16 + r) * NN * 4, 0, OUT_AUX);
            }
        STAMP(blockIdx.x * 128 + wave * 16 + 10);
    }
}

__device__ __forceinline__ void consumer(const Params& p, char* lds) {
    const int lane = threadIdx.x & 63;
    const int wave = __builtin_amdgcn_readfirstlane(threadIdx.x >> 6);
    const int t = blockIdx.x - NS;
    const int x = t & 7, i = t >> 3;
    int c, f0, nfr;
    if (i < 12) { c = 4 * x + i / 3; f0 = (i % 3) * 11; nfr = (i % 3 == 2) ? 10 : 11; }
    else        { c = 32 + 4 * x + ((i - 12) >> 2); f0 = ((i - 12) & 3) * 8; nfr = 8; }
    const int n2 = nfr - 8;
    const int lf0 = wave < n2 ? 2 * wave : n2 + wave;
    char* const myws = p.ws[p.rank];
    u32* const cnt = (u32*)(myws + CNT_OFF + (size_t)(blockIdx.x * 8 + wave) * 16);
    const u32 call = __builtin_amdgcn_readfirstlane(*cnt + 1);
    const size_t par = DATA_OFF + (size_t)(call & 1) * PAR_BYTES;
    STAMP(blockIdx.x * 128 + wave * 16 + 0);
    STAMP_HW(blockIdx.x * 128 + wave * 16 + 15);
    if (lane == 0) ((u32*)(lds + 65536 + 32))[wave] = 0;
    if (wave == (t & 7)) send_x(p, call, par, blockIdx.x, lane);
    if (wave < n2) consumer_wave<2>(p, lds, c, wave, f0 + lf0, lf0, call, par);
    else consumer_wave<1>(p, lds, c, wave, f0 + lf0, lf0, call, par);
    if (lane == 0) *cnt = call;
}

__global__ void __launch_bounds__(512, 2) gemm_rs_kernel(Params p) {
    __shared__ __attribute__((aligned(16))) char lds[LDS_BYTES];
    if (blockIdx.x < NS) {
        const int wave = threadIdx.x >> 6;
        if (wave < SW) {
            if (wave * NS + (int)blockIdx.x < NW) sender(p, lds, wave);
        } else if (wave == 7) {
            const int lane = threadIdx.x & 63;
            u32* const cnt = (u32*)(p.ws[p.rank] + CNT_OFF + (size_t)(blockIdx.x * 8 + wave) * 16);
            const u32 call = __builtin_amdgcn_readfirstlane(*cnt + 1);
            send_x(p, call, DATA_OFF + (size_t)(call & 1) * PAR_BYTES, blockIdx.x, lane);
            if (lane == 0) *cnt = call;
        }
    } else {
        consumer(p, lds);
    }
}

}

extern "C" void kernel_launch(int rank, int nranks, ncclComm_t comm, hipStream_t stream,
                              void* const* d_in, const int* in_sizes, int n_in,
                              void* d_out, int out_size, const int* in_dtypes, int out_dtype,
                              void* const* d_out_all, void* const* d_ws_all, size_t ws_bytes) {
    (void)comm; (void)in_dtypes; (void)out_dtype; (void)d_out_all;
    if (nranks != NR || n_in != 4 || rank < 0 || rank >= NR || ws_bytes < WS_NEED ||
        in_sizes[0] != 4096 * KS || in_sizes[1] != KS * NN || out_size != MS * NN) {
        fprintf(stderr, "gemm_rs: unexpected problem shape\n");
        return;
    }
    Params p;
    p.x = (const unsigned char*)d_in[0];
    p.w = (const unsigned char*)d_in[1];
    p.sx = (const float*)d_in[2];
    p.sw = (const float*)d_in[3];
    p.out = (float*)d_out;
    for (int r = 0; r < NR; ++r) p.ws[r] = (char*)d_ws_all[r];
    p.rank = rank;
    p.pad = 0;
    hipLaunchKernelGGL(gemm_rs_kernel, dim3(GRID), dim3(512), 0, stream, p);
    HIP_CHECK(hipGetLastError());
}
```

```cpp
#include <hip/hip_runtime.h>
#include <rccl/rccl.h>
#include <cstdio>
#include <cstdint>

#define HIP_CHECK(expr) do { hipError_t _e = (expr); if (_e != hipSuccess) { \
    fprintf(stderr, "HIP error %s at %s:%d\n", hipGetErrorString(_e), __FILE__, __LINE__); \
    return; } } while (0)

namespace {

typedef unsigned u32;
typedef unsigned long long u64;
typedef __attribute__((address_space(1))) unsigned gu32;
typedef unsigned u32x4 __attribute__((ext_vector_type(4)));
typedef unsigned u32x2 __attribute__((ext_vector_type(2)));
typedef unsigned u32x3 __attribute__((ext_vector_type(3)));
typedef int i32x8 __attribute__((ext_vector_type(8)));
typedef float f32x4 __attribute__((ext_vector_type(4)));

constexpr int NR = 8;
constexpr int MS = 512;
constexpr int KS = 512;
constexpr int NN = 4096;
constexpr int NJ = 8;
constexpr int NC = 64;
constexpr int NU = NJ * NC;

#define NSEND 32
#define NENC 5
#define RINGN 4
#define RINGL 3
#define RINGL_FROM 96
#define FLUSH_IDLE 1
constexpr int NS = NSEND;
constexpr int NE = NENC;
constexpr int RING = RINGN;
constexpr int NW = NS;
constexpr int NCONS = 224;
constexpr int GRID = NS + NCONS;
constexpr int XN = 1;
static_assert(GRID == 256 && NE <= 6, "one workgroup per CU, 256 x half-units");

constexpr size_t CNT_OFF   = 0;
constexpr size_t TMO_OFF   = 64 << 10;
constexpr size_t TICK_OFF  = 96 << 10;
constexpr size_t DATA_OFF  = 1 << 20;
constexpr u32    SLOT      = 4224;
constexpr u32    WF_OFF    = 0;
constexpr u32    XF_OFF    = 64 << 10;
constexpr u32    X_OFF     = 128 << 10;
constexpr u32    W_OFF     = 384 << 10;
constexpr u32    BLK       = 2560 << 10;
constexpr size_t PAR_BYTES = (size_t)NR * BLK;
constexpr size_t WS_NEED   = DATA_OFF + 2 * PAR_BYTES;
static_assert(W_OFF + NU * SLOT <= BLK, "block too small");
static_assert((size_t)GRID * 8 * 16 <= TMO_OFF, "counter region too small");

constexpr u64 DEADLINE = 40000000ull;
constexpr int AUX_SYS = 17;
#define PRETOUCH 1
#define W5_ROUNDS 5
constexpr int W5_UNITS = W5_ROUNDS * NC;
#define EARLY_ST 2
#define EARLY_ST2 7
#define OUT_AUX 16

struct Params {
    const unsigned char* x;
    const unsigned char* w;
    const float* sx;
    const float* sw;
    float* out;
    char* ws[NR];
    int rank;
    int pad;
};

__device__ __forceinline__ u32 slot_off(int src, int u) { return (u32)src * BLK + W_OFF + (u32)u * SLOT; }
template <int N>
__device__ __forceinline__ u32 shl(u32 x) {
    u32 r;
    asm("v_lshlrev_b32 %0, %1, %2" : "=v"(r) : "i"(N), "v"(x));
    return r;
}
__device__ __forceinline__ u32 perm(u32 hi, u32 lo, u32 sel) { return __builtin_amdgcn_perm(hi, lo, sel); }

template <int IMM>
__device__ __forceinline__ void st16_sys(char* base, u32 off, u32x4 v) {
    asm volatile("global_store_dwordx4 %0, %1, %2 offset:%3 sc0 sc1\n\ts_nop 1"
                 :: "v"(off), "v"(v), "s"(base), "i"(IMM) : "memory");
}
__device__ __forceinline__ char* uni(char* p) {
    const u64 v = (u64)p;
    const u32 lo = __builtin_amdgcn_readfirstlane((u32)v);
    const u32 hi = __builtin_amdgcn_readfirstlane((u32)(v >> 32));
    char* r = (char*)(((u64)hi << 32) | lo);
    asm volatile("s_nop 4" : "+s"(r));
    return r;
}
__device__ __forceinline__ void wait_vm0() { asm volatile("s_waitcnt vmcnt(0)" ::: "memory"); }

#define STAMP(slot) do {} while (0)
#define STAMP_HW(slot) do {} while (0)

__device__ __forceinline__ bool wait_flags(const gu32* flag, bool active, u32 call, gu32* tmo) {
    const u64 t0 = __builtin_amdgcn_s_memrealtime();
    bool bad = false;
    for (u32 it = 0;; ++it) {
        u32 v = call;
        if (active) v = __hip_atomic_load(flag, __ATOMIC_RELAXED, __HIP_MEMORY_SCOPE_SYSTEM);
        if (__all(v >= call)) break;
        if ((it & 15) == 15) {
            if (__hip_atomic_load(tmo, __ATOMIC_RELAXED, __HIP_MEMORY_SCOPE_AGENT) == call ||
                __builtin_amdgcn_s_memrealtime() - t0 > DEADLINE) {
                __hip_atomic_store(tmo, call, __ATOMIC_RELAXED, __HIP_MEMORY_SCOPE_AGENT);
                bad = true;
                break;
            }
        }
        __builtin_amdgcn_s_sleep(2);
    }
    asm volatile("" ::: "memory");
    return bad;
}

template <int CTRL, int RMASK>
__device__ __forceinline__ u32 dpp0(u32 v) {
    return __builtin_amdgcn_update_dpp(0u, v, CTRL, RMASK, 0xf, false);
}
__device__ __forceinline__ u32 wave_incl_scan(u32 v) {
    v += dpp0<0x111, 0xf>(v);
    v += dpp0<0x112, 0xf>(v);
    v += dpp0<0x114, 0xf>(v);
    v += dpp0<0x118, 0xf>(v);
    v += dpp0<0x142, 0xa>(v);
    v += dpp0<0x143, 0xc>(v);
    return v;
}
__device__ __forceinline__ u32 wave_max(u32 v) {
    v = max(v, dpp0<0x111, 0xf>(v));
    v = max(v, dpp0<0x112, 0xf>(v));
    v = max(v, dpp0<0x114, 0xf>(v));
    v = max(v, dpp0<0x118, 0xf>(v));
    v = max(v, dpp0<0x142, 0xa>(v));
    v = max(v, dpp0<0x143, 0xc>(v));
    return __builtin_amdgcn_readlane(v, 63);
}

constexpr int A7_OFF = 65536 + 256;
constexpr int LDS_BYTES = A7_OFF + 11 * 8192;

constexpr u32 MODE_W6 = 0, MODE_RAW = 1, MODE_W5 = 2;
constexpr u32 NINL = 62;
constexpr u32 MAX_EXC = 510;
constexpr u32 REPICK = 127;
constexpr u32 MAX_ESC = 1344;

struct Enc {
    u32x4 c0, c1, c2, c3;
    u32x4 hv;
    u32 lh, l2, l3;
};

typedef __attribute__((address_space(3))) char lchar;
__device__ __forceinline__ u32 lds_addr(char* p) { return (u32)(u64)(lchar*)p; }
__device__ __forceinline__ void lds_xor(u32 addr, u32 v) {
    __hip_atomic_fetch_xor((__attribute__((address_space(3))) u32*)(u64)addr, v, __ATOMIC_RELAXED, __HIP_MEMORY_SCOPE_WORKGROUP);
}
__device__ __forceinline__ void lds_or(u32 addr, u32 v) {
    __hip_atomic_fetch_or((__attribute__((address_space(3))) u32*)(u64)addr, v, __ATOMIC_RELAXED, __HIP_MEMORY_SCOPE_WORKGROUP);
}

__device__ __forceinline__ void dma_raw(const unsigned char* src, u32 ldsdst) {
    u32 keep;
    const unsigned char* s1 = src + 16 * NN;
    const unsigned char* s2 = src + 32 * NN;
    const unsigned char* s3 = src + 48 * NN;
    asm volatile("s_mov_b32 %0, m0\n\t"
                 "s_mov_b32 m0, %5\n\ts_nop 0\n\tglobal_load_lds_dwordx4 %1, off\n\t"
                 "s_add_u32 m0, m0, 0x400\n\ts_nop 0\n\tglobal_load_lds_dwordx4 %2, off\n\t"
                 "s_add_u32 m0, m0, 0x400\n\ts_nop 0\n\tglobal_load_lds_dwordx4 %3, off\n\t"
                 "s_add_u32 m0, m0, 0x400\n\ts_nop 0\n\tglobal_load_lds_dwordx4 %4, off\n\t"
                 "s_mov_b32 m0, %0"
                 : "=&s"(keep) : "v"(src), "v"(s1), "v"(s2), "v"(s3), "s"(ldsdst) : "memory", "scc");
}
__device__ __forceinline__ void draw_ticket(u32& t, u32* ctr) {
    u64 save;
    asm volatile("s_mov_b64 %1, exec\n\ts_mov_b64 exec, 1\n\tglobal_atomic_add %0, %2, %3, %4 sc0\n\ts_mov_b64 exec, %1"
                 : "=&v"(t), "=&s"(save) : "v"(0u), "v"(1u), "s"(ctr) : "memory");
}
template <int N>
__device__ __forceinline__ void wait_tk(u32& t) {
    asm volatile("s_waitcnt vmcnt(%1)" : "+v"(t) : "n"(N) : "memory");
}

__device__ __forceinline__ void pick_bases(const u32 (&r)[16], u32* hist, int lane, u32& b8, u32& b3) {
    if (lane < 40) hist[lane] = 0;
    asm volatile("" ::: "memory");
    __hip_atomic_fetch_add(&hist[(r[0] >> 2) & 31], 1u, __ATOMIC_RELAXED, __HIP_MEMORY_SCOPE_WORKGROUP);
    __hip_atomic_fetch_add(&hist[(r[5] >> 10) & 31], 1u, __ATOMIC_RELAXED, __HIP_MEMORY_SCOPE_WORKGROUP);
    __hip_atomic_fetch_add(&hist[(r[10] >> 18) & 31], 1u, __ATOMIC_RELAXED, __HIP_MEMORY_SCOPE_WORKGROUP);
    __hip_atomic_fetch_add(&hist[(r[15] >> 26) & 31], 1u, __ATOMIC_RELAXED, __HIP_MEMORY_SCOPE_WORKGROUP);
    asm volatile("" ::: "memory");
    u32 h[8];
    const int hl = lane & 31;
#pragma unroll
    for (int i = 0; i < 8; ++i) h[i] = hist[hl + i];
    const u32 s3 = h[0] + h[1] + h[2];
    const u32 s8 = s3 + h[3] + h[4] + h[5] + h[6] + h[7];
    b8 = 31 - (wave_max((lane <= 24) ? (s8 * 32 + (31 - lane)) : 0u) & 31);
    b3 = 31 - (wave_max((lane <= 28) ? (s3 * 32 + (31 - lane)) : 0u) & 31);
}

__device__ __forceinline__ void w5_table(u32 b, u32& lo, u32& hi) {
    const u32 B4 = (b << 2) * 0x01010101u;
    lo = (((B4 | 0x80808080u) - 0x0C080400u) & 0x7C7C7C00u) | ((b + 3) << 2);
    hi = ((B4 | 0x80808080u) - 0x1C181410u) & 0x7C7C7C7Cu;
}

struct SendLds {
    char* raw;
    unsigned short* list;
    u32* hist;
    char* stage;
    const u32* lut;
};
__device__ __forceinline__ SendLds send_lds(char* base) {
    return SendLds{base, (unsigned short*)(base + 8192), (u32*)(base + 9344), base + 9536, (const u32*)(base + 10560)};
}
constexpr int SND_LDS = 10752;

__device__ __forceinline__ void encode(Enc& e, const char* raw, const SendLds& L, int lane, u32& b8, u32& b3, bool allow5) {
    const int q = lane & 15, g = lane >> 4;
    unsigned short* const list = L.list;
    u32 r[16];
#pragma unroll
    for (int i = 0; i < 16; ++i) r[i] = *(const u32*)(raw + (g * 16 + i) * 64 + q * 4);
    u32 T[16];
#pragma unroll
    for (int t = 0; t < 4; ++t) {
        const u32 t0 = perm(r[4 * t + 1], r[4 * t + 0], 0x05010400u);
        const u32 t1 = perm(r[4 * t + 1], r[4 * t + 0], 0x07030602u);
        const u32 t2 = perm(r[4 * t + 3], r[4 * t + 2], 0x05010400u);
        const u32 t3 = perm(r[4 * t + 3], r[4 * t + 2], 0x07030602u);
        T[0 + t]  = perm(t2, t0, 0x05040100u);
        T[4 + t]  = perm(t2, t0, 0x07060302u);
        T[8 + t]  = perm(t3, t1, 0x05040100u);
        T[12 + t] = perm(t3, t1, 0x07060302u);
    }
    bool picked = false;
    if (b8 >= 32) { pick_bases(r, L.hist, lane, b8, b3); picked = true; }
    const u32 posb = ((g & 1) * 512 + (g >> 1) * 256 + q * 16) << 4;

    if (allow5) {
        u32 code[16], kb[16], em[2], xm[2], cnt, incl, nesc, nexc;
        while (true) {
            const u32 B4 = (b3 << 2) * 0x01010101u;
            em[0] = em[1] = xm[0] = xm[1] = 0;
#pragma unroll
            for (int i = 0; i < 16; ++i) {
                const u32 t = T[i];
                const u32 u = ((t & 0x7C7C7C7Cu) | 0x80808080u) - B4;
                const u32 bl = (~u >> 7) & 0x01010101u;
                const u32 v = u & 0x7C7C7C7Cu;
                const u32 esc = (((v + 0x74747474u) >> 7) & 0x01010101u) | bl;
                code[i] = (t & 0x83838383u) | (v & 0x0C0C0C0Cu) | shl<3>(esc) | shl<2>(esc);
                const u32 k4 = (0x80808080u - v) & 0x7C7C7C7Cu;
                const u32 big = shl<1>(k4) | (k4 << 2), ge4 = v + 0x70707070u;
                const u32 m8 = bl << 7;
                const u32 exc = (((m8 & big) | (~m8 & ge4)) >> 7) & 0x01010101u;
                kb[i] = (k4 >> 2) & (shl<3>(bl) - bl);
                em[i >> 3] |= __builtin_amdgcn_udot4(esc, 0x08040201u, 0u, false) << (4 * (i & 7));
                xm[i >> 3] |= __builtin_amdgcn_udot4(exc, 0x08040201u, 0u, false) << (4 * (i & 7));
            }
            const u32 ne = __builtin_popcount(em[0]) + __builtin_popcount(em[1]);
            cnt = ne | ((__builtin_popcount(xm[0]) + __builtin_popcount(xm[1])) << 16);
            incl = wave_incl_scan(cnt);
            const u32 tot = __builtin_amdgcn_readlane(incl, 63);
            nesc = tot & 0xFFFFu;
            nexc = tot >> 16;
            if (__builtin_amdgcn_ballot_w64(ne > 32)) nesc = MAX_ESC + 1;
            if ((nesc <= MAX_ESC && nexc <= NINL) || picked) break;
            picked = true;
            const u32 old = b3;
            pick_bases(r, L.hist, lane, b8, b3);
            if (b3 == old) break;
        }
        if (nesc <= MAX_ESC && nexc <= NINL) {
            u32* const strm = (u32*)(L.stage + 512);
            if (lane < 32) *(u32x4*)(L.stage + 512 + lane * 16) = u32x4{0, 0, 0, 0};
            const u32 excl = incl - cnt;
            u32 sel[16], pc[16];
#pragma unroll
            for (int i = 0; i < 16; ++i) {
                const u32 m = (em[i >> 3] >> (4 * (i & 7))) & 15u;
                sel[i] = L.lut[m];
                pc[i] = __builtin_popcount(m);
            }
            const u32 first = excl & 0xFFFFu;
            const u32 bit0 = 3 * first;
            u32 wi = lds_addr((char*)strm) + ((bit0 >> 5) << 2);
            u32 nb = bit0 & 31;
            u64 accb = 0;
            asm volatile("" ::: "memory");
#pragma unroll
            for (int i = 0; i < 16; ++i) {
                const u32 cmp = perm(0u, kb[i], sel[i]);
                const u32 g12 = __builtin_amdgcn_udot4(cmp, 0x00400801u, (cmp >> 24) << 9, false);
                accb |= (u64)g12 << nb;
                nb += shl<1>(pc[i]) + pc[i];
                if ((i & 1) && i < 15) {
                    const bool full = nb >= 32;
                    lds_or(wi, full ? (u32)accb : 0u);
                    accb = full ? accb >> 32 : accb;
                    nb &= 31;
                    wi += full ? 4 : 0;
                }
            }
            lds_or(wi, (u32)accb);
            lds_or(wi + 4, (u32)(accb >> 32));
            {
                u64 xm64 = ((u64)xm[1] << 32) | xm[0];
                const u64 em64 = ((u64)em[1] << 32) | em[0];
                u32 idx = 2 + (excl >> 16);
                while (xm64) {
                    const u32 j = __builtin_ctzll(xm64);
                    xm64 &= xm64 - 1;
                    const u32 ex = ((u32)(unsigned char)raw[(g * 16 + (j & 15)) * 64 + q * 4 + (j >> 4)] >> 2) & 31u;
                    const u32 k0 = ex < b3 ? (b3 - ex) & 7u : 0u;
                    const u32 k = 2 - ((ex ^ b3) & 1u);
                    const u32 bit = 3 * (first + __builtin_popcountll(em64 & ((1ull << j) - 1)));
                    const u64 fix = (u64)(k0 ^ k) << (bit & 31);
                    const u32 wa = lds_addr((char*)strm) + ((bit >> 5) << 2);
                    lds_xor(wa, (u32)fix);
                    lds_xor(wa + 4, (u32)(fix >> 32));
                    list[idx++] = (unsigned short)(posb + (j >> 4) * 16384 + (j & 15) * 16 + ((ex ^ ((b3 - k) & 31u)) >> 1));
                }
            }
            if (lane == 0) *(u32*)list = b3 | (MODE_W5 << 5) | (nexc << 7) | (nesc << 16);
            u32 cw[10];
#pragma unroll
            for (int h = 0; h < 2; ++h) {
                const u32* cd = code + 8 * h;
                cw[5 * h + 0] = cd[0] | ((cd[5] & 0x07070707u) << 4);
                cw[5 * h + 1] = cd[1] | ((cd[5] & 0x08080808u) << 1) | ((cd[5] & 0x80808080u) >> 2) | ((cd[6] & 0x80808080u) >> 1);
                cw[5 * h + 2] = cd[2] | ((cd[6] & 0x07070707u) << 4);
                cw[5 * h + 3] = cd[3] | ((cd[6] & 0x08080808u) << 1) | ((cd[7] & 0x08080808u) << 2) | ((cd[7] & 0x80808080u) >> 1);
                cw[5 * h + 4] = cd[4] | ((cd[7] & 0x07070707u) << 4);
            }
            *(u32x2*)(L.stage + lane * 8) = u32x2{cw[8], cw[9]};
            asm volatile("" ::: "memory");
            e.c2 = *(const u32x4*)(L.stage + lane * 16);
            e.hv = *(const u32x4*)((const char*)list + (lane & 7) * 16);
            e.c0 = u32x4{cw[0], cw[1], cw[2], cw[3]};
            e.c1 = u32x4{cw[4], cw[5], cw[6], cw[7]};
            e.c3 = e.c2;
            e.lh = (nexc + 2 + 7) >> 3;
            e.l2 = 32 + ((nesc * 3 + 127) >> 7);
            e.l3 = 0;
            asm volatile("" ::: "memory");
            return;
        }
    }

    u32 code[16], D[4], n, incl, total;
    while (true) {
        const u32 B4 = (b8 << 2) * 0x01010101u;
        const u32 NB4 = (((8 - b8) & 7) << 2) * 0x01010101u;
        n = 0;
#pragma unroll
        for (int k = 0; k < 4; ++k) {
            u32 v[4];
#pragma unroll
            for (int t = 0; t < 4; ++t) {
                const int i = 4 * k + t;
                const u32 d4 = ((T[i] & 0x1C1C1C1Cu) + NB4) & 0x1C1C1C1Cu;
                code[i] = (T[i] & 0x83838383u) | d4;
                const u32 z = T[i] ^ (code[i] + B4);
                v[t] = __builtin_amdgcn_udot4(z, 0x40100401u, 0u, false);
            }
            D[k] = (v[0] >> 5) | (v[1] << 3) | (v[2] << 11) | (v[3] << 19);
            n += __builtin_popcount((D[k] | (D[k] >> 1)) & 0x55555555u);
        }
        incl = wave_incl_scan(n);
        total = __builtin_amdgcn_readlane(incl, 63);
        if (total <= REPICK || picked) break;
        picked = true;
        const u32 old = b8;
        pick_bases(r, L.hist, lane, b8, b3);
        if (b8 == old) break;
    }
    if (total <= MAX_EXC) {
        u32 idx = 2 + incl - n;
#pragma unroll
        for (int k = 0; k < 4; ++k) {
            u32 dd = D[k];
            while (dd) {
                const u32 bit = __builtin_ctz(dd) & ~1u;
                list[idx++] = (unsigned short)(posb + k * 16384 + (bit << 3) + (((dd >> bit) & 3) << 2));
                dd &= ~(3u << bit);
            }
        }
        if (lane == 0) *(u32*)list = b8 | (MODE_W6 << 5) | (total << 7);
        u32 cw[12];
#pragma unroll
        for (int h = 0; h < 2; ++h) {
            const u32* cd = code + 8 * h;
            cw[6 * h + 0] = cd[0] | ((cd[6] & 0x03030303u) << 5);
            cw[6 * h + 1] = cd[1] | ((cd[6] & 0x0C0C0C0Cu) << 3);
            cw[6 * h + 2] = cd[2] | ((cd[6] & 0x10101010u) << 1) | ((cd[6] & 0x80808080u) >> 1);
            cw[6 * h + 3] = cd[3] | ((cd[7] & 0x03030303u) << 5);
            cw[6 * h + 4] = cd[4] | ((cd[7] & 0x0C0C0C0Cu) << 3);
            cw[6 * h + 5] = cd[5] | ((cd[7] & 0x10101010u) << 1) | ((cd[7] & 0x80808080u) >> 1);
        }
        asm volatile("" ::: "memory");
        e.hv = *(const u32x4*)((const char*)list + (lane & 7) * 16);
        e.c3 = *(const u32x4*)((const char*)list + 128 + (lane < 56 ? lane : 0) * 16);
        e.c0 = u32x4{cw[0], cw[1], cw[2], cw[3]};
        e.c1 = u32x4{cw[4], cw[5], cw[6], cw[7]};
        e.c2 = u32x4{cw[8], cw[9], cw[10], cw[11]};
        e.lh = (min(total, NINL) + 2 + 7) >> 3;
        e.l2 = 64;
        e.l3 = total > NINL ? (total - NINL + 7) >> 3 : 0;
        asm volatile("" ::: "memory");
    } else {
        e.hv = u32x4{(lane == 0) ? (MODE_RAW << 5) : 0u, 0u, 0u, 0u};
        e.c0 = u32x4{T[0], T[1], T[2], T[3]};
        e.c1 = u32x4{T[4], T[5], T[6], T[7]};
        e.c2 = u32x4{T[8], T[9], T[10], T[11]};
        e.c3 = u32x4{T[12], T[13], T[14], T[15]};
        e.lh = 1;
        e.l2 = 64;
        e.l3 = 64;
    }
}

template <int IMM>
__device__ __forceinline__ void st16_all(char* const (&dst)[NR], u32 off, u32x4 v, u32 lanes) {
    const u64 mask = lanes >= 64 ? ~0ull : (1ull << lanes) - 1;
    u64 save;
    asm volatile("s_mov_b64 %0, exec\n\ts_mov_b64 exec, %1\n\t"
                 "global_store_dwordx4 %2, %3, %4 offset:%12 sc0 sc1\n\t"
                 "global_store_dwordx4 %2, %3, %5 offset:%12 sc0 sc1\n\t"
                 "global_store_dwordx4 %2, %3, %6 offset:%12 sc0 sc1\n\t"
                 "global_store_dwordx4 %2, %3, %7 offset:%12 sc0 sc1\n\t"
                 "global_store_dwordx4 %2, %3, %8 offset:%12 sc0 sc1\n\t"
                 "global_store_dwordx4 %2, %3, %9 offset:%12 sc0 sc1\n\t"
                 "global_store_dwordx4 %2, %3, %10 offset:%12 sc0 sc1\n\t"
                 "global_store_dwordx4 %2, %3, %11 offset:%12 sc0 sc1\n\t"
                 "s_mov_b64 exec, %0\n\ts_nop 1"
                 : "=&s"(save)
                 : "s"(mask), "v"(off), "v"(v), "s"(dst[0]), "s"(dst[1]), "s"(dst[2]), "s"(dst[3]), "s"(dst[4]),
                   "s"(dst[5]), "s"(dst[6]), "s"(dst[7]), "i"(IMM)
                 : "memory");
}
__device__ __forceinline__ void store_unit(const Enc& e, char* const (&dst)[NR], u32 slotoff, int lane) {
    const u32 off = slotoff + 128 + lane * 16;
    st16_all<0>(dst, off, e.c0, 64);
    st16_all<1024>(dst, off, e.c1, 64);
    st16_all<2048>(dst, off, e.c2, e.l2);
    st16_all<0>(dst, slotoff + lane * 16, e.hv, e.lh);
    if (e.l3) st16_all<3072>(dst, off, e.c3, e.l3);
}

__device__ __forceinline__ void flag_store(char* a, u32 call) {
    __hip_atomic_store((gu32*)(u64)a, call, __ATOMIC_RELAXED, __HIP_MEMORY_SCOPE_SYSTEM);
}

constexpr int RSLOT = 5 * 1024 + 64;
struct Ring {
    char* slot;
    u32* head;
    u32* done;
    u32* ready;
};
__device__ __forceinline__ Ring ring_of(char* lds) {
    char* b = lds + NE * SND_LDS;
    return Ring{b + 64, (u32*)b, (u32*)(b + 4), (u32*)(b + 16)};
}
__device__ __forceinline__ u32 lds_load(const u32* p) {
    return __hip_atomic_load(p, __ATOMIC_RELAXED, __HIP_MEMORY_SCOPE_WORKGROUP);
}

__device__ __forceinline__ void encoder(const Params& p, char* lds, int e) {
    const int lane = threadIdx.x & 63;
    char* const myws = p.ws[p.rank];
    u32* const cnt = (u32*)(myws + CNT_OFF + (size_t)(blockIdx.x * 8 + e + 1) * 16);
    const SendLds L = send_lds(lds + e * SND_LDS);
    const Ring R = ring_of(lds);
    const u32 rawaddr = __builtin_amdgcn_readfirstlane(lds_addr(L.raw));
    if (lane < 16) {
        u32 sel = 0x0C0C0C0Cu, n = 0;
#pragma unroll
        for (int y = 0; y < 4; ++y)
            if ((lane >> y) & 1) { sel = (sel & ~(0xFFu << (8 * n))) | ((u32)y << (8 * n)); ++n; }
        ((u32*)L.lut)[lane] = sel;
    }
    const unsigned char* const wsrc = p.w + (size_t)(lane >> 2) * NN + (lane & 3) * 16;
    u32* const tick = (u32*)uni(myws + TICK_OFF);
    const u32 call = __builtin_amdgcn_readfirstlane(*cnt + 1);
    const u32 tbase = (call - 1) * (NU + NS * NE);
    u32 b8 = 32, b3 = 32, lastu = 0;
    while (true) {
        u32 pos = 0;
        if (lane == 0) pos = __hip_atomic_fetch_add(R.head, 1u, __ATOMIC_RELAXED, __HIP_MEMORY_SCOPE_WORKGROUP);
        pos = __builtin_amdgcn_readfirstlane(pos);
        const u32 depth = lastu + RINGL_FROM >= (u32)W5_UNITS ? RINGL : RING;
        while (pos >= __builtin_amdgcn_readfirstlane(lds_load(R.done)) + depth) __builtin_amdgcn_s_sleep(2);
        asm volatile("" ::: "memory");
        u32 tk;
        draw_ticket(tk, tick);
        wait_tk<0>(tk);
        const u32 u = __builtin_amdgcn_readfirstlane(tk) - tbase;
        char* const sl = R.slot + (pos % RING) * RSLOT;
        u32* const meta = (u32*)(sl + 5 * 1024);
        if (u >= (u32)NU) {
            if (lane == 0) {
                meta[0] = NU;
                __hip_atomic_store(&R.ready[pos % RING], pos + 1, __ATOMIC_RELEASE, __HIP_MEMORY_SCOPE_WORKGROUP);
                *cnt = call;
            }
            return;
        }
        lastu = u;
        dma_raw(wsrc + (size_t)(u / NC) * 64 * NN + (u % NC) * 64, rawaddr);
        wait_vm0();
        Enc en;
        encode(en, L.raw, L, lane, b8, b3, u < (u32)W5_UNITS);
        *(u32x4*)(sl + lane * 16) = en.c0;
        *(u32x4*)(sl + 1024 + lane * 16) = en.c1;
        *(u32x4*)(sl + 2048 + lane * 16) = en.c2;
        *(u32x4*)(sl + 3072 + lane * 16) = en.c3;
        *(u32x4*)(sl + 4096 + lane * 16) = en.hv;
        if (lane == 0) {
            *(u32x4*)meta = u32x4{u, en.lh, en.l2, en.l3};
            __hip_atomic_store(&R.ready[pos % RING], pos + 1, __ATOMIC_RELEASE, __HIP_MEMORY_SCOPE_WORKGROUP);
        }
    }
}

__device__ __forceinline__ void storer(const Params& p, char* lds) {
    const int lane = threadIdx.x & 63;
    const int rank = p.rank;
    const int wid = blockIdx.x;
    char* const myws = p.ws[rank];
    u32* const cnt = (u32*)(myws + CNT_OFF + (size_t)(blockIdx.x * 8) * 16);
    STAMP(blockIdx.x * 128 + 0);
    STAMP_HW(blockIdx.x * 128 + 15);
    const Ring R = ring_of(lds);
    const u32 call = __builtin_amdgcn_readfirstlane(*cnt + 1);
    const size_t par = DATA_OFF + (size_t)(call & 1) * PAR_BYTES;
    char* flagbase = p.ws[lane & (NR - 1)] + par + (size_t)rank * BLK + WF_OFF;
    char* dst[NR];
    const int rot = (wid + (wid >> 3)) & (NR - 1);
#pragma unroll
    for (int d = 0; d < NR; ++d) dst[d] = uni(p.ws[(rank + 1 + d + rot) & (NR - 1)] + par);
    asm volatile("" : "+v"(flagbase));

    int uprev = -1;
    int ends = 0;
    for (u32 pos = 0; ends < NE; ++pos) {
        while (__builtin_amdgcn_readfirstlane(lds_load(&R.ready[pos % RING])) != pos + 1) {
            if (uprev >= 0 && FLUSH_IDLE) {
                wait_vm0();
                if (lane < NR) flag_store(flagbase + (size_t)uprev * 128, call);
                uprev = -1;
            }
            __builtin_amdgcn_s_sleep(1);
        }
        asm volatile("" ::: "memory");
        const char* const sl = R.slot + (pos % RING) * RSLOT;
        const u32x4 meta = *(const u32x4*)(sl + 5 * 1024);
        const u32 u = __builtin_amdgcn_readfirstlane(meta.x);
        Enc e;
        if (u < (u32)NU) {
            e.c0 = *(const u32x4*)(sl + lane * 16);
            e.c1 = *(const u32x4*)(sl + 1024 + lane * 16);
            e.c2 = *(const u32x4*)(sl + 2048 + lane * 16);
            e.c3 = *(const u32x4*)(sl + 3072 + lane * 16);
            e.hv = *(const u32x4*)(sl + 4096 + lane * 16);
            e.lh = __builtin_amdgcn_readfirstlane(meta.y);
            e.l2 = __builtin_amdgcn_readfirstlane(meta.z);
            e.l3 = __builtin_amdgcn_readfirstlane(meta.w);
            asm volatile("" :: "v"(e.c0), "v"(e.c1), "v"(e.c2), "v"(e.c3), "v"(e.hv));
        }
        if (lane == 0) __hip_atomic_store(R.done, pos + 1, __ATOMIC_RELEASE, __HIP_MEMORY_SCOPE_WORKGROUP);
        if (u >= (u32)NU) { ++ends; continue; }
        store_unit(e, dst, slot_off(rank, u), lane);
        asm volatile("s_waitcnt vmcnt(32)" ::: "memory");
        if (uprev >= 0) {
            if (lane < NR) flag_store(flagbase + (size_t)uprev * 128, call);
        }
        uprev = u;
    }
    if (uprev >= 0) {
        wait_vm0();
        if (lane < NR) flag_store(flagbase + (size_t)uprev * 128, call);
    }
    if (lane == 0) *cnt = call;
}

__device__ __forceinline__ void decode_unit(char* img, __amdgpu_buffer_rsrc_t rs, u32 slot, int lane, u32x4 cw0,
                                            u32x4 cw1, u32x4 cw2, u32 ent, u32x2 t2, u32x2 vv, bool maybe5) {
    const u32 hdr = __builtin_amdgcn_readfirstlane(ent) & 0xFFFFu;
    const u32 mode = (hdr >> 5) & 3;
    const u32 total = hdr >> 7;
    const int q = lane & 15, g = lane >> 4;
    char* const wp = img + (g & 1) * 512 + (g >> 1) * 256 + q * 16;
    if (mode == MODE_RAW) {
        const u32x4 v3 = __builtin_amdgcn_raw_buffer_load_b128(rs, slot + 128 + 3072 + lane * 16, 0, AUX_SYS);
        *(u32x4*)(wp + 0) = cw0;
        *(u32x4*)(wp + 1024) = cw1;
        *(u32x4*)(wp + 2048) = cw2;
        *(u32x4*)(wp + 3072) = v3;
        return;
    }
    const u32 B4 = ((hdr & 31) << 2) * 0x01010101u;
    if (maybe5 && mode == MODE_W5) {
        ((u32*)img)[lane] = vv.x;
        ((u32*)img)[64 + lane] = vv.y;
        const u32 c[10] = {cw0.x, cw0.y, cw0.z, cw0.w, cw1.x, cw1.y, cw1.z, cw1.w, t2.x, t2.y};
        u32 o[16], n = 0;
#pragma unroll
        for (int h = 0; h < 2; ++h) {
            const u32* cc = c + 5 * h;
            u32* oo = o + 8 * h;
#pragma unroll
            for (int i = 0; i < 5; ++i) oo[i] = cc[i] & 0x8F8F8F8Fu;
            oo[5] = ((cc[0] >> 4) & 0x07070707u) | ((cc[1] >> 1) & 0x08080808u) | ((cc[1] << 2) & 0x80808080u);
            oo[6] = ((cc[2] >> 4) & 0x07070707u) | ((cc[3] >> 1) & 0x08080808u) | ((cc[1] << 1) & 0x80808080u);
            oo[7] = ((cc[4] >> 4) & 0x07070707u) | ((cc[3] >> 2) & 0x08080808u) | ((cc[3] << 1) & 0x80808080u);
        }
        u32 fm[16], m4[16], t3[16];
#pragma unroll
        for (int i = 0; i < 16; ++i) {
            m4[i] = o[i] & (o[i] >> 1) & 0x04040404u;
            fm[i] = shl<5>(m4[i]) - m4[i];
            t3[i] = __builtin_popcount(fm[i] & 0x1C1C1C1Cu);
            n += t3[i];
        }
        const u32 bit0 = wave_incl_scan(n) - n;
        u32 tlo, thi;
        w5_table(hdr & 31, tlo, thi);
        asm volatile("" ::: "memory");
        const u32* const sw = (const u32*)img + (bit0 >> 5);
        const u32 w0 = sw[0], w1 = sw[1], w2 = sw[2], w3 = sw[3];
        u32 v0 = __builtin_amdgcn_alignbit(w1, w0, bit0), v1 = __builtin_amdgcn_alignbit(w2, w1, bit0),
            v2 = __builtin_amdgcn_alignbit(w3, w2, bit0);
#pragma unroll
        for (int i = 0; i < 16; ++i) {
            const u32 t6 = (v0 & 0x3Fu) | (shl<10>(v0) & 0x003F0000u);
            const u32 kx = (t6 | shl<5>(t6)) & 0x07070707u;
            v0 = __builtin_amdgcn_alignbit(v1, v0, t3[i]);
            v1 = __builtin_amdgcn_alignbit(v2, v1, t3[i]);
            v2 >>= t3[i];
            u32 rk;
            asm("v_mul_u32_u24 %0, %1, %2" : "=v"(rk) : "v"(m4[i]), "s"(0x404040u));
            const u32 eb = perm(thi, tlo, perm(0u, kx, rk));
            o[i] = (fm[i] & eb) | (~fm[i] & (o[i] + B4));
        }
        asm volatile("" ::: "memory");
#pragma unroll
        for (int f = 0; f < 4; ++f) *(u32x4*)(wp + f * 1024) = u32x4{o[4 * f], o[4 * f + 1], o[4 * f + 2], o[4 * f + 3]};
    } else {
        const u32 c[12] = {cw0.x, cw0.y, cw0.z, cw0.w, cw1.x, cw1.y, cw1.z, cw1.w, cw2.x, cw2.y, cw2.z, cw2.w};
#pragma unroll
        for (int h = 0; h < 2; ++h) {
            const u32* cc = c + 6 * h;
            u32x4 a, bq;
            a.x = (cc[0] & 0x9F9F9F9Fu) + B4;
            a.y = (cc[1] & 0x9F9F9F9Fu) + B4;
            a.z = (cc[2] & 0x9F9F9F9Fu) + B4;
            a.w = (cc[3] & 0x9F9F9F9Fu) + B4;
            bq.x = (cc[4] & 0x9F9F9F9Fu) + B4;
            bq.y = (cc[5] & 0x9F9F9F9Fu) + B4;
            bq.z = (((cc[0] >> 5) & 0x03030303u) | ((cc[1] >> 3) & 0x0C0C0C0Cu) |
                    ((cc[2] >> 1) & 0x10101010u) | ((cc[2] << 1) & 0x80808080u)) + B4;
            bq.w = (((cc[3] >> 5) & 0x03030303u) | ((cc[4] >> 3) & 0x0C0C0C0Cu) |
                    ((cc[5] >> 1) & 0x10101010u) | ((cc[5] << 1) & 0x80808080u)) + B4;
            *(u32x4*)(wp + (2 * h) * 1024) = a;
            *(u32x4*)(wp + (2 * h + 1) * 1024) = bq;
        }
    }
    {
        const u32 e = (lane < 2 || (u32)lane >= total + 2) ? 0u : (ent & 0xFFFFu);
        const u32 pos = e >> 4;
        __hip_atomic_fetch_xor((u32*)(img + (pos & ~3u)), (e & 15) << (3 + 8 * (pos & 3)), __ATOMIC_RELAXED,
                               __HIP_MEMORY_SCOPE_WORKGROUP);
    }
#pragma clang loop unroll(disable)
    for (u32 i0 = NINL; i0 < total; i0 += 64) {
        const u32 i = i0 + lane;
        u32 e = __builtin_amdgcn_raw_buffer_load_b16(rs, slot + 128 + 3072 + (i - NINL) * 2, 0, AUX_SYS);
        e = (i < total) ? (e & 0xFFFFu) : 0u;
        const u32 pos = e >> 4;
        __hip_atomic_fetch_xor((u32*)(img + (pos & ~3u)), (e & 15) << (3 + 8 * (pos & 3)), __ATOMIC_RELAXED,
                               __HIP_MEMORY_SCOPE_WORKGROUP);
    }
}

__device__ __forceinline__ void send_x(const Params& p, u32 call, size_t par, int h0, int lane) {
    u32x4 v[XN][8];
    char* base[XN];
    char* xf[XN];
#pragma unroll
    for (int n = 0; n < XN; ++n) {
        const int dest = __builtin_amdgcn_readfirstlane((p.rank + 1 + (h0 + n) + ((h0 + n) >> 3)) & (NR - 1));
        const int rf = __builtin_amdgcn_readfirstlane(((h0 + n) >> 3) & 31);
        const unsigned char* src = p.x + (size_t)(dest * MS + rf * 16 + (lane & 15)) * KS
                                 + ((lane >> 4) & 1) * 32 + (lane >> 5) * 16;
#pragma unroll
        for (int j = 0; j < 8; ++j) v[n][j] = *(const u32x4*)(src + j * 64);
        base[n] = uni(p.ws[dest] + par + (size_t)p.rank * BLK + X_OFF + rf * 8192);
        xf[n] = uni(p.ws[dest] + par + (size_t)p.rank * BLK + XF_OFF + rf * 128);
    }
#pragma unroll
    for (int n = 0; n < XN; ++n)
#pragma unroll
        for (int j = 0; j < 8; ++j) asm volatile("" :: "v"(v[n][j]));
#pragma unroll
    for (int n = 0; n < XN; ++n)
#pragma unroll
        for (int j = 0; j < 8; j += 4) {
            st16_sys<0>(base[n] + j * 1024, lane * 16, v[n][j]);
            st16_sys<1024>(base[n] + j * 1024, lane * 16, v[n][j + 1]);
            st16_sys<2048>(base[n] + j * 1024, lane * 16, v[n][j + 2]);
            st16_sys<3072>(base[n] + j * 1024, lane * 16, v[n][j + 3]);
        }
    wait_vm0();
    if (lane == 0) {
#pragma unroll
        for (int n = 0; n < XN; ++n)
            __hip_atomic_store((gu32*)(u64)xf[n], call, __ATOMIC_RELAXED, __HIP_MEMORY_SCOPE_SYSTEM);
    }
}

template <int NF>
__device__ __forceinline__ void consumer_wave(const Params& p, char* lds, int c, int wave, int rf0, int lf0, u32 call, size_t par) {
    const int lane = threadIdx.x & 63;
    char* const myws = uni(p.ws[p.rank]);
    gu32* const tmo = (gu32*)(u64)(myws + TMO_OFF);
    const float scale = p.sx[0] * p.sw[0];
    const __amdgpu_buffer_rsrc_t rs =
        __builtin_amdgcn_make_buffer_rsrc(uni(myws + par), 0, (int)PAR_BYTES, 0x00020000);
    u32* const badw = (u32*)(lds + 65536);
    u32* const rdy = (u32*)(lds + 65536 + 32);
    const __amdgpu_buffer_rsrc_t ro = __builtin_amdgcn_make_buffer_rsrc(uni((char*)p.out), 0, MS * NN * 4, 0x00020000);

    f32x4 acc[NF][4];

    const u32 xoff = (u32)((lane >> 5) * BLK + X_OFF + rf0 * 8192 + (lane & 31) * 16);
    const int L16 = lane & 15, G = lane >> 4;
    const u32 bread = (G >> 1) * 4096 + (G & 1) * 256 + L16 * 16;

    u32x4 A[4 * NF * 2];
    auto load_a = [&](int P, int j) {
#pragma unroll
        for (int a = 0; a < NF; ++a)
#pragma unroll
            for (int i = 0; i < 2; ++i)
                A[(P * NF + a) * 2 + i] = __builtin_amdgcn_raw_buffer_load_b128(
                    rs, xoff + j * 1024 + i * 512, P * 2 * BLK + a * 8192, AUX_SYS);
    };
    char* const a7 = lds + A7_OFF + lf0 * 8192;
    auto dma_a7 = [&]() {
        const char* src = myws + par + xoff + (NJ - 1) * 1024;
        asm volatile("" : "+v"(src));
        const u32 dst = __builtin_amdgcn_readfirstlane(lds_addr(a7));
#pragma unroll
        for (int a = 0; a < NF; ++a)
#pragma unroll
            for (int P = 0; P < 4; ++P) {
                u32 keep;
                const char* s0 = src + a * 8192 + (size_t)P * 2 * BLK;
                asm volatile("s_mov_b32 %0, m0\n\ts_mov_b32 m0, %2\n\ts_nop 0\n\tglobal_load_lds_dwordx4 %1, off sc0 sc1\n\t"
                             "s_add_u32 m0, m0, 0x400\n\ts_nop 0\n\tglobal_load_lds_dwordx4 %3, off sc0 sc1\n\ts_mov_b32 m0, %0"
                             : "=&s"(keep) : "v"(s0), "s"(dst + a * 8192 + P * 2048), "v"(s0 + 512) : "memory", "scc");
            }
    };
    auto mfma = [&](int P, int f, const u32x4& b0, const u32x4& b1) {
        const i32x8 B = {(int)b0.x, (int)b0.y, (int)b0.z, (int)b0.w, (int)b1.x, (int)b1.y, (int)b1.z, (int)b1.w};
#pragma unroll
        for (int a = 0; a < NF; ++a) {
            const u32x4 a0 = A[(P * NF + a) * 2], a1 = A[(P * NF + a) * 2 + 1];
            const i32x8 Av = {(int)a0.x, (int)a0.y, (int)a0.z, (int)a0.w, (int)a1.x, (int)a1.y, (int)a1.z, (int)a1.w};
            acc[a][f] = __builtin_amdgcn_mfma_scale_f32_16x16x128_f8f6f4(Av, B, acc[a][f], 0, 1, 0, 0, 0, 0);
        }
    };
    u32x4 cw0, cw1, cw2;
    u32x2 t2 = {0, 0};
    u32x2 vv = {0, 0};
    u32 ent;
    auto unit_slot = [&](int j) { return slot_off(wave, j * NC + c); };
    auto issue = [&](int j, u32 oob) {
        const u32 slot = unit_slot(j) | oob;
        ent = __builtin_amdgcn_raw_buffer_load_b16(rs, slot + lane * 2, 0, AUX_SYS);
        cw0 = __builtin_amdgcn_raw_buffer_load_b128(rs, slot + 128 + lane * 16, 0, AUX_SYS);
        cw1 = __builtin_amdgcn_raw_buffer_load_b128(rs, slot + 128 + 1024 + lane * 16, 0, AUX_SYS);
        cw2 = __builtin_amdgcn_raw_buffer_load_b128(rs, slot + 128 + 2048 + lane * 16, 0, AUX_SYS);
        if (j < W5_ROUNDS) {
            t2 = __builtin_amdgcn_raw_buffer_load_b64(rs, slot + 128 + 2048 + lane * 8, 0, AUX_SYS);
            vv.x = __builtin_amdgcn_raw_buffer_load_b32(rs, slot + 128 + 2560 + lane * 4, 0, AUX_SYS);
            vv.y = __builtin_amdgcn_raw_buffer_load_b32(rs, slot + 128 + 2560 + 256 + lane * 4, 0, AUX_SYS);
        }
    };
    auto decode = [&](int j, u32 oob) {
        decode_unit(lds + (j & 1) * 32768 + wave * 4096, rs, unit_slot(j) | oob, lane, cw0, cw1, cw2, ent, t2, vv,
                    j < W5_ROUNDS);
    };
    auto wflag = [&](int j) { return (const gu32*)(u64)(myws + par + (size_t)wave * BLK + WF_OFF + (j * NC + c) * 128); };
    bool bad = false;

    bool have = false;
    auto round = [&](int j) {
        if (!have) {
            bad |= wait_flags(wflag(j), lane == 0, call, tmo);
            issue(j, 0);
        }
        decode(j, 0);
        __syncthreads();
        if (j == NJ - 3) {
            const u32 dummy = __builtin_amdgcn_raw_buffer_load_b32(ro, (u32)(((rf0 * 16 + G * 4) * NN + c * 64 + L16 * 4) * 4), 0, 16);
            asm volatile("" :: "v"(dummy));
        }
        u32 pv = __hip_atomic_load(wflag(j + 1), __ATOMIC_RELAXED, __HIP_MEMORY_SCOPE_SYSTEM);
        const char* const buf = lds + (j & 1) * 32768;
        u32x4 Bf[2][2][2];
#pragma unroll
        for (int f = 0; f < 2; ++f) {
            Bf[0][f][0] = *(const u32x4*)(buf + f * 1024 + bread);
            Bf[0][f][1] = *(const u32x4*)(buf + f * 1024 + bread + 512);
        }
#pragma unroll
        for (int st = 0; st < 8; ++st) {
            if (st < 7) {
                const int Pn = (st + 1) >> 1, fn = ((st + 1) & 1) * 2;
#pragma unroll
                for (int f = 0; f < 2; ++f) {
                    Bf[(st + 1) & 1][f][0] = *(const u32x4*)(buf + Pn * 8192 + (fn + f) * 1024 + bread);
                    Bf[(st + 1) & 1][f][1] = *(const u32x4*)(buf + Pn * 8192 + (fn + f) * 1024 + bread + 512);
                }
            }
            __builtin_amdgcn_sched_barrier(0);
#pragma unroll
            for (int f = 0; f < 2; ++f) mfma(st >> 1, (st & 1) * 2 + f, Bf[st & 1][f][0], Bf[st & 1][f][1]);
            if ((st & 1) && j + 2 < NJ) load_a(st >> 1, j + 1);
            if (st == EARLY_ST) {
                have = __builtin_amdgcn_readfirstlane(pv) >= call;
                asm volatile("" ::: "memory");
                if (have) issue(j + 1, 0);
                else pv = __hip_atomic_load(wflag(j + 1), __ATOMIC_RELAXED, __HIP_MEMORY_SCOPE_SYSTEM);
            }
            if (st == EARLY_ST2 && !have) {
                have = __builtin_amdgcn_readfirstlane(pv) >= call;
                asm volatile("" ::: "memory");
                if (have) issue(j + 1, 0);
            }
            __builtin_amdgcn_sched_barrier(0);
        }
        STAMP(blockIdx.x * 128 + wave * 16 + 2 + j);
    };

#pragma clang loop unroll(disable)
    for (u32 pass = 0; pass < 2; ++pass) {
        asm volatile("" : "+s"(pass));
        const u32 oob = pass ? 0u : 0x80000000u;
#pragma unroll
        for (int a = 0; a < NF; ++a)
#pragma unroll
            for (int f = 0; f < 4; ++f) acc[a][f] = f32x4{0.f, 0.f, 0.f, 0.f};
        if (pass) {
            bad = wait_flags((const gu32*)(u64)(myws + par + (size_t)(lane & 7) * BLK + XF_OFF + (rf0 + ((lane >> 3) & (NF - 1))) * 128),
                             lane < NF * NR, call, tmo);
            STAMP(blockIdx.x * 128 + wave * 16 + 1);
            dma_a7();
#pragma unroll
            for (int P = 0; P < 4; ++P) load_a(P, 0);
#pragma clang loop unroll(disable)
            for (int j = 0; j < NJ - 1; ++j) round(j);
            asm volatile("s_waitcnt vmcnt(0)" ::: "memory");
        }

        {
            constexpr int j = NJ - 1;
            const char* const bbuf = lds + (j & 1) * 32768 + bread;
            const char* const abuf = a7 + lane * 16;
            const gu32* const fl = wflag(j);
            const u64 t0 = __builtin_amdgcn_s_memrealtime();
            asm volatile("" :: "s"(t0));
            bool own = false;
            u32 todo = 15, avail = 0, it = 0;
            auto pick = [&]() {
                if (!pass) { own = false; avail = 0; todo = 15; }
                while (!avail) {
                    u32 v = call;
                    if (!own && pass && !have) v = __hip_atomic_load(fl, __ATOMIC_RELAXED, __HIP_MEMORY_SCOPE_SYSTEM);
                    const u32x4 r0 = *(const u32x4*)rdy, r1 = *(const u32x4*)(rdy + 4);
                    const u32 m = (min(r0.x, r0.y) > pass ? 1u : 0u) | (min(r0.z, r0.w) > pass ? 2u : 0u) |
                                  (min(r1.x, r1.y) > pass ? 4u : 0u) | (min(r1.z, r1.w) > pass ? 8u : 0u);
                    avail = __builtin_amdgcn_readfirstlane(m & todo);
                    todo &= ~avail;
                    if (!own) {
                        bool go = __builtin_amdgcn_readfirstlane(v) >= call;
                        if (!go && (++it & 15) == 15 &&
                            (__hip_atomic_load(tmo, __ATOMIC_RELAXED, __HIP_MEMORY_SCOPE_AGENT) == call ||
                             __builtin_amdgcn_s_memrealtime() - t0 > DEADLINE)) {
                            __hip_atomic_store(tmo, call, __ATOMIC_RELAXED, __HIP_MEMORY_SCOPE_AGENT);
                            bad = true;
                            go = true;
                        }
                        if (go) {
                            asm volatile("" ::: "memory");
                            STAMP(blockIdx.x * 128 + wave * 16 + 11);
                            if (!have) issue(j, oob);
                            decode(j, oob);
                            if (lane == 0) {
                                badw[wave] = bad;
                                __hip_atomic_store(&rdy[wave], pass + 1, __ATOMIC_RELEASE, __HIP_MEMORY_SCOPE_WORKGROUP);
                            }
                            STAMP(blockIdx.x * 128 + wave * 16 + 13);
                            own = true;
                        }
                    } else if (!avail) {
                        __builtin_amdgcn_s_sleep(1);
                    }
                }
                asm volatile("" ::: "memory");
                const int P = __builtin_ctz(avail);
                avail &= avail - 1;
                return P;
            };
            struct Frags { u32x4 a[NF][2], b[4][2]; };
            auto rd_a = [&](Frags& F, int P, int a) {
                F.a[a][0] = *(const u32x4*)(abuf + a * 8192 + P * 2048);
                F.a[a][1] = *(const u32x4*)(abuf + a * 8192 + P * 2048 + 1024);
            };
            auto rd_b = [&](Frags& F, int P, int f) {
                F.b[f][0] = *(const u32x4*)(bbuf + P * 8192 + f * 1024);
                F.b[f][1] = *(const u32x4*)(bbuf + P * 8192 + f * 1024 + 512);
            };
            auto mm_f = [&](const Frags& F, int f) {
                const i32x8 B = {(int)F.b[f][0].x, (int)F.b[f][0].y, (int)F.b[f][0].z, (int)F.b[f][0].w,
                                 (int)F.b[f][1].x, (int)F.b[f][1].y, (int)F.b[f][1].z, (int)F.b[f][1].w};
#pragma unroll
                for (int a = 0; a < NF; ++a) {
                    const i32x8 Av = {(int)F.a[a][0].x, (int)F.a[a][0].y, (int)F.a[a][0].z, (int)F.a[a][0].w,
                                      (int)F.a[a][1].x, (int)F.a[a][1].y, (int)F.a[a][1].z, (int)F.a[a][1].w};
                    acc[a][f] = __builtin_amdgcn_mfma_scale_f32_16x16x128_f8f6f4(Av, B, acc[a][f], 0, 1, 0, 0, 0, 0);
                }
            };
            auto mmrd = [&](const Frags& Fc, Frags& Fn, int P) {
#pragma unroll
                for (int f = 0; f < 4; ++f) {
                    mm_f(Fc, f);
                    if (f < NF) rd_a(Fn, P, f);
                    rd_b(Fn, P, f);
                    __builtin_amdgcn_sched_barrier(0);
                }
            };
            Frags F0, F1;
            {
                const int P = pick();
#pragma unroll
                for (int a = 0; a < NF; ++a) rd_a(F0, P, a);
#pragma unroll
                for (int f = 0; f < 4; ++f) rd_b(F0, P, f);
            }
            mmrd(F0, F1, pick());
            mmrd(F1, F0, pick());
            mmrd(F0, F1, pick());
#pragma unroll
            for (int f = 0; f < 4; ++f) mm_f(F1, f);
        }
        STAMP(blockIdx.x * 128 + wave * 16 + 9);

        const u32x4 b0 = *(const u32x4*)badw, b1 = *(const u32x4*)(badw + 4);
        const float s = (b0.x | b0.y | b0.z | b0.w | b1.x | b1.y | b1.z | b1.w) ? __builtin_nanf("") : scale;
        const u32 obase = (u32)(((rf0 * 16 + G * 4) * NN + c * 64 + L16 * 4) * 4) | oob;
#pragma unroll
        for (int a = 0; a < NF; ++a)
#pragma unroll
            for (int r = 0; r < 4; ++r) {
                const f32x4 v = {acc[a][0][r] * s, acc[a][1][r] * s, acc[a][2][r] * s, acc[a][3][r] * s};
                __builtin_amdgcn_raw_buffer_store_b128(__builtin_bit_cast(u32x4, v), ro, obase + (a * 16 + r) * NN * 4, 0, OUT_AUX);
            }
        STAMP(blockIdx.x * 128 + wave * 16 + 10);
    }
}

__device__ __forceinline__ void consumer(const Params& p, char* lds) {
    const int lane = threadIdx.x & 63;
    const int wave = __builtin_amdgcn_readfirstlane(threadIdx.x >> 6);
    const int t = blockIdx.x - NS;
    const int x = t & 7, i = t >> 3;
    int c, f0, nfr;
    if (i < 12) { c = 4 * x + i / 3; f0 = (i % 3) * 11; nfr = (i % 3 == 2) ? 10 : 11; }
    else        { c = 32 + 4 * x + ((i - 12) >> 2); f0 = ((i - 12) & 3) * 8; nfr = 8; }
    const int n2 = nfr - 8;
    const int lf0 = wave < n2 ? 2 * wave : n2 + wave;
    char* const myws = p.ws[p.rank];
    u32* const cnt = (u32*)(myws + CNT_OFF + (size_t)(blockIdx.x * 8 + wave) * 16);
    const u32 call = __builtin_amdgcn_readfirstlane(*cnt + 1);
    const size_t par = DATA_OFF + (size_t)(call & 1) * PAR_BYTES;
    STAMP(blockIdx.x * 128 + wave * 16 + 0);
    STAMP_HW(blockIdx.x * 128 + wave * 16 + 15);
    if (lane == 0) ((u32*)(lds + 65536 + 32))[wave] = 0;
    if (wave == (t & 7)) send_x(p, call, par, blockIdx.x, lane);
    if (wave < n2) consumer_wave<2>(p, lds, c, wave, f0 + lf0, lf0, call, par);
    else consumer_wave<1>(p, lds, c, wave, f0 + lf0, lf0, call, par);
    if (lane == 0) *cnt = call;
}

__global__ void __launch_bounds__(512, 2) gemm_rs_kernel(Params p) {
    __shared__ __attribute__((aligned(16))) char lds[LDS_BYTES];
    if (blockIdx.x < NS) {
        const int wave = __builtin_amdgcn_readfirstlane(threadIdx.x >> 6);
        if (threadIdx.x < 16) ((u32*)(lds + NE * SND_LDS))[threadIdx.x] = 0;
        __syncthreads();
        if (wave == 0) {
            storer(p, lds);
        } else if (wave <= NE) {
            encoder(p, lds, wave - 1);
        } else if (wave == 7) {
            const int lane = threadIdx.x & 63;
            u32* const cnt = (u32*)(p.ws[p.rank] + CNT_OFF + (size_t)(blockIdx.x * 8 + wave) * 16);
            const u32 call = __builtin_amdgcn_readfirstlane(*cnt + 1);
            send_x(p, call, DATA_OFF + (size_t)(call & 1) * PAR_BYTES, blockIdx.x, lane);
            if (lane == 0) *cnt = call;
        }
    } else {
        consumer(p, lds);
    }
}

}

extern "C" void kernel_launch(int rank, int nranks, ncclComm_t comm, hipStream_t stream,
                              void* const* d_in, const int* in_sizes, int n_in,
                              void* d_out, int out_size, const int* in_dtypes, int out_dtype,
                              void* const* d_out_all, void* const* d_ws_all, size_t ws_bytes) {
    (void)comm; (void)in_dtypes; (void)out_dtype; (void)d_out_all;
    if (nranks != NR || n_in != 4 || rank < 0 || rank >= NR || ws_bytes < WS_NEED ||
        in_sizes[0] != 4096 * KS || in_sizes[1] != KS * NN || out_size != MS * NN) {
        fprintf(stderr, "gemm_rs: unexpected problem shape\n");
        return;
    }
    Params p;
    p.x = (const unsigned char*)d_in[0];
    p.w = (const unsigned char*)d_in[1];
    p.sx = (const float*)d_in[2];
    p.sw = (const float*)d_in[3];
    p.out = (float*)d_out;
    for (int r = 0; r < NR; ++r) p.ws[r] = (char*)d_ws_all[r];
    p.rank = rank;
    p.pad = 0;
    hipLaunchKernelGGL(gemm_rs_kernel, dim3(GRID), dim3(512), 0, stream, p);
    HIP_CHECK(hipGetLastError());
}
```

```cpp
#include <hip/hip_runtime.h>
#include <rccl/rccl.h>
#include <cstdio>
#include <cstdint>

#define HIP_CHECK(expr) do { hipError_t _e = (expr); if (_e != hipSuccess) { \
    fprintf(stderr, "HIP error %s at %s:%d\n", hipGetErrorString(_e), __FILE__, __LINE__); \
    return; } } while (0)

namespace {

typedef unsigned u32;
typedef unsigned long long u64;
typedef __attribute__((address_space(1))) unsigned gu32;
typedef unsigned u32x4 __attribute__((ext_vector_type(4)));
typedef unsigned u32x2 __attribute__((ext_vector_type(2)));
typedef unsigned u32x3 __attribute__((ext_vector_type(3)));
typedef int i32x8 __attribute__((ext_vector_type(8)));
typedef float f32x4 __attribute__((ext_vector_type(4)));

constexpr int NR = 8;
constexpr int MS = 512;
constexpr int KS = 512;
constexpr int NN = 4096;
constexpr int NJ = 8;
constexpr int NC = 64;
constexpr int NU = NJ * NC;

#define NSEND 32
#define NENC 5
#define RINGN 4
#define RINGL 3
#define RINGL_FROM 96
#define FLUSH_IDLE 1
constexpr int NS = NSEND;
constexpr int NE = NENC;
constexpr int RING = RINGN;
constexpr int NW = NS;
constexpr int NCONS = 224;
constexpr int GRID = NS + NCONS;
constexpr int XN = 1;
static_assert(GRID == 256 && NE <= 6, "one workgroup per CU, 256 x half-units");

constexpr size_t CNT_OFF   = 0;
constexpr size_t TMO_OFF   = 64 << 10;
constexpr size_t TICK_OFF  = 96 << 10;
constexpr size_t DATA_OFF  = 1 << 20;
constexpr u32    SLOT      = 4224;
constexpr u32    WF_OFF    = 0;
constexpr u32    XF_OFF    = 64 << 10;
constexpr u32    X_OFF     = 128 << 10;
constexpr u32    W_OFF     = 384 << 10;
constexpr u32    BLK       = 2560 << 10;
constexpr size_t PAR_BYTES = (size_t)NR * BLK;
constexpr size_t WS_NEED   = DATA_OFF + 2 * PAR_BYTES;
static_assert(W_OFF + NU * SLOT <= BLK, "block too small");
static_assert((size_t)GRID * 8 * 16 <= TMO_OFF, "counter region too small");

constexpr u64 DEADLINE = 40000000ull;
constexpr int AUX_SYS = 17;
#define PRETOUCH 1
#define W5_ROUNDS 4
#define W5_FROM 1
constexpr int W5_UNITS = W5_ROUNDS * NC;
__device__ __forceinline__ bool w5_round(int j) { return j >= W5_FROM && j < W5_ROUNDS; }
#define EARLY_ST 2
#define EARLY_ST2 7
#define OUT_AUX 16

struct Params {
    const unsigned char* x;
    const unsigned char* w;
    const float* sx;
    const float* sw;
    float* out;
    char* ws[NR];
    int rank;
    int pad;
};

__device__ __forceinline__ u32 slot_off(int src, int u) { return (u32)src * BLK + W_OFF + (u32)u * SLOT; }
template <int N>
__device__ __forceinline__ u32 shl(u32 x) {
    u32 r;
    asm("v_lshlrev_b32 %0, %1, %2" : "=v"(r) : "i"(N), "v"(x));
    return r;
}
__device__ __forceinline__ u32 perm(u32 hi, u32 lo, u32 sel) { return __builtin_amdgcn_perm(hi, lo, sel); }

template <int IMM>
__device__ __forceinline__ void st16_sys(char* base, u32 off, u32x4 v) {
    asm volatile("global_store_dwordx4 %0, %1, %2 offset:%3 sc0 sc1\n\ts_nop 1"
                 :: "v"(off), "v"(v), "s"(base), "i"(IMM) : "memory");
}
__device__ __forceinline__ char* uni(char* p) {
    const u64 v = (u64)p;
    const u32 lo = __builtin_amdgcn_readfirstlane((u32)v);
    const u32 hi = __builtin_amdgcn_readfirstlane((u32)(v >> 32));
    char* r = (char*)(((u64)hi << 32) | lo);
    asm volatile("s_nop 4" : "+s"(r));
    return r;
}
__device__ __forceinline__ void wait_vm0() { asm volatile("s_waitcnt vmcnt(0)" ::: "memory"); }

#define STAMP(slot) do {} while (0)
#define STAMP_HW(slot) do {} while (0)

__device__ __forceinline__ bool wait_flags(const gu32* flag, bool active, u32 call, gu32* tmo) {
    const u64 t0 = __builtin_amdgcn_s_memrealtime();
    bool bad = false;
    for (u32 it = 0;; ++it) {
        u32 v = call;
        if (active) v = __hip_atomic_load(flag, __ATOMIC_RELAXED, __HIP_MEMORY_SCOPE_SYSTEM);
        if (__all(v >= call)) break;
        if ((it & 15) == 15) {
            if (__hip_atomic_load(tmo, __ATOMIC_RELAXED, __HIP_MEMORY_SCOPE_AGENT) == call ||
                __builtin_amdgcn_s_memrealtime() - t0 > DEADLINE) {
                __hip_atomic_store(tmo, call, __ATOMIC_RELAXED, __HIP_MEMORY_SCOPE_AGENT);
                bad = true;
                break;
            }
        }
        __builtin_amdgcn_s_sleep(2);
    }
    asm volatile("" ::: "memory");
    return bad;
}

template <int CTRL, int RMASK>
__device__ __forceinline__ u32 dpp0(u32 v) {
    return __builtin_amdgcn_update_dpp(0u, v, CTRL, RMASK, 0xf, false);
}
__device__ __forceinline__ u32 wave_incl_scan(u32 v) {
    v += dpp0<0x111, 0xf>(v);
    v += dpp0<0x112, 0xf>(v);
    v += dpp0<0x114, 0xf>(v);
    v += dpp0<0x118, 0xf>(v);
    v += dpp0<0x142, 0xa>(v);
    v += dpp0<0x143, 0xc>(v);
    return v;
}
__device__ __forceinline__ u32 wave_max(u32 v) {
    v = max(v, dpp0<0x111, 0xf>(v));
    v = max(v, dpp0<0x112, 0xf>(v));
    v = max(v, dpp0<0x114, 0xf>(v));
    v = max(v, dpp0<0x118, 0xf>(v));
    v = max(v, dpp0<0x142, 0xa>(v));
    v = max(v, dpp0<0x143, 0xc>(v));
    return __builtin_amdgcn_readlane(v, 63);
}

constexpr int A7_OFF = 65536 + 256;
constexpr int LDS_BYTES = A7_OFF + 11 * 8192;

constexpr u32 MODE_W6 = 0, MODE_RAW = 1, MODE_W5 = 2;
constexpr u32 NINL = 62;
constexpr u32 MAX_EXC = 510;
constexpr u32 REPICK = 127;
constexpr u32 MAX_ESC = 1344;

struct Enc {
    u32x4 c0, c1, c2, c3;
    u32x4 hv;
    u32 lh, l2, l3;
};

typedef __attribute__((address_space(3))) char lchar;
__device__ __forceinline__ u32 lds_addr(char* p) { return (u32)(u64)(lchar*)p; }
__device__ __forceinline__ void lds_xor(u32 addr, u32 v) {
    __hip_atomic_fetch_xor((__attribute__((address_space(3))) u32*)(u64)addr, v, __ATOMIC_RELAXED, __HIP_MEMORY_SCOPE_WORKGROUP);
}
__device__ __forceinline__ void lds_or(u32 addr, u32 v) {
    __hip_atomic_fetch_or((__attribute__((address_space(3))) u32*)(u64)addr, v, __ATOMIC_RELAXED, __HIP_MEMORY_SCOPE_WORKGROUP);
}

__device__ __forceinline__ void dma_raw(const unsigned char* src, u32 ldsdst) {
    u32 keep;
    const unsigned char* s1 = src + 16 * NN;
    const unsigned char* s2 = src + 32 * NN;
    const unsigned char* s3 = src + 48 * NN;
    asm volatile("s_mov_b32 %0, m0\n\t"
                 "s_mov_b32 m0, %5\n\ts_nop 0\n\tglobal_load_lds_dwordx4 %1, off\n\t"
                 "s_add_u32 m0, m0, 0x400\n\ts_nop 0\n\tglobal_load_lds_dwordx4 %2, off\n\t"
                 "s_add_u32 m0, m0, 0x400\n\ts_nop 0\n\tglobal_load_lds_dwordx4 %3, off\n\t"
                 "s_add_u32 m0, m0, 0x400\n\ts_nop 0\n\tglobal_load_lds_dwordx4 %4, off\n\t"
                 "s_mov_b32 m0, %0"
                 : "=&s"(keep) : "v"(src), "v"(s1), "v"(s2), "v"(s3), "s"(ldsdst) : "memory", "scc");
}
__device__ __forceinline__ void draw_ticket(u32& t, u32* ctr) {
    u64 save;
    asm volatile("s_mov_b64 %1, exec\n\ts_mov_b64 exec, 1\n\tglobal_atomic_add %0, %2, %3, %4 sc0\n\ts_mov_b64 exec, %1"
                 : "=&v"(t), "=&s"(save) : "v"(0u), "v"(1u), "s"(ctr) : "memory");
}
template <int N>
__device__ __forceinline__ void wait_tk(u32& t) {
    asm volatile("s_waitcnt vmcnt(%1)" : "+v"(t) : "n"(N) : "memory");
}

__device__ __forceinline__ void pick_bases(const u32 (&r)[16], u32* hist, int lane, u32& b8, u32& b3) {
    if (lane < 40) hist[lane] = 0;
    asm volatile("" ::: "memory");
    __hip_atomic_fetch_add(&hist[(r[0] >> 2) & 31], 1u, __ATOMIC_RELAXED, __HIP_MEMORY_SCOPE_WORKGROUP);
    __hip_atomic_fetch_add(&hist[(r[5] >> 10) & 31], 1u, __ATOMIC_RELAXED, __HIP_MEMORY_SCOPE_WORKGROUP);
    __hip_atomic_fetch_add(&hist[(r[10] >> 18) & 31], 1u, __ATOMIC_RELAXED, __HIP_MEMORY_SCOPE_WORKGROUP);
    __hip_atomic_fetch_add(&hist[(r[15] >> 26) & 31], 1u, __ATOMIC_RELAXED, __HIP_MEMORY_SCOPE_WORKGROUP);
    asm volatile("" ::: "memory");
    u32 h[8];
    const int hl = lane & 31;
#pragma unroll
    for (int i = 0; i < 8; ++i) h[i] = hist[hl + i];
    const u32 s3 = h[0] + h[1] + h[2];
    const u32 s8 = s3 + h[3] + h[4] + h[5] + h[6] + h[7];
    b8 = 31 - (wave_max((lane <= 24) ? (s8 * 32 + (31 - lane)) : 0u) & 31);
    b3 = 31 - (wave_max((lane <= 28) ? (s3 * 32 + (31 - lane)) : 0u) & 31);
}

__device__ __forceinline__ void w5_table(u32 b, u32& lo, u32& hi) {
    const u32 B4 = (b << 2) * 0x01010101u;
    lo = (((B4 | 0x80808080u) - 0x0C080400u) & 0x7C7C7C00u) | ((b + 3) << 2);
    hi = ((B4 | 0x80808080u) - 0x1C181410u) & 0x7C7C7C7Cu;
}

struct SendLds {
    char* raw;
    unsigned short* list;
    u32* hist;
    char* stage;
    const u32* lut;
};
__device__ __forceinline__ SendLds send_lds(char* base) {
    return SendLds{base, (unsigned short*)(base + 8192), (u32*)(base + 9344), base + 9536, (const u32*)(base + 10560)};
}
constexpr int SND_LDS = 10752;

__device__ __forceinline__ void encode(Enc& e, const char* raw, const SendLds& L, int lane, u32& b8, u32& b3, bool allow5) {
    const int q = lane & 15, g = lane >> 4;
    unsigned short* const list = L.list;
    u32 r[16];
#pragma unroll
    for (int i = 0; i < 16; ++i) r[i] = *(const u32*)(raw + (g * 16 + i) * 64 + q * 4);
    u32 T[16];
#pragma unroll
    for (int t = 0; t < 4; ++t) {
        const u32 t0 = perm(r[4 * t + 1], r[4 * t + 0], 0x05010400u);
        const u32 t1 = perm(r[4 * t + 1], r[4 * t + 0], 0x07030602u);
        const u32 t2 = perm(r[4 * t + 3], r[4 * t + 2], 0x05010400u);
        const u32 t3 = perm(r[4 * t + 3], r[4 * t + 2], 0x07030602u);
        T[0 + t]  = perm(t2, t0, 0x05040100u);
        T[4 + t]  = perm(t2, t0, 0x07060302u);
        T[8 + t]  = perm(t3, t1, 0x05040100u);
        T[12 + t] = perm(t3, t1, 0x07060302u);
    }
    bool picked = false;
    if (b8 >= 32) { pick_bases(r, L.hist, lane, b8, b3); picked = true; }
    const u32 posb = ((g & 1) * 512 + (g >> 1) * 256 + q * 16) << 4;

    if (allow5) {
        u32 code[16], kb[16], em[2], xm[2], cnt, incl, nesc, nexc;
        while (true) {
            const u32 B4 = (b3 << 2) * 0x01010101u;
            em[0] = em[1] = xm[0] = xm[1] = 0;
#pragma unroll
            for (int i = 0; i < 16; ++i) {
                const u32 t = T[i];
                const u32 u = ((t & 0x7C7C7C7Cu) | 0x80808080u) - B4;
                const u32 bl = (~u >> 7) & 0x01010101u;
                const u32 v = u & 0x7C7C7C7Cu;
                const u32 esc = (((v + 0x74747474u) >> 7) & 0x01010101u) | bl;
                code[i] = (t & 0x83838383u) | (v & 0x0C0C0C0Cu) | shl<3>(esc) | shl<2>(esc);
                const u32 k4 = (0x80808080u - v) & 0x7C7C7C7Cu;
                const u32 big = shl<1>(k4) | (k4 << 2), ge4 = v + 0x70707070u;
                const u32 m8 = bl << 7;
                const u32 exc = (((m8 & big) | (~m8 & ge4)) >> 7) & 0x01010101u;
                kb[i] = (k4 >> 2) & (shl<3>(bl) - bl);
                em[i >> 3] |= __builtin_amdgcn_udot4(esc, 0x08040201u, 0u, false) << (4 * (i & 7));
                xm[i >> 3] |= __builtin_amdgcn_udot4(exc, 0x08040201u, 0u, false) << (4 * (i & 7));
            }
            const u32 ne = __builtin_popcount(em[0]) + __builtin_popcount(em[1]);
            cnt = ne | ((__builtin_popcount(xm[0]) + __builtin_popcount(xm[1])) << 16);
            incl = wave_incl_scan(cnt);
            const u32 tot = __builtin_amdgcn_readlane(incl, 63);
            nesc = tot & 0xFFFFu;
            nexc = tot >> 16;
            if (__builtin_amdgcn_ballot_w64(ne > 32)) nesc = MAX_ESC + 1;
            if ((nesc <= MAX_ESC && nexc <= NINL) || picked) break;
            picked = true;
            const u32 old = b3;
            pick_bases(r, L.hist, lane, b8, b3);
            if (b3 == old) break;
        }
        if (nesc <= MAX_ESC && nexc <= NINL) {
            u32* const strm = (u32*)(L.stage + 512);
            if (lane < 32) *(u32x4*)(L.stage + 512 + lane * 16) = u32x4{0, 0, 0, 0};
            const u32 excl = incl - cnt;
            u32 sel[16], pc[16];
#pragma unroll
            for (int i = 0; i < 16; ++i) {
                const u32 m = (em[i >> 3] >> (4 * (i & 7))) & 15u;
                sel[i] = L.lut[m];
                pc[i] = __builtin_popcount(m);
            }
            const u32 first = excl & 0xFFFFu;
            const u32 bit0 = 3 * first;
            u32 wi = lds_addr((char*)strm) + ((bit0 >> 5) << 2);
            u32 nb = bit0 & 31;
            u64 accb = 0;
            asm volatile("" ::: "memory");
#pragma unroll
            for (int i = 0; i < 16; ++i) {
                const u32 cmp = perm(0u, kb[i], sel[i]);
                const u32 g12 = __builtin_amdgcn_udot4(cmp, 0x00400801u, (cmp >> 24) << 9, false);
                accb |= (u64)g12 << nb;
                nb += shl<1>(pc[i]) + pc[i];
                if ((i & 1) && i < 15) {
                    const bool full = nb >= 32;
                    lds_or(wi, full ? (u32)accb : 0u);
                    accb = full ? accb >> 32 : accb;
                    nb &= 31;
                    wi += full ? 4 : 0;
                }
            }
            lds_or(wi, (u32)accb);
            lds_or(wi + 4, (u32)(accb >> 32));
            {
                u64 xm64 = ((u64)xm[1] << 32) | xm[0];
                const u64 em64 = ((u64)em[1] << 32) | em[0];
                u32 idx = 2 + (excl >> 16);
                while (xm64) {
                    const u32 j = __builtin_ctzll(xm64);
                    xm64 &= xm64 - 1;
                    const u32 ex = ((u32)(unsigned char)raw[(g * 16 + (j & 15)) * 64 + q * 4 + (j >> 4)] >> 2) & 31u;
                    const u32 k0 = ex < b3 ? (b3 - ex) & 7u : 0u;
                    const u32 k = 2 - ((ex ^ b3) & 1u);
                    const u32 bit = 3 * (first + __builtin_popcountll(em64 & ((1ull << j) - 1)));
                    const u64 fix = (u64)(k0 ^ k) << (bit & 31);
                    const u32 wa = lds_addr((char*)strm) + ((bit >> 5) << 2);
                    lds_xor(wa, (u32)fix);
                    lds_xor(wa + 4, (u32)(fix >> 32));
                    list[idx++] = (unsigned short)(posb + (j >> 4) * 16384 + (j & 15) * 16 + ((ex ^ ((b3 - k) & 31u)) >> 1));
                }
            }
            if (lane == 0) *(u32*)list = b3 | (MODE_W5 << 5) | (nexc << 7) | (nesc << 16);
            u32 cw[10];
#pragma unroll
            for (int h = 0; h < 2; ++h) {
                const u32* cd = code + 8 * h;
                cw[5 * h + 0] = cd[0] | ((cd[5] & 0x07070707u) << 4);
                cw[5 * h + 1] = cd[1] | ((cd[5] & 0x08080808u) << 1) | ((cd[5] & 0x80808080u) >> 2) | ((cd[6] & 0x80808080u) >> 1);
                cw[5 * h + 2] = cd[2] | ((cd[6] & 0x07070707u) << 4);
                cw[5 * h + 3] = cd[3] | ((cd[6] & 0x08080808u) << 1) | ((cd[7] & 0x08080808u) << 2) | ((cd[7] & 0x80808080u) >> 1);
                cw[5 * h + 4] = cd[4] | ((cd[7] & 0x07070707u) << 4);
            }
            *(u32x2*)(L.stage + lane * 8) = u32x2{cw[8], cw[9]};
            asm volatile("" ::: "memory");
            e.c2 = *(const u32x4*)(L.stage + lane * 16);
            e.hv = *(const u32x4*)((const char*)list + (lane & 7) * 16);
            e.c0 = u32x4{cw[0], cw[1], cw[2], cw[3]};
            e.c1 = u32x4{cw[4], cw[5], cw[6], cw[7]};
            e.c3 = e.c2;
            e.lh = (nexc + 2 + 7) >> 3;
            e.l2 = 32 + ((nesc * 3 + 127) >> 7);
            e.l3 = 0;
            asm volatile("" ::: "memory");
            return;
        }
    }

    u32 code[16], D[4], n, incl, total;
    while (true) {
        const u32 B4 = (b8 << 2) * 0x01010101u;
        const u32 NB4 = (((8 - b8) & 7) << 2) * 0x01010101u;
        n = 0;
#pragma unroll
        for (int k = 0; k < 4; ++k) {
            u32 v[4];
#pragma unroll
            for (int t = 0; t < 4; ++t) {
                const int i = 4 * k + t;
                const u32 d4 = ((T[i] & 0x1C1C1C1Cu) + NB4) & 0x1C1C1C1Cu;
                code[i] = (T[i] & 0x83838383u) | d4;
                const u32 z = T[i] ^ (code[i] + B4);
                v[t] = __builtin_amdgcn_udot4(z, 0x40100401u, 0u, false);
            }
            D[k] = (v[0] >> 5) | (v[1] << 3) | (v[2] << 11) | (v[3] << 19);
            n += __builtin_popcount((D[k] | (D[k] >> 1)) & 0x55555555u);
        }
        incl = wave_incl_scan(n);
        total = __builtin_amdgcn_readlane(incl, 63);
        if (total <= REPICK || picked) break;
        picked = true;
        const u32 old = b8;
        pick_bases(r, L.hist, lane, b8, b3);
        if (b8 == old) break;
    }
    if (total <= MAX_EXC) {
        u32 idx = 2 + incl - n;
#pragma unroll
        for (int k = 0; k < 4; ++k) {
            u32 dd = D[k];
            while (dd) {
                const u32 bit = __builtin_ctz(dd) & ~1u;
                list[idx++] = (unsigned short)(posb + k * 16384 + (bit << 3) + (((dd >> bit) & 3) << 2));
                dd &= ~(3u << bit);
            }
        }
        if (lane == 0) *(u32*)list = b8 | (MODE_W6 << 5) | (total << 7);
        u32 cw[12];
#pragma unroll
        for (int h = 0; h < 2; ++h) {
            const u32* cd = code + 8 * h;
            cw[6 * h + 0] = cd[0] | ((cd[6] & 0x03030303u) << 5);
            cw[6 * h + 1] = cd[1] | ((cd[6] & 0x0C0C0C0Cu) << 3);
            cw[6 * h + 2] = cd[2] | ((cd[6] & 0x10101010u) << 1) | ((cd[6] & 0x80808080u) >> 1);
            cw[6 * h + 3] = cd[3] | ((cd[7] & 0x03030303u) << 5);
            cw[6 * h + 4] = cd[4] | ((cd[7] & 0x0C0C0C0Cu) << 3);
            cw[6 * h + 5] = cd[5] | ((cd[7] & 0x10101010u) << 1) | ((cd[7] & 0x80808080u) >> 1);
        }
        asm volatile("" ::: "memory");
        e.hv = *(const u32x4*)((const char*)list + (lane & 7) * 16);
        e.c3 = *(const u32x4*)((const char*)list + 128 + (lane < 56 ? lane : 0) * 16);
        e.c0 = u32x4{cw[0], cw[1], cw[2], cw[3]};
        e.c1 = u32x4{cw[4], cw[5], cw[6], cw[7]};
        e.c2 = u32x4{cw[8], cw[9], cw[10], cw[11]};
        e.lh = (min(total, NINL) + 2 + 7) >> 3;
        e.l2 = 64;
        e.l3 = total > NINL ? (total - NINL + 7) >> 3 : 0;
        asm volatile("" ::: "memory");
    } else {
        e.hv = u32x4{(lane == 0) ? (MODE_RAW << 5) : 0u, 0u, 0u, 0u};
        e.c0 = u32x4{T[0], T[1], T[2], T[3]};
        e.c1 = u32x4{T[4], T[5], T[6], T[7]};
        e.c2 = u32x4{T[8], T[9], T[10], T[11]};
        e.c3 = u32x4{T[12], T[13], T[14], T[15]};
        e.lh = 1;
        e.l2 = 64;
        e.l3 = 64;
    }
}

template <int IMM>
__device__ __forceinline__ void st16_all(char* const (&dst)[NR], u32 off, u32x4 v, u32 lanes) {
    const u64 mask = lanes >= 64 ? ~0ull : (1ull << lanes) - 1;
    u64 save;
    asm volatile("s_mov_b64 %0, exec\n\ts_mov_b64 exec, %1\n\t"
                 "global_store_dwordx4 %2, %3, %4 offset:%12 sc0 sc1\n\t"
                 "global_store_dwordx4 %2, %3, %5 offset:%12 sc0 sc1\n\t"
                 "global_store_dwordx4 %2, %3, %6 offset:%12 sc0 sc1\n\t"
                 "global_store_dwordx4 %2, %3, %7 offset:%12 sc0 sc1\n\t"
                 "global_store_dwordx4 %2, %3, %8 offset:%12 sc0 sc1\n\t"
                 "global_store_dwordx4 %2, %3, %9 offset:%12 sc0 sc1\n\t"
                 "global_store_dwordx4 %2, %3, %10 offset:%12 sc0 sc1\n\t"
                 "global_store_dwordx4 %2, %3, %11 offset:%12 sc0 sc1\n\t"
                 "s_mov_b64 exec, %0\n\ts_nop 1"
                 : "=&s"(save)
                 : "s"(mask), "v"(off), "v"(v), "s"(dst[0]), "s"(dst[1]), "s"(dst[2]), "s"(dst[3]), "s"(dst[4]),
                   "s"(dst[5]), "s"(dst[6]), "s"(dst[7]), "i"(IMM)
                 : "memory");
}
__device__ __forceinline__ void store_unit(const Enc& e, char* const (&dst)[NR], u32 slotoff, int lane) {
    const u32 off = slotoff + 128 + lane * 16;
    st16_all<0>(dst, off, e.c0, 64);
    st16_all<1024>(dst, off, e.c1, 64);
    st16_all<2048>(dst, off, e.c2, e.l2);
    st16_all<0>(dst, slotoff + lane * 16, e.hv, e.lh);
    if (e.l3) st16_all<3072>(dst, off, e.c3, e.l3);
}

__device__ __forceinline__ void flag_store(char* a, u32 call) {
    __hip_atomic_store((gu32*)(u64)a, call, __ATOMIC_RELAXED, __HIP_MEMORY_SCOPE_SYSTEM);
}

constexpr int RSLOT = 5 * 1024 + 64;
struct Ring {
    char* slot;
    u32* head;
    u32* done;
    u32* ready;
};
__device__ __forceinline__ Ring ring_of(char* lds) {
    char* b = lds + NE * SND_LDS;
    return Ring{b + 64, (u32*)b, (u32*)(b + 4), (u32*)(b + 16)};
}
__device__ __forceinline__ u32 lds_load(const u32* p) {
    return __hip_atomic_load(p, __ATOMIC_RELAXED, __HIP_MEMORY_SCOPE_WORKGROUP);
}

__device__ __forceinline__ void encoder(const Params& p, char* lds, int e) {
    const int lane = threadIdx.x & 63;
    char* const myws = p.ws[p.rank];
    u32* const cnt = (u32*)(myws + CNT_OFF + (size_t)(blockIdx.x * 8 + e + 1) * 16);
    const SendLds L = send_lds(lds + e * SND_LDS);
    const Ring R = ring_of(lds);
    const u32 rawaddr = __builtin_amdgcn_readfirstlane(lds_addr(L.raw));
    if (lane < 16) {
        u32 sel = 0x0C0C0C0Cu, n = 0;
#pragma unroll
        for (int y = 0; y < 4; ++y)
            if ((lane >> y) & 1) { sel = (sel & ~(0xFFu << (8 * n))) | ((u32)y << (8 * n)); ++n; }
        ((u32*)L.lut)[lane] = sel;
    }
    const unsigned char* const wsrc = p.w + (size_t)(lane >> 2) * NN + (lane & 3) * 16;
    u32* const tick = (u32*)uni(myws + TICK_OFF);
    const u32 call = __builtin_amdgcn_readfirstlane(*cnt + 1);
    const u32 tbase = (call - 1) * (NU + NS * NE);
    u32 b8 = 32, b3 = 32, lastu = 0;
    while (true) {
        u32 pos = 0;
        if (lane == 0) pos = __hip_atomic_fetch_add(R.head, 1u, __ATOMIC_RELAXED, __HIP_MEMORY_SCOPE_WORKGROUP);
        pos = __builtin_amdgcn_readfirstlane(pos);
        const u32 depth = lastu + RINGL_FROM >= (u32)W5_UNITS ? RINGL : RING;
        while (pos >= __builtin_amdgcn_readfirstlane(lds_load(R.done)) + depth) __builtin_amdgcn_s_sleep(2);
        asm volatile("" ::: "memory");
        u32 tk;
        draw_ticket(tk, tick);
        wait_tk<0>(tk);
        const u32 u = __builtin_amdgcn_readfirstlane(tk) - tbase;
        char* const sl = R.slot + (pos % RING) * RSLOT;
        u32* const meta = (u32*)(sl + 5 * 1024);
        if (u >= (u32)NU) {
            if (lane == 0) {
                meta[0] = NU;
                __hip_atomic_store(&R.ready[pos % RING], pos + 1, __ATOMIC_RELEASE, __HIP_MEMORY_SCOPE_WORKGROUP);
                *cnt = call;
            }
            return;
        }
        lastu = u;
        dma_raw(wsrc + (size_t)(u / NC) * 64 * NN + (u % NC) * 64, rawaddr);
        wait_vm0();
        Enc en;
        encode(en, L.raw, L, lane, b8, b3, w5_round((int)(u / NC)));
        *(u32x4*)(sl + lane * 16) = en.c0;
        *(u32x4*)(sl + 1024 + lane * 16) = en.c1;
        *(u32x4*)(sl + 2048 + lane * 16) = en.c2;
        *(u32x4*)(sl + 3072 + lane * 16) = en.c3;
        *(u32x4*)(sl + 4096 + lane * 16) = en.hv;
        if (lane == 0) {
            *(u32x4*)meta = u32x4{u, en.lh, en.l2, en.l3};
            __hip_atomic_store(&R.ready[pos % RING], pos + 1, __ATOMIC_RELEASE, __HIP_MEMORY_SCOPE_WORKGROUP);
        }
    }
}

__device__ __forceinline__ void storer(const Params& p, char* lds) {
    const int lane = threadIdx.x & 63;
    const int rank = p.rank;
    const int wid = blockIdx.x;
    char* const myws = p.ws[rank];
    u32* const cnt = (u32*)(myws + CNT_OFF + (size_t)(blockIdx.x * 8) * 16);
    STAMP(blockIdx.x * 128 + 0);
    STAMP_HW(blockIdx.x * 128 + 15);
    const Ring R = ring_of(lds);
    const u32 call = __builtin_amdgcn_readfirstlane(*cnt + 1);
    const size_t par = DATA_OFF + (size_t)(call & 1) * PAR_BYTES;
    char* flagbase = p.ws[lane & (NR - 1)] + par + (size_t)rank * BLK + WF_OFF;
    char* dst[NR];
    const int rot = (wid + (wid >> 3)) & (NR - 1);
#pragma unroll
    for (int d = 0; d < NR; ++d) dst[d] = uni(p.ws[(rank + 1 + d + rot) & (NR - 1)] + par);
    asm volatile("" : "+v"(flagbase));

    int uprev = -1;
    int ends = 0;
    for (u32 pos = 0; ends < NE; ++pos) {
        while (__builtin_amdgcn_readfirstlane(lds_load(&R.ready[pos % RING])) != pos + 1) {
            if (uprev >= 0 && FLUSH_IDLE) {
                wait_vm0();
                if (lane < NR) flag_store(flagbase + (size_t)uprev * 128, call);
                uprev = -1;
            }
            __builtin_amdgcn_s_sleep(1);
        }
        asm volatile("" ::: "memory");
        const char* const sl = R.slot + (pos % RING) * RSLOT;
        const u32x4 meta = *(const u32x4*)(sl + 5 * 1024);
        const u32 u = __builtin_amdgcn_readfirstlane(meta.x);
        Enc e;
        if (u < (u32)NU) {
            e.c0 = *(const u32x4*)(sl + lane * 16);
            e.c1 = *(const u32x4*)(sl + 1024 + lane * 16);
            e.c2 = *(const u32x4*)(sl + 2048 + lane * 16);
            e.c3 = *(const u32x4*)(sl + 3072 + lane * 16);
            e.hv = *(const u32x4*)(sl + 4096 + lane * 16);
            e.lh = __builtin_amdgcn_readfirstlane(meta.y);
            e.l2 = __builtin_amdgcn_readfirstlane(meta.z);
            e.l3 = __builtin_amdgcn_readfirstlane(meta.w);
            asm volatile("" :: "v"(e.c0), "v"(e.c1), "v"(e.c2), "v"(e.c3), "v"(e.hv));
        }
        if (lane == 0) __hip_atomic_store(R.done, pos + 1, __ATOMIC_RELEASE, __HIP_MEMORY_SCOPE_WORKGROUP);
        if (u >= (u32)NU) { ++ends; continue; }
        store_unit(e, dst, slot_off(rank, u), lane);
        asm volatile("s_waitcnt vmcnt(32)" ::: "memory");
        if (uprev >= 0) {
            if (lane < NR) flag_store(flagbase + (size_t)uprev * 128, call);
        }
        uprev = u;
    }
    if (uprev >= 0) {
        wait_vm0();
        if (lane < NR) flag_store(flagbase + (size_t)uprev * 128, call);
    }
    if (lane == 0) *cnt = call;
}

__device__ __forceinline__ void decode_unit(char* img, __amdgpu_buffer_rsrc_t rs, u32 slot, int lane, u32x4 cw0,
                                            u32x4 cw1, u32x4 cw2, u32 ent, u32x2 t2, u32x2 vv, bool maybe5) {
    const u32 hdr = __builtin_amdgcn_readfirstlane(ent) & 0xFFFFu;
    const u32 mode = (hdr >> 5) & 3;
    const u32 total = hdr >> 7;
    const int q = lane & 15, g = lane >> 4;
    char* const wp = img + (g & 1) * 512 + (g >> 1) * 256 + q * 16;
    if (mode == MODE_RAW) {
        const u32x4 v3 = __builtin_amdgcn_raw_buffer_load_b128(rs, slot + 128 + 3072 + lane * 16, 0, AUX_SYS);
        *(u32x4*)(wp + 0) = cw0;
        *(u32x4*)(wp + 1024) = cw1;
        *(u32x4*)(wp + 2048) = cw2;
        *(u32x4*)(wp + 3072) = v3;
        return;
    }
    const u32 B4 = ((hdr & 31) << 2) * 0x01010101u;
    if (maybe5 && mode == MODE_W5) {
        ((u32*)img)[lane] = vv.x;
        ((u32*)img)[64 + lane] = vv.y;
        const u32 c[10] = {cw0.x, cw0.y, cw0.z, cw0.w, cw1.x, cw1.y, cw1.z, cw1.w, t2.x, t2.y};
        u32 o[16], n = 0;
#pragma unroll
        for (int h = 0; h < 2; ++h) {
            const u32* cc = c + 5 * h;
            u32* oo = o + 8 * h;
#pragma unroll
            for (int i = 0; i < 5; ++i) oo[i] = cc[i] & 0x8F8F8F8Fu;
            oo[5] = ((cc[0] >> 4) & 0x07070707u) | ((cc[1] >> 1) & 0x08080808u) | ((cc[1] << 2) & 0x80808080u);
            oo[6] = ((cc[2] >> 4) & 0x07070707u) | ((cc[3] >> 1) & 0x08080808u) | ((cc[1] << 1) & 0x80808080u);
            oo[7] = ((cc[4] >> 4) & 0x07070707u) | ((cc[3] >> 2) & 0x08080808u) | ((cc[3] << 1) & 0x80808080u);
        }
        u32 fm[16], m4[16], t3[16];
#pragma unroll
        for (int i = 0; i < 16; ++i) {
            m4[i] = o[i] & (o[i] >> 1) & 0x04040404u;
            fm[i] = shl<5>(m4[i]) - m4[i];
            t3[i] = __builtin_popcount(fm[i] & 0x1C1C1C1Cu);
            n += t3[i];
        }
        const u32 bit0 = wave_incl_scan(n) - n;
        u32 tlo, thi;
        w5_table(hdr & 31, tlo, thi);
        asm volatile("" ::: "memory");
        const u32* const sw = (const u32*)img + (bit0 >> 5);
        const u32 w0 = sw[0], w1 = sw[1], w2 = sw[2], w3 = sw[3];
        u32 v0 = __builtin_amdgcn_alignbit(w1, w0, bit0), v1 = __builtin_amdgcn_alignbit(w2, w1, bit0),
            v2 = __builtin_amdgcn_alignbit(w3, w2, bit0);
#pragma unroll
        for (int i = 0; i < 16; ++i) {
            const u32 t6 = (v0 & 0x3Fu) | (shl<10>(v0) & 0x003F0000u);
            const u32 kx = (t6 | shl<5>(t6)) & 0x07070707u;
            v0 = __builtin_amdgcn_alignbit(v1, v0, t3[i]);
            v1 = __builtin_amdgcn_alignbit(v2, v1, t3[i]);
            v2 >>= t3[i];
            u32 rk;
            asm("v_mul_u32_u24 %0, %1, %2" : "=v"(rk) : "v"(m4[i]), "s"(0x404040u));
            const u32 eb = perm(thi, tlo, perm(0u, kx, rk));
            o[i] = (fm[i] & eb) | (~fm[i] & (o[i] + B4));
        }
        asm volatile("" ::: "memory");
#pragma unroll
        for (int f = 0; f < 4; ++f) *(u32x4*)(wp + f * 1024) = u32x4{o[4 * f], o[4 * f + 1], o[4 * f + 2], o[4 * f + 3]};
    } else {
        const u32 c[12] = {cw0.x, cw0.y, cw0.z, cw0.w, cw1.x, cw1.y, cw1.z, cw1.w, cw2.x, cw2.y, cw2.z, cw2.w};
#pragma unroll
        for (int h = 0; h < 2; ++h) {
            const u32* cc = c + 6 * h;
            u32x4 a, bq;
            a.x = (cc[0] & 0x9F9F9F9Fu) + B4;
            a.y = (cc[1] & 0x9F9F9F9Fu) + B4;
            a.z = (cc[2] & 0x9F9F9F9Fu) + B4;
            a.w = (cc[3] & 0x9F9F9F9Fu) + B4;
            bq.x = (cc[4] & 0x9F9F9F9Fu) + B4;
            bq.y = (cc[5] & 0x9F9F9F9Fu) + B4;
            bq.z = (((cc[0] >> 5) & 0x03030303u) | ((cc[1] >> 3) & 0x0C0C0C0Cu) |
                    ((cc[2] >> 1) & 0x10101010u) | ((cc[2] << 1) & 0x80808080u)) + B4;
            bq.w = (((cc[3] >> 5) & 0x03030303u) | ((cc[4] >> 3) & 0x0C0C0C0Cu) |
                    ((cc[5] >> 1) & 0x10101010u) | ((cc[5] << 1) & 0x80808080u)) + B4;
            *(u32x4*)(wp + (2 * h) * 1024) = a;
            *(u32x4*)(wp + (2 * h + 1) * 1024) = bq;
        }
    }
    {
        const u32 e = (lane < 2 || (u32)lane >= total + 2) ? 0u : (ent & 0xFFFFu);
        const u32 pos = e >> 4;
        __hip_atomic_fetch_xor((u32*)(img + (pos & ~3u)), (e & 15) << (3 + 8 * (pos & 3)), __ATOMIC_RELAXED,
                               __HIP_MEMORY_SCOPE_WORKGROUP);
    }
#pragma clang loop unroll(disable)
    for (u32 i0 = NINL; i0 < total; i0 += 64) {
        const u32 i = i0 + lane;
        u32 e = __builtin_amdgcn_raw_buffer_load_b16(rs, slot + 128 + 3072 + (i - NINL) * 2, 0, AUX_SYS);
        e = (i < total) ? (e & 0xFFFFu) : 0u;
        const u32 pos = e >> 4;
        __hip_atomic_fetch_xor((u32*)(img + (pos & ~3u)), (e & 15) << (3 + 8 * (pos & 3)), __ATOMIC_RELAXED,
                               __HIP_MEMORY_SCOPE_WORKGROUP);
    }
}

__device__ __forceinline__ void send_x(const Params& p, u32 call, size_t par, int h0, int lane) {
    u32x4 v[XN][8];
    char* base[XN];
    char* xf[XN];
#pragma unroll
    for (int n = 0; n < XN; ++n) {
        const int dest = __builtin_amdgcn_readfirstlane((p.rank + 1 + (h0 + n) + ((h0 + n) >> 3)) & (NR - 1));
        const int rf = __builtin_amdgcn_readfirstlane(((h0 + n) >> 3) & 31);
        const unsigned char* src = p.x + (size_t)(dest * MS + rf * 16 + (lane & 15)) * KS
                                 + ((lane >> 4) & 1) * 32 + (lane >> 5) * 16;
#pragma unroll
        for (int j = 0; j < 8; ++j) v[n][j] = *(const u32x4*)(src + j * 64);
        base[n] = uni(p.ws[dest] + par + (size_t)p.rank * BLK + X_OFF + rf * 8192);
        xf[n] = uni(p.ws[dest] + par + (size_t)p.rank * BLK + XF_OFF + rf * 128);
    }
#pragma unroll
    for (int n = 0; n < XN; ++n)
#pragma unroll
        for (int j = 0; j < 8; ++j) asm volatile("" :: "v"(v[n][j]));
#pragma unroll
    for (int n = 0; n < XN; ++n)
#pragma unroll
        for (int j = 0; j < 8; j += 4) {
            st16_sys<0>(base[n] + j * 1024, lane * 16, v[n][j]);
            st16_sys<1024>(base[n] + j * 1024, lane * 16, v[n][j + 1]);
            st16_sys<2048>(base[n] + j * 1024, lane * 16, v[n][j + 2]);
            st16_sys<3072>(base[n] + j * 1024, lane * 16, v[n][j + 3]);
        }
    wait_vm0();
    if (lane == 0) {
#pragma unroll
        for (int n = 0; n < XN; ++n)
            __hip_atomic_store((gu32*)(u64)xf[n], call, __ATOMIC_RELAXED, __HIP_MEMORY_SCOPE_SYSTEM);
    }
}

template <int NF>
__device__ __forceinline__ void consumer_wave(const Params& p, char* lds, int c, int wave, int rf0, int lf0, u32 call, size_t par) {
    const int lane = threadIdx.x & 63;
    char* const myws = uni(p.ws[p.rank]);
    gu32* const tmo = (gu32*)(u64)(myws + TMO_OFF);
    const float scale = p.sx[0] * p.sw[0];
    const __amdgpu_buffer_rsrc_t rs =
        __builtin_amdgcn_make_buffer_rsrc(uni(myws + par), 0, (int)PAR_BYTES, 0x00020000);
    u32* const badw = (u32*)(lds + 65536);
    u32* const rdy = (u32*)(lds + 65536 + 32);
    const __amdgpu_buffer_rsrc_t ro = __builtin_amdgcn_make_buffer_rsrc(uni((char*)p.out), 0, MS * NN * 4, 0x00020000);

    f32x4 acc[NF][4];

    const u32 xoff = (u32)((lane >> 5) * BLK + X_OFF + rf0 * 8192 + (lane & 31) * 16);
    const int L16 = lane & 15, G = lane >> 4;
    const u32 bread = (G >> 1) * 4096 + (G & 1) * 256 + L16 * 16;

    u32x4 A[4 * NF * 2];
    auto load_a = [&](int P, int j) {
#pragma unroll
        for (int a = 0; a < NF; ++a)
#pragma unroll
            for (int i = 0; i < 2; ++i)
                A[(P * NF + a) * 2 + i] = __builtin_amdgcn_raw_buffer_load_b128(
                    rs, xoff + j * 1024 + i * 512, P * 2 * BLK + a * 8192, AUX_SYS);
    };
    char* const a7 = lds + A7_OFF + lf0 * 8192;
    auto dma_a7 = [&]() {
        const char* src = myws + par + xoff + (NJ - 1) * 1024;
        asm volatile("" : "+v"(src));
        const u32 dst = __builtin_amdgcn_readfirstlane(lds_addr(a7));
#pragma unroll
        for (int a = 0; a < NF; ++a)
#pragma unroll
            for (int P = 0; P < 4; ++P) {
                u32 keep;
                const char* s0 = src + a * 8192 + (size_t)P * 2 * BLK;
                asm volatile("s_mov_b32 %0, m0\n\ts_mov_b32 m0, %2\n\ts_nop 0\n\tglobal_load_lds_dwordx4 %1, off sc0 sc1\n\t"
                             "s_add_u32 m0, m0, 0x400\n\ts_nop 0\n\tglobal_load_lds_dwordx4 %3, off sc0 sc1\n\ts_mov_b32 m0, %0"
                             : "=&s"(keep) : "v"(s0), "s"(dst + a * 8192 + P * 2048), "v"(s0 + 512) : "memory", "scc");
            }
    };
    auto mfma = [&](int P, int f, const u32x4& b0, const u32x4& b1) {
        const i32x8 B = {(int)b0.x, (int)b0.y, (int)b0.z, (int)b0.w, (int)b1.x, (int)b1.y, (int)b1.z, (int)b1.w};
#pragma unroll
        for (int a = 0; a < NF; ++a) {
            const u32x4 a0 = A[(P * NF + a) * 2], a1 = A[(P * NF + a) * 2 + 1];
            const i32x8 Av = {(int)a0.x, (int)a0.y, (int)a0.z, (int)a0.w, (int)a1.x, (int)a1.y, (int)a1.z, (int)a1.w};
            acc[a][f] = __builtin_amdgcn_mfma_scale_f32_16x16x128_f8f6f4(Av, B, acc[a][f], 0, 1, 0, 0, 0, 0);
        }
    };
    u32x4 cw0, cw1, cw2;
    u32x2 t2 = {0, 0};
    u32x2 vv = {0, 0};
    u32 ent;
    auto unit_slot = [&](int j) { return slot_off(wave, j * NC + c); };
    auto issue = [&](int j, u32 oob) {
        const u32 slot = unit_slot(j) | oob;
        ent = __builtin_amdgcn_raw_buffer_load_b16(rs, slot + lane * 2, 0, AUX_SYS);
        cw0 = __builtin_amdgcn_raw_buffer_load_b128(rs, slot + 128 + lane * 16, 0, AUX_SYS);
        cw1 = __builtin_amdgcn_raw_buffer_load_b128(rs, slot + 128 + 1024 + lane * 16, 0, AUX_SYS);
        cw2 = __builtin_amdgcn_raw_buffer_load_b128(rs, slot + 128 + 2048 + lane * 16, 0, AUX_SYS);
        if (w5_round(j)) {
            t2 = __builtin_amdgcn_raw_buffer_load_b64(rs, slot + 128 + 2048 + lane * 8, 0, AUX_SYS);
            vv.x = __builtin_amdgcn_raw_buffer_load_b32(rs, slot + 128 + 2560 + lane * 4, 0, AUX_SYS);
            vv.y = __builtin_amdgcn_raw_buffer_load_b32(rs, slot + 128 + 2560 + 256 + lane * 4, 0, AUX_SYS);
        }
    };
    auto decode = [&](int j, u32 oob) {
        decode_unit(lds + (j & 1) * 32768 + wave * 4096, rs, unit_slot(j) | oob, lane, cw0, cw1, cw2, ent, t2, vv,
                    w5_round(j));
    };
    auto wflag = [&](int j) { return (const gu32*)(u64)(myws + par + (size_t)wave * BLK + WF_OFF + (j * NC + c) * 128); };
    bool bad = false;

    bool have = false;
    auto round = [&](int j) {
        if (!have) {
            bad |= wait_flags(wflag(j), lane == 0, call, tmo);
            issue(j, 0);
        }
        decode(j, 0);
        __syncthreads();
        if (j == NJ - 3) {
            const u32 dummy = __builtin_amdgcn_raw_buffer_load_b32(ro, (u32)(((rf0 * 16 + G * 4) * NN + c * 64 + L16 * 4) * 4), 0, 16);
            asm volatile("" :: "v"(dummy));
        }
        u32 pv = __hip_atomic_load(wflag(j + 1), __ATOMIC_RELAXED, __HIP_MEMORY_SCOPE_SYSTEM);
        const char* const buf = lds + (j & 1) * 32768;
        u32x4 Bf[2][2][2];
#pragma unroll
        for (int f = 0; f < 2; ++f) {
            Bf[0][f][0] = *(const u32x4*)(buf + f * 1024 + bread);
            Bf[0][f][1] = *(const u32x4*)(buf + f * 1024 + bread + 512);
        }
#pragma unroll
        for (int st = 0; st < 8; ++st) {
            if (st < 7) {
                const int Pn = (st + 1) >> 1, fn = ((st + 1) & 1) * 2;
#pragma unroll
                for (int f = 0; f < 2; ++f) {
                    Bf[(st + 1) & 1][f][0] = *(const u32x4*)(buf + Pn * 8192 + (fn + f) * 1024 + bread);
                    Bf[(st + 1) & 1][f][1] = *(const u32x4*)(buf + Pn * 8192 + (fn + f) * 1024 + bread + 512);
                }
            }
            __builtin_amdgcn_sched_barrier(0);
#pragma unroll
            for (int f = 0; f < 2; ++f) mfma(st >> 1, (st & 1) * 2 + f, Bf[st & 1][f][0], Bf[st & 1][f][1]);
            if ((st & 1) && j + 2 < NJ) load_a(st >> 1, j + 1);
            if (st == EARLY_ST) {
                have = __builtin_amdgcn_readfirstlane(pv) >= call;
                asm volatile("" ::: "memory");
                if (have) issue(j + 1, 0);
                else pv = __hip_atomic_load(wflag(j + 1), __ATOMIC_RELAXED, __HIP_MEMORY_SCOPE_SYSTEM);
            }
            if (st == EARLY_ST2 && !have) {
                have = __builtin_amdgcn_readfirstlane(pv) >= call;
                asm volatile("" ::: "memory");
                if (have) issue(j + 1, 0);
            }
            __builtin_amdgcn_sched_barrier(0);
        }
        STAMP(blockIdx.x * 128 + wave * 16 + 2 + j);
    };

#pragma clang loop unroll(disable)
    for (u32 pass = 0; pass < 2; ++pass) {
        asm volatile("" : "+s"(pass));
        const u32 oob = pass ? 0u : 0x80000000u;
#pragma unroll
        for (int a = 0; a < NF; ++a)
#pragma unroll
            for (int f = 0; f < 4; ++f) acc[a][f] = f32x4{0.f, 0.f, 0.f, 0.f};
        if (pass) {
            bad = wait_flags((const gu32*)(u64)(myws + par + (size_t)(lane & 7) * BLK + XF_OFF + (rf0 + ((lane >> 3) & (NF - 1))) * 128),
                             lane < NF * NR, call, tmo);
            STAMP(blockIdx.x * 128 + wave * 16 + 1);
            dma_a7();
#pragma unroll
            for (int P = 0; P < 4; ++P) load_a(P, 0);
#pragma clang loop unroll(disable)
            for (int j = 0; j < NJ - 1; ++j) round(j);
            asm volatile("s_waitcnt vmcnt(0)" ::: "memory");
        }

        {
            constexpr int j = NJ - 1;
            const char* const bbuf = lds + (j & 1) * 32768 + bread;
            const char* const abuf = a7 + lane * 16;
            const gu32* const fl = wflag(j);
            const u64 t0 = __builtin_amdgcn_s_memrealtime();
            asm volatile("" :: "s"(t0));
            bool own = false;
            u32 todo = 15, avail = 0, it = 0;
            auto pick = [&]() {
                if (!pass) { own = false; avail = 0; todo = 15; }
                while (!avail) {
                    u32 v = call;
                    if (!own && pass && !have) v = __hip_atomic_load(fl, __ATOMIC_RELAXED, __HIP_MEMORY_SCOPE_SYSTEM);
                    const u32x4 r0 = *(const u32x4*)rdy, r1 = *(const u32x4*)(rdy + 4);
                    const u32 m = (min(r0.x, r0.y) > pass ? 1u : 0u) | (min(r0.z, r0.w) > pass ? 2u : 0u) |
                                  (min(r1.x, r1.y) > pass ? 4u : 0u) | (min(r1.z, r1.w) > pass ? 8u : 0u);
                    avail = __builtin_amdgcn_readfirstlane(m & todo);
                    todo &= ~avail;
                    if (!own) {
                        bool go = __builtin_amdgcn_readfirstlane(v) >= call;
                        if (!go && (++it & 15) == 15 &&
                            (__hip_atomic_load(tmo, __ATOMIC_RELAXED, __HIP_MEMORY_SCOPE_AGENT) == call ||
                             __builtin_amdgcn_s_memrealtime() - t0 > DEADLINE)) {
                            __hip_atomic_store(tmo, call, __ATOMIC_RELAXED, __HIP_MEMORY_SCOPE_AGENT);
                            bad = true;
                            go = true;
                        }
                        if (go) {
                            asm volatile("" ::: "memory");
                            STAMP(blockIdx.x * 128 + wave * 16 + 11);
                            if (!have) issue(j, oob);
                            decode(j, oob);
                            if (lane == 0) {
                                badw[wave] = bad;
                                __hip_atomic_store(&rdy[wave], pass + 1, __ATOMIC_RELEASE, __HIP_MEMORY_SCOPE_WORKGROUP);
                            }
                            STAMP(blockIdx.x * 128 + wave * 16 + 13);
                            own = true;
                        }
                    } else if (!avail) {
                        __builtin_amdgcn_s_sleep(1);
                    }
                }
                asm volatile("" ::: "memory");
                const int P = __builtin_ctz(avail);
                avail &= avail - 1;
                return P;
            };
            struct Frags { u32x4 a[NF][2], b[4][2]; };
            auto rd_a = [&](Frags& F, int P, int a) {
                F.a[a][0] = *(const u32x4*)(abuf + a * 8192 + P * 2048);
                F.a[a][1] = *(const u32x4*)(abuf + a * 8192 + P * 2048 + 1024);
            };
            auto rd_b = [&](Frags& F, int P, int f) {
                F.b[f][0] = *(const u32x4*)(bbuf + P * 8192 + f * 1024);
                F.b[f][1] = *(const u32x4*)(bbuf + P * 8192 + f * 1024 + 512);
            };
            auto mm_f = [&](const Frags& F, int f) {
                const i32x8 B = {(int)F.b[f][0].x, (int)F.b[f][0].y, (int)F.b[f][0].z, (int)F.b[f][0].w,
                                 (int)F.b[f][1].x, (int)F.b[f][1].y, (int)F.b[f][1].z, (int)F.b[f][1].w};
#pragma unroll
                for (int a = 0; a < NF; ++a) {
                    const i32x8 Av = {(int)F.a[a][0].x, (int)F.a[a][0].y, (int)F.a[a][0].z, (int)F.a[a][0].w,
                                      (int)F.a[a][1].x, (int)F.a[a][1].y, (int)F.a[a][1].z, (int)F.a[a][1].w};
                    acc[a][f] = __builtin_amdgcn_mfma_scale_f32_16x16x128_f8f6f4(Av, B, acc[a][f], 0, 1, 0, 0, 0, 0);
                }
            };
            auto mmrd = [&](const Frags& Fc, Frags& Fn, int P) {
#pragma unroll
                for (int f = 0; f < 4; ++f) {
                    mm_f(Fc, f);
                    if (f < NF) rd_a(Fn, P, f);
                    rd_b(Fn, P, f);
                    __builtin_amdgcn_sched_barrier(0);
                }
            };
            Frags F0, F1;
            {
                const int P = pick();
#pragma unroll
                for (int a = 0; a < NF; ++a) rd_a(F0, P, a);
#pragma unroll
                for (int f = 0; f < 4; ++f) rd_b(F0, P, f);
            }
            mmrd(F0, F1, pick());
            mmrd(F1, F0, pick());
            mmrd(F0, F1, pick());
#pragma unroll
            for (int f = 0; f < 4; ++f) mm_f(F1, f);
        }
        STAMP(blockIdx.x * 128 + wave * 16 + 9);

        const u32x4 b0 = *(const u32x4*)badw, b1 = *(const u32x4*)(badw + 4);
        const float s = (b0.x | b0.y | b0.z | b0.w | b1.x | b1.y | b1.z | b1.w) ? __builtin_nanf("") : scale;
        const u32 obase = (u32)(((rf0 * 16 + G * 4) * NN + c * 64 + L16 * 4) * 4) | oob;
#pragma unroll
        for (int a = 0; a < NF; ++a)
#pragma unroll
            for (int r = 0; r < 4; ++r) {
                const f32x4 v = {acc[a][0][r] * s, acc[a][1][r] * s, acc[a][2][r] * s, acc[a][3][r] * s};
                __builtin_amdgcn_raw_buffer_store_b128(__builtin_bit_cast(u32x4, v), ro, obase + (a * 16 + r) * NN * 4, 0, OUT_AUX);
            }
        STAMP(blockIdx.x * 128 + wave * 16 + 10);
    }
}

__device__ __forceinline__ void consumer(const Params& p, char* lds) {
    const int lane = threadIdx.x & 63;
    const int wave = __builtin_amdgcn_readfirstlane(threadIdx.x >> 6);
    const int t = blockIdx.x - NS;
    const int x = t & 7, i = t >> 3;
    int c, f0, nfr;
    if (i < 12) { c = 4 * x + i / 3; f0 = (i % 3) * 11; nfr = (i % 3 == 2) ? 10 : 11; }
    else        { c = 32 + 4 * x + ((i - 12) >> 2); f0 = ((i - 12) & 3) * 8; nfr = 8; }
    const int n2 = nfr - 8;
    const int lf0 = wave < n2 ? 2 * wave : n2 + wave;
    char* const myws = p.ws[p.rank];
    u32* const cnt = (u32*)(myws + CNT_OFF + (size_t)(blockIdx.x * 8 + wave) * 16);
    const u32 call = __builtin_amdgcn_readfirstlane(*cnt + 1);
    const size_t par = DATA_OFF + (size_t)(call & 1) * PAR_BYTES;
    STAMP(blockIdx.x * 128 + wave * 16 + 0);
    STAMP_HW(blockIdx.x * 128 + wave * 16 + 15);
    if (lane == 0) ((u32*)(lds + 65536 + 32))[wave] = 0;
    if (wave == (t & 7)) send_x(p, call, par, blockIdx.x, lane);
    if (wave < n2) consumer_wave<2>(p, lds, c, wave, f0 + lf0, lf0, call, par);
    else consumer_wave<1>(p, lds, c, wave, f0 + lf0, lf0, call, par);
    if (lane == 0) *cnt = call;
}

__global__ void __launch_bounds__(512, 2) gemm_rs_kernel(Params p) {
    __shared__ __attribute__((aligned(16))) char lds[LDS_BYTES];
    if (blockIdx.x < NS) {
        const int wave = __builtin_amdgcn_readfirstlane(threadIdx.x >> 6);
        if (threadIdx.x < 16) ((u32*)(lds + NE * SND_LDS))[threadIdx.x] = 0;
        __syncthreads();
        if (wave == 0) {
            storer(p, lds);
        } else if (wave <= NE) {
            encoder(p, lds, wave - 1);
        } else if (wave == 7) {
            const int lane = threadIdx.x & 63;
            u32* const cnt = (u32*)(p.ws[p.rank] + CNT_OFF + (size_t)(blockIdx.x * 8 + wave) * 16);
            const u32 call = __builtin_amdgcn_readfirstlane(*cnt + 1);
            send_x(p, call, DATA_OFF + (size_t)(call & 1) * PAR_BYTES, blockIdx.x, lane);
            if (lane == 0) *cnt = call;
        }
    } else {
        consumer(p, lds);
    }
}

}

extern "C" void kernel_launch(int rank, int nranks, ncclComm_t comm, hipStream_t stream,
                              void* const* d_in, const int* in_sizes, int n_in,
                              void* d_out, int out_size, const int* in_dtypes, int out_dtype,
                              void* const* d_out_all, void* const* d_ws_all, size_t ws_bytes) {
    (void)comm; (void)in_dtypes; (void)out_dtype; (void)d_out_all;
    if (nranks != NR || n_in != 4 || rank < 0 || rank >= NR || ws_bytes < WS_NEED ||
        in_sizes[0] != 4096 * KS || in_sizes[1] != KS * NN || out_size != MS * NN) {
        fprintf(stderr, "gemm_rs: unexpected problem shape\n");
        return;
    }
    Params p;
    p.x = (const unsigned char*)d_in[0];
    p.w = (const unsigned char*)d_in[1];
    p.sx = (const float*)d_in[2];
    p.sw = (const float*)d_in[3];
    p.out = (float*)d_out;
    for (int r = 0; r < NR; ++r) p.ws[r] = (char*)d_ws_all[r];
    p.rank = rank;
    p.pad = 0;
    hipLaunchKernelGGL(gemm_rs_kernel, dim3(GRID), dim3(512), 0, stream, p);
    HIP_CHECK(hipGetLastError());
}
```

```cpp
#include <hip/hip_runtime.h>
#include <rccl/rccl.h>
#include <cstdio>
#include <cstdint>

#define HIP_CHECK(expr) do { hipError_t _e = (expr); if (_e != hipSuccess) { \
    fprintf(stderr, "HIP error %s at %s:%d\n", hipGetErrorString(_e), __FILE__, __LINE__); \
    return; } } while (0)

namespace {

typedef unsigned u32;
typedef unsigned long long u64;
typedef __attribute__((address_space(1))) unsigned gu32;
typedef unsigned u32x4 __attribute__((ext_vector_type(4)));
typedef unsigned u32x2 __attribute__((ext_vector_type(2)));
typedef int i32x8 __attribute__((ext_vector_type(8)));
typedef float f32x4 __attribute__((ext_vector_type(4)));

constexpr int NR = 8;
constexpr int MS = 512;
constexpr int KS = 512;
constexpr int NN = 4096;
constexpr int NJ = 8;
constexpr int NC = 64;
constexpr int NU = NJ * NC;

constexpr int NS = 32;
constexpr int NE = 5;
constexpr int RING = 4;
constexpr int RINGL = 3;
constexpr int RINGL_FROM = 96;
constexpr int NCONS = 224;
constexpr int GRID = NS + NCONS;
static_assert(GRID == 256 && NE <= 6, "one workgroup per CU, 256 x half-units");

constexpr size_t CNT_OFF   = 0;
constexpr size_t TMO_OFF   = 64 << 10;
constexpr size_t TICK_OFF  = 96 << 10;
constexpr size_t DATA_OFF  = 1 << 20;
constexpr u32    SLOT      = 4224;
constexpr u32    WF_OFF    = 0;
constexpr u32    XF_OFF    = 64 << 10;
constexpr u32    X_OFF     = 128 << 10;
constexpr u32    W_OFF     = 384 << 10;
constexpr u32    BLK       = 2560 << 10;
constexpr size_t PAR_BYTES = (size_t)NR * BLK;
constexpr size_t WS_NEED   = DATA_OFF + 2 * PAR_BYTES;
static_assert(W_OFF + NU * SLOT <= BLK, "block too small");
static_assert((size_t)GRID * 8 * 16 <= TMO_OFF, "counter region too small");

constexpr u64 DEADLINE = 40000000ull;
constexpr int AUX_SYS = 17;
constexpr int OUT_AUX = 16;
constexpr int W5_FROM = 1, W5_ROUNDS = 4;
constexpr int W5_UNITS = W5_ROUNDS * NC;
__device__ __forceinline__ bool w5_round(int j) { return j >= W5_FROM && j < W5_ROUNDS; }
constexpr int EARLY_ST = 2, EARLY_ST2 = 7;

struct Params {
    const unsigned char* x;
    const unsigned char* w;
    const float* sx;
    const float* sw;
    float* out;
    char* ws[NR];
    int rank;
    int pad;
};

__device__ __forceinline__ u32 slot_off(int src, int u) { return (u32)src * BLK + W_OFF + (u32)u * SLOT; }
template <int N>
__device__ __forceinline__ u32 shl(u32 x) {
    u32 r;
    asm("v_lshlrev_b32 %0, %1, %2" : "=v"(r) : "i"(N), "v"(x));
    return r;
}
__device__ __forceinline__ u32 perm(u32 hi, u32 lo, u32 sel) { return __builtin_amdgcn_perm(hi, lo, sel); }

template <int IMM>
__device__ __forceinline__ void st16_sys(char* base, u32 off, u32x4 v) {
    asm volatile("global_store_dwordx4 %0, %1, %2 offset:%3 sc0 sc1\n\ts_nop 1"
                 :: "v"(off), "v"(v), "s"(base), "i"(IMM) : "memory");
}
__device__ __forceinline__ char* uni(char* p) {
    const u64 v = (u64)p;
    const u32 lo = __builtin_amdgcn_readfirstlane((u32)v);
    const u32 hi = __builtin_amdgcn_readfirstlane((u32)(v >> 32));
    char* r = (char*)(((u64)hi << 32) | lo);
    asm volatile("s_nop 4" : "+s"(r));
    return r;
}
__device__ __forceinline__ void wait_vm0() { asm volatile("s_waitcnt vmcnt(0)" ::: "memory"); }


__device__ __forceinline__ bool wait_flags(const gu32* flag, bool active, u32 call, gu32* tmo) {
    const u64 t0 = __builtin_amdgcn_s_memrealtime();
    bool bad = false;
    for (u32 it = 0;; ++it) {
        u32 v = call;
        if (active) v = __hip_atomic_load(flag, __ATOMIC_RELAXED, __HIP_MEMORY_SCOPE_SYSTEM);
        if (__all(v >= call)) break;
        if ((it & 15) == 15) {
            if (__hip_atomic_load(tmo, __ATOMIC_RELAXED, __HIP_MEMORY_SCOPE_AGENT) == call ||
                __builtin_amdgcn_s_memrealtime() - t0 > DEADLINE) {
                __hip_atomic_store(tmo, call, __ATOMIC_RELAXED, __HIP_MEMORY_SCOPE_AGENT);
                bad = true;
                break;
            }
        }
        __builtin_amdgcn_s_sleep(2);
    }
    asm volatile("" ::: "memory");
    return bad;
}

template <int CTRL, int RMASK>
__device__ __forceinline__ u32 dpp0(u32 v) {
    return __builtin_amdgcn_update_dpp(0u, v, CTRL, RMASK, 0xf, false);
}
__device__ __forceinline__ u32 wave_incl_scan(u32 v) {
    v += dpp0<0x111, 0xf>(v);
    v += dpp0<0x112, 0xf>(v);
    v += dpp0<0x114, 0xf>(v);
    v += dpp0<0x118, 0xf>(v);
    v += dpp0<0x142, 0xa>(v);
    v += dpp0<0x143, 0xc>(v);
    return v;
}
__device__ __forceinline__ u32 wave_max(u32 v) {
    v = max(v, dpp0<0x111, 0xf>(v));
    v = max(v, dpp0<0x112, 0xf>(v));
    v = max(v, dpp0<0x114, 0xf>(v));
    v = max(v, dpp0<0x118, 0xf>(v));
    v = max(v, dpp0<0x142, 0xa>(v));
    v = max(v, dpp0<0x143, 0xc>(v));
    return __builtin_amdgcn_readlane(v, 63);
}

constexpr int A7_OFF = 65536 + 256;
constexpr int LDS_BYTES = A7_OFF + 11 * 8192;

constexpr u32 MODE_W6 = 0, MODE_RAW = 1, MODE_W5 = 2;
constexpr u32 NINL = 62;
constexpr u32 MAX_EXC = 510;
constexpr u32 REPICK = 127;
constexpr u32 MAX_ESC = 1344;

struct Enc {
    u32x4 c0, c1, c2, c3;
    u32x4 hv;
    u32 lh, l2, l3;
};

typedef __attribute__((address_space(3))) char lchar;
__device__ __forceinline__ u32 lds_addr(char* p) { return (u32)(u64)(lchar*)p; }
__device__ __forceinline__ void lds_xor(u32 addr, u32 v) {
    __hip_atomic_fetch_xor((__attribute__((address_space(3))) u32*)(u64)addr, v, __ATOMIC_RELAXED, __HIP_MEMORY_SCOPE_WORKGROUP);
}
__device__ __forceinline__ void lds_or(u32 addr, u32 v) {
    __hip_atomic_fetch_or((__attribute__((address_space(3))) u32*)(u64)addr, v, __ATOMIC_RELAXED, __HIP_MEMORY_SCOPE_WORKGROUP);
}

__device__ __forceinline__ void dma_raw(const unsigned char* src, u32 ldsdst) {
    u32 keep;
    const unsigned char* s1 = src + 16 * NN;
    const unsigned char* s2 = src + 32 * NN;
    const unsigned char* s3 = src + 48 * NN;
    asm volatile("s_mov_b32 %0, m0\n\t"
                 "s_mov_b32 m0, %5\n\ts_nop 0\n\tglobal_load_lds_dwordx4 %1, off\n\t"
                 "s_add_u32 m0, m0, 0x400\n\ts_nop 0\n\tglobal_load_lds_dwordx4 %2, off\n\t"
                 "s_add_u32 m0, m0, 0x400\n\ts_nop 0\n\tglobal_load_lds_dwordx4 %3, off\n\t"
                 "s_add_u32 m0, m0, 0x400\n\ts_nop 0\n\tglobal_load_lds_dwordx4 %4, off\n\t"
                 "s_mov_b32 m0, %0"
                 : "=&s"(keep) : "v"(src), "v"(s1), "v"(s2), "v"(s3), "s"(ldsdst) : "memory", "scc");
}
__device__ __forceinline__ void draw_ticket(u32& t, u32* ctr) {
    u64 save;
    asm volatile("s_mov_b64 %1, exec\n\ts_mov_b64 exec, 1\n\tglobal_atomic_add %0, %2, %3, %4 sc0\n\ts_mov_b64 exec, %1"
                 : "=&v"(t), "=&s"(save) : "v"(0u), "v"(1u), "s"(ctr) : "memory");
}
template <int N>
__device__ __forceinline__ void wait_tk(u32& t) {
    asm volatile("s_waitcnt vmcnt(%1)" : "+v"(t) : "n"(N) : "memory");
}

__device__ __forceinline__ void pick_bases(const u32 (&r)[16], u32* hist, int lane, u32& b8, u32& b3) {
    if (lane < 40) hist[lane] = 0;
    asm volatile("" ::: "memory");
    __hip_atomic_fetch_add(&hist[(r[0] >> 2) & 31], 1u, __ATOMIC_RELAXED, __HIP_MEMORY_SCOPE_WORKGROUP);
    __hip_atomic_fetch_add(&hist[(r[5] >> 10) & 31], 1u, __ATOMIC_RELAXED, __HIP_MEMORY_SCOPE_WORKGROUP);
    __hip_atomic_fetch_add(&hist[(r[10] >> 18) & 31], 1u, __ATOMIC_RELAXED, __HIP_MEMORY_SCOPE_WORKGROUP);
    __hip_atomic_fetch_add(&hist[(r[15] >> 26) & 31], 1u, __ATOMIC_RELAXED, __HIP_MEMORY_SCOPE_WORKGROUP);
    asm volatile("" ::: "memory");
    u32 h[8];
    const int hl = lane & 31;
#pragma unroll
    for (int i = 0; i < 8; ++i) h[i] = hist[hl + i];
    const u32 s3 = h[0] + h[1] + h[2];
    const u32 s8 = s3 + h[3] + h[4] + h[5] + h[6] + h[7];
    b8 = 31 - (wave_max((lane <= 24) ? (s8 * 32 + (31 - lane)) : 0u) & 31);
    b3 = 31 - (wave_max((lane <= 28) ? (s3 * 32 + (31 - lane)) : 0u) & 31);
}

__device__ __forceinline__ void w5_table(u32 b, u32& lo, u32& hi) {
    const u32 B4 = (b << 2) * 0x01010101u;
    lo = (((B4 | 0x80808080u) - 0x0C080400u) & 0x7C7C7C00u) | ((b + 3) << 2);
    hi = ((B4 | 0x80808080u) - 0x1C181410u) & 0x7C7C7C7Cu;
}

struct SendLds {
    char* raw;
    unsigned short* list;
    u32* hist;
    char* stage;
    const u32* lut;
};
__device__ __forceinline__ SendLds send_lds(char* base) {
    return SendLds{base, (unsigned short*)(base + 8192), (u32*)(base + 9344), base + 9536, (const u32*)(base + 10560)};
}
constexpr int SND_LDS = 10752;

__device__ __forceinline__ void encode(Enc& e, const char* raw, const SendLds& L, int lane, u32& b8, u32& b3, bool allow5) {
    const int q = lane & 15, g = lane >> 4;
    unsigned short* const list = L.list;
    u32 r[16];
#pragma unroll
    for (int i = 0; i < 16; ++i) r[i] = *(const u32*)(raw + (g * 16 + i) * 64 + q * 4);
    u32 T[16];
#pragma unroll
    for (int t = 0; t < 4; ++t) {
        const u32 t0 = perm(r[4 * t + 1], r[4 * t + 0], 0x05010400u);
        const u32 t1 = perm(r[4 * t + 1], r[4 * t + 0], 0x07030602u);
        const u32 t2 = perm(r[4 * t + 3], r[4 * t + 2], 0x05010400u);
        const u32 t3 = perm(r[4 * t + 3], r[4 * t + 2], 0x07030602u);
        T[0 + t]  = perm(t2, t0, 0x05040100u);
        T[4 + t]  = perm(t2, t0, 0x07060302u);
        T[8 + t]  = perm(t3, t1, 0x05040100u);
        T[12 + t] = perm(t3, t1, 0x07060302u);
    }
    bool picked = false;
    if (b8 >= 32) { pick_bases(r, L.hist, lane, b8, b3); picked = true; }
    const u32 posb = ((g & 1) * 512 + (g >> 1) * 256 + q * 16) << 4;

    if (allow5) {
        u32 code[16], kb[16], em[2], xm[2], cnt, incl, nesc, nexc;
        while (true) {
            const u32 B4 = (b3 << 2) * 0x01010101u;
            em[0] = em[1] = xm[0] = xm[1] = 0;
#pragma unroll
            for (int i = 0; i < 16; ++i) {
                const u32 t = T[i];
                const u32 u = ((t & 0x7C7C7C7Cu) | 0x80808080u) - B4;
                const u32 bl = (~u >> 7) & 0x01010101u;
                const u32 v = u & 0x7C7C7C7Cu;
                const u32 esc = (((v + 0x74747474u) >> 7) & 0x01010101u) | bl;
                code[i] = (t & 0x83838383u) | (v & 0x0C0C0C0Cu) | shl<3>(esc) | shl<2>(esc);
                const u32 k4 = (0x80808080u - v) & 0x7C7C7C7Cu;
                const u32 big = shl<1>(k4) | (k4 << 2), ge4 = v + 0x70707070u;
                const u32 m8 = bl << 7;
                const u32 exc = (((m8 & big) | (~m8 & ge4)) >> 7) & 0x01010101u;
                kb[i] = (k4 >> 2) & (shl<3>(bl) - bl);
                em[i >> 3] |= __builtin_amdgcn_udot4(esc, 0x08040201u, 0u, false) << (4 * (i & 7));
                xm[i >> 3] |= __builtin_amdgcn_udot4(exc, 0x08040201u, 0u, false) << (4 * (i & 7));
            }
            const u32 ne = __builtin_popcount(em[0]) + __builtin_popcount(em[1]);
            cnt = ne | ((__builtin_popcount(xm[0]) + __builtin_popcount(xm[1])) << 16);
            incl = wave_incl_scan(cnt);
            const u32 tot = __builtin_amdgcn_readlane(incl, 63);
            nesc = tot & 0xFFFFu;
            nexc = tot >> 16;
            if (__builtin_amdgcn_ballot_w64(ne > 32)) nesc = MAX_ESC + 1;
            if ((nesc <= MAX_ESC && nexc <= NINL) || picked) break;
            picked = true;
            const u32 old = b3;
            pick_bases(r, L.hist, lane, b8, b3);
            if (b3 == old) break;
        }
        if (nesc <= MAX_ESC && nexc <= NINL) {
            u32* const strm = (u32*)(L.stage + 512);
            if (lane < 32) *(u32x4*)(L.stage + 512 + lane * 16) = u32x4{0, 0, 0, 0};
            const u32 excl = incl - cnt;
            u32 sel[16], pc[16];
#pragma unroll
            for (int i = 0; i < 16; ++i) {
                const u32 m = (em[i >> 3] >> (4 * (i & 7))) & 15u;
                sel[i] = L.lut[m];
                pc[i] = __builtin_popcount(m);
            }
            const u32 first = excl & 0xFFFFu;
            const u32 bit0 = 3 * first;
            u32 wi = lds_addr((char*)strm) + ((bit0 >> 5) << 2);
            u32 nb = bit0 & 31;
            u64 accb = 0;
            asm volatile("" ::: "memory");
#pragma unroll
            for (int i = 0; i < 16; ++i) {
                const u32 cmp = perm(0u, kb[i], sel[i]);
                const u32 g12 = __builtin_amdgcn_udot4(cmp, 0x00400801u, (cmp >> 24) << 9, false);
                accb |= (u64)g12 << nb;
                nb += shl<1>(pc[i]) + pc[i];
                if ((i & 1) && i < 15) {
                    const bool full = nb >= 32;
                    lds_or(wi, full ? (u32)accb : 0u);
                    accb = full ? accb >> 32 : accb;
                    nb &= 31;
                    wi += full ? 4 : 0;
                }
            }
            lds_or(wi, (u32)accb);
            lds_or(wi + 4, (u32)(accb >> 32));
            {
                u64 xm64 = ((u64)xm[1] << 32) | xm[0];
                const u64 em64 = ((u64)em[1] << 32) | em[0];
                u32 idx = 2 + (excl >> 16);
                while (xm64) {
                    const u32 j = __builtin_ctzll(xm64);
                    xm64 &= xm64 - 1;
                    const u32 ex = ((u32)(unsigned char)raw[(g * 16 + (j & 15)) * 64 + q * 4 + (j >> 4)] >> 2) & 31u;
                    const u32 k0 = ex < b3 ? (b3 - ex) & 7u : 0u;
                    const u32 k = 2 - ((ex ^ b3) & 1u);
                    const u32 bit = 3 * (first + __builtin_popcountll(em64 & ((1ull << j) - 1)));
                    const u64 fix = (u64)(k0 ^ k) << (bit & 31);
                    const u32 wa = lds_addr((char*)strm) + ((bit >> 5) << 2);
                    lds_xor(wa, (u32)fix);
                    lds_xor(wa + 4, (u32)(fix >> 32));
                    list[idx++] = (unsigned short)(posb + (j >> 4) * 16384 + (j & 15) * 16 + ((ex ^ ((b3 - k) & 31u)) >> 1));
                }
            }
            if (lane == 0) *(u32*)list = b3 | (MODE_W5 << 5) | (nexc << 7) | (nesc << 16);
            u32 cw[10];
#pragma unroll
            for (int h = 0; h < 2; ++h) {
                const u32* cd = code + 8 * h;
                cw[5 * h + 0] = cd[0] | ((cd[5] & 0x07070707u) << 4);
                cw[5 * h + 1] = cd[1] | ((cd[5] & 0x08080808u) << 1) | ((cd[5] & 0x80808080u) >> 2) | ((cd[6] & 0x80808080u) >> 1);
                cw[5 * h + 2] = cd[2] | ((cd[6] & 0x07070707u) << 4);
                cw[5 * h + 3] = cd[3] | ((cd[6] & 0x08080808u) << 1) | ((cd[7] & 0x08080808u) << 2) | ((cd[7] & 0x80808080u) >> 1);
                cw[5 * h + 4] = cd[4] | ((cd[7] & 0x07070707u) << 4);
            }
            *(u32x2*)(L.stage + lane * 8) = u32x2{cw[8], cw[9]};
            asm volatile("" ::: "memory");
            e.c2 = *(const u32x4*)(L.stage + lane * 16);
            e.hv = *(const u32x4*)((const char*)list + (lane & 7) * 16);
            e.c0 = u32x4{cw[0], cw[1], cw[2], cw[3]};
            e.c1 = u32x4{cw[4], cw[5], cw[6], cw[7]};
            e.c3 = e.c2;
            e.lh = (nexc + 2 + 7) >> 3;
            e.l2 = 32 + ((nesc * 3 + 127) >> 7);
            e.l3 = 0;
            asm volatile("" ::: "memory");
            return;
        }
    }

    u32 code[16], D[4], n, incl, total;
    while (true) {
        const u32 B4 = (b8 << 2) * 0x01010101u;
        const u32 NB4 = (((8 - b8) & 7) << 2) * 0x01010101u;
        n = 0;
#pragma unroll
        for (int k = 0; k < 4; ++k) {
            u32 v[4];
#pragma unroll
            for (int t = 0; t < 4; ++t) {
                const int i = 4 * k + t;
                const u32 d4 = ((T[i] & 0x1C1C1C1Cu) + NB4) & 0x1C1C1C1Cu;
                code[i] = (T[i] & 0x83838383u) | d4;
                const u32 z = T[i] ^ (code[i] + B4);
                v[t] = __builtin_amdgcn_udot4(z, 0x40100401u, 0u, false);
            }
            D[k] = (v[0] >> 5) | (v[1] << 3) | (v[2] << 11) | (v[3] << 19);
            n += __builtin_popcount((D[k] | (D[k] >> 1)) & 0x55555555u);
        }
        incl = wave_incl_scan(n);
        total = __builtin_amdgcn_readlane(incl, 63);
        if (total <= REPICK || picked) break;
        picked = true;
        const u32 old = b8;
        pick_bases(r, L.hist, lane, b8, b3);
        if (b8 == old) break;
    }
    if (total <= MAX_EXC) {
        u32 idx = 2 + incl - n;
#pragma unroll
        for (int k = 0; k < 4; ++k) {
            u32 dd = D[k];
            while (dd) {
                const u32 bit = __builtin_ctz(dd) & ~1u;
                list[idx++] = (unsigned short)(posb + k * 16384 + (bit << 3) + (((dd >> bit) & 3) << 2));
                dd &= ~(3u << bit);
            }
        }
        if (lane == 0) *(u32*)list = b8 | (MODE_W6 << 5) | (total << 7);
        u32 cw[12];
#pragma unroll
        for (int h = 0; h < 2; ++h) {
            const u32* cd = code + 8 * h;
            cw[6 * h + 0] = cd[0] | ((cd[6] & 0x03030303u) << 5);
            cw[6 * h + 1] = cd[1] | ((cd[6] & 0x0C0C0C0Cu) << 3);
            cw[6 * h + 2] = cd[2] | ((cd[6] & 0x10101010u) << 1) | ((cd[6] & 0x80808080u) >> 1);
            cw[6 * h + 3] = cd[3] | ((cd[7] & 0x03030303u) << 5);
            cw[6 * h + 4] = cd[4] | ((cd[7] & 0x0C0C0C0Cu) << 3);
            cw[6 * h + 5] = cd[5] | ((cd[7] & 0x10101010u) << 1) | ((cd[7] & 0x80808080u) >> 1);
        }
        asm volatile("" ::: "memory");
        e.hv = *(const u32x4*)((const char*)list + (lane & 7) * 16);
        e.c3 = *(const u32x4*)((const char*)list + 128 + (lane < 56 ? lane : 0) * 16);
        e.c0 = u32x4{cw[0], cw[1], cw[2], cw[3]};
        e.c1 = u32x4{cw[4], cw[5], cw[6], cw[7]};
        e.c2 = u32x4{cw[8], cw[9], cw[10], cw[11]};
        e.lh = (min(total, NINL) + 2 + 7) >> 3;
        e.l2 = 64;
        e.l3 = total > NINL ? (total - NINL + 7) >> 3 : 0;
        asm volatile("" ::: "memory");
    } else {
        e.hv = u32x4{(lane == 0) ? (MODE_RAW << 5) : 0u, 0u, 0u, 0u};
        e.c0 = u32x4{T[0], T[1], T[2], T[3]};
        e.c1 = u32x4{T[4], T[5], T[6], T[7]};
        e.c2 = u32x4{T[8], T[9], T[10], T[11]};
        e.c3 = u32x4{T[12], T[13], T[14], T[15]};
        e.lh = 1;
        e.l2 = 64;
        e.l3 = 64;
    }
}

template <int IMM>
__device__ __forceinline__ void st16_all(char* const (&dst)[NR], u32 off, u32x4 v, u32 lanes) {
    const u64 mask = lanes >= 64 ? ~0ull : (1ull << lanes) - 1;
    u64 save;
    asm volatile("s_mov_b64 %0, exec\n\ts_mov_b64 exec, %1\n\t"
                 "global_store_dwordx4 %2, %3, %4 offset:%12 sc0 sc1\n\t"
                 "global_store_dwordx4 %2, %3, %5 offset:%12 sc0 sc1\n\t"
                 "global_store_dwordx4 %2, %3, %6 offset:%12 sc0 sc1\n\t"
                 "global_store_dwordx4 %2, %3, %7 offset:%12 sc0 sc1\n\t"
                 "global_store_dwordx4 %2, %3, %8 offset:%12 sc0 sc1\n\t"
                 "global_store_dwordx4 %2, %3, %9 offset:%12 sc0 sc1\n\t"
                 "global_store_dwordx4 %2, %3, %10 offset:%12 sc0 sc1\n\t"
                 "global_store_dwordx4 %2, %3, %11 offset:%12 sc0 sc1\n\t"
                 "s_mov_b64 exec, %0\n\ts_nop 1"
                 : "=&s"(save)
                 : "s"(mask), "v"(off), "v"(v), "s"(dst[0]), "s"(dst[1]), "s"(dst[2]), "s"(dst[3]), "s"(dst[4]),
                   "s"(dst[5]), "s"(dst[6]), "s"(dst[7]), "i"(IMM)
                 : "memory");
}
__device__ __forceinline__ void store_unit(const Enc& e, char* const (&dst)[NR], u32 slotoff, int lane) {
    const u32 off = slotoff + 128 + lane * 16;
    st16_all<0>(dst, off, e.c0, 64);
    st16_all<1024>(dst, off, e.c1, 64);
    st16_all<2048>(dst, off, e.c2, e.l2);
    st16_all<0>(dst, slotoff + lane * 16, e.hv, e.lh);
    if (e.l3) st16_all<3072>(dst, off, e.c3, e.l3);
}

__device__ __forceinline__ void flag_store(char* a, u32 call) {
    __hip_atomic_store((gu32*)(u64)a, call, __ATOMIC_RELAXED, __HIP_MEMORY_SCOPE_SYSTEM);
}

constexpr int RSLOT = 5 * 1024 + 64;
struct Ring {
    char* slot;
    u32* head;
    u32* done;
    u32* ready;
};
__device__ __forceinline__ Ring ring_of(char* lds) {
    char* b = lds + NE * SND_LDS;
    return Ring{b + 64, (u32*)b, (u32*)(b + 4), (u32*)(b + 16)};
}
__device__ __forceinline__ u32 lds_load(const u32* p) {
    return __hip_atomic_load(p, __ATOMIC_RELAXED, __HIP_MEMORY_SCOPE_WORKGROUP);
}

__device__ __forceinline__ void encoder(const Params& p, char* lds, int e) {
    const int lane = threadIdx.x & 63;
    char* const myws = p.ws[p.rank];
    u32* const cnt = (u32*)(myws + CNT_OFF + (size_t)(blockIdx.x * 8 + e + 1) * 16);
    const SendLds L = send_lds(lds + e * SND_LDS);
    const Ring R = ring_of(lds);
    const u32 rawaddr = __builtin_amdgcn_readfirstlane(lds_addr(L.raw));
    if (lane < 16) {
        u32 sel = 0x0C0C0C0Cu, n = 0;
#pragma unroll
        for (int y = 0; y < 4; ++y)
            if ((lane >> y) & 1) { sel = (sel & ~(0xFFu << (8 * n))) | ((u32)y << (8 * n)); ++n; }
        ((u32*)L.lut)[lane] = sel;
    }
    const unsigned char* const wsrc = p.w + (size_t)(lane >> 2) * NN + (lane & 3) * 16;
    u32* const tick = (u32*)uni(myws + TICK_OFF);
    const u32 call = __builtin_amdgcn_readfirstlane(*cnt + 1);
    const u32 tbase = (call - 1) * (NU + NS * NE);
    u32 b8 = 32, b3 = 32, lastu = 0;
    while (true) {
        u32 pos = 0;
        if (lane == 0) pos = __hip_atomic_fetch_add(R.head, 1u, __ATOMIC_RELAXED, __HIP_MEMORY_SCOPE_WORKGROUP);
        pos = __builtin_amdgcn_readfirstlane(pos);
        const u32 depth = lastu + RINGL_FROM >= (u32)W5_UNITS ? RINGL : RING;
        while (pos >= __builtin_amdgcn_readfirstlane(lds_load(R.done)) + depth) __builtin_amdgcn_s_sleep(2);
        asm volatile("" ::: "memory");
        u32 tk;
        draw_ticket(tk, tick);
        wait_tk<0>(tk);
        const u32 u = __builtin_amdgcn_readfirstlane(tk) - tbase;
        char* const sl = R.slot + (pos % RING) * RSLOT;
        u32* const meta = (u32*)(sl + 5 * 1024);
        if (u >= (u32)NU) {
            if (lane == 0) {
                meta[0] = NU;
                __hip_atomic_store(&R.ready[pos % RING], pos + 1, __ATOMIC_RELEASE, __HIP_MEMORY_SCOPE_WORKGROUP);
                *cnt = call;
            }
            return;
        }
        lastu = u;
        dma_raw(wsrc + (size_t)(u / NC) * 64 * NN + (u % NC) * 64, rawaddr);
        wait_vm0();
        Enc en;
        encode(en, L.raw, L, lane, b8, b3, w5_round((int)(u / NC)));
        *(u32x4*)(sl + lane * 16) = en.c0;
        *(u32x4*)(sl + 1024 + lane * 16) = en.c1;
        *(u32x4*)(sl + 2048 + lane * 16) = en.c2;
        *(u32x4*)(sl + 3072 + lane * 16) = en.c3;
        *(u32x4*)(sl + 4096 + lane * 16) = en.hv;
        if (lane == 0) {
            *(u32x4*)meta = u32x4{u, en.lh, en.l2, en.l3};
            __hip_atomic_store(&R.ready[pos % RING], pos + 1, __ATOMIC_RELEASE, __HIP_MEMORY_SCOPE_WORKGROUP);
        }
    }
}

__device__ __forceinline__ void storer(const Params& p, char* lds) {
    const int lane = threadIdx.x & 63;
    const int rank = p.rank;
    const int wid = blockIdx.x;
    char* const myws = p.ws[rank];
    u32* const cnt = (u32*)(myws + CNT_OFF + (size_t)(blockIdx.x * 8) * 16);
    const Ring R = ring_of(lds);
    const u32 call = __builtin_amdgcn_readfirstlane(*cnt + 1);
    const size_t par = DATA_OFF + (size_t)(call & 1) * PAR_BYTES;
    char* flagbase = p.ws[lane & (NR - 1)] + par + (size_t)rank * BLK + WF_OFF;
    char* dst[NR];
    const int rot = (wid + (wid >> 3)) & (NR - 1);
#pragma unroll
    for (int d = 0; d < NR; ++d) dst[d] = uni(p.ws[(rank + 1 + d + rot) & (NR - 1)] + par);
    asm volatile("" : "+v"(flagbase));

    int uprev = -1;
    int ends = 0;
    for (u32 pos = 0; ends < NE; ++pos) {
        while (__builtin_amdgcn_readfirstlane(lds_load(&R.ready[pos % RING])) != pos + 1) {
            if (uprev >= 0) {
                wait_vm0();
                if (lane < NR) flag_store(flagbase + (size_t)uprev * 128, call);
                uprev = -1;
            }
            __builtin_amdgcn_s_sleep(1);
        }
        asm volatile("" ::: "memory");
        const char* const sl = R.slot + (pos % RING) * RSLOT;
        const u32x4 meta = *(const u32x4*)(sl + 5 * 1024);
        const u32 u = __builtin_amdgcn_readfirstlane(meta.x);
        Enc e;
        if (u < (u32)NU) {
            e.c0 = *(const u32x4*)(sl + lane * 16);
            e.c1 = *(const u32x4*)(sl + 1024 + lane * 16);
            e.c2 = *(const u32x4*)(sl + 2048 + lane * 16);
            e.c3 = *(const u32x4*)(sl + 3072 + lane * 16);
            e.hv = *(const u32x4*)(sl + 4096 + lane * 16);
            e.lh = __builtin_amdgcn_readfirstlane(meta.y);
            e.l2 = __builtin_amdgcn_readfirstlane(meta.z);
            e.l3 = __builtin_amdgcn_readfirstlane(meta.w);
            asm volatile("" :: "v"(e.c0), "v"(e.c1), "v"(e.c2), "v"(e.c3), "v"(e.hv));
        }
        if (lane == 0) __hip_atomic_store(R.done, pos + 1, __ATOMIC_RELEASE, __HIP_MEMORY_SCOPE_WORKGROUP);
        if (u >= (u32)NU) { ++ends; continue; }
        store_unit(e, dst, slot_off(rank, u), lane);
        asm volatile("s_waitcnt vmcnt(32)" ::: "memory");
        if (uprev >= 0 && lane < NR) flag_store(flagbase + (size_t)uprev * 128, call);
        uprev = u;
    }
    if (uprev >= 0) {
        wait_vm0();
        if (lane < NR) flag_store(flagbase + (size_t)uprev * 128, call);
    }
    if (lane == 0) *cnt = call;
}

__device__ __forceinline__ void decode_unit(char* img, __amdgpu_buffer_rsrc_t rs, u32 slot, int lane, u32x4 cw0,
                                            u32x4 cw1, u32x4 cw2, u32 ent, u32x2 t2, u32x2 vv, bool maybe5) {
    const u32 hdr = __builtin_amdgcn_readfirstlane(ent) & 0xFFFFu;
    const u32 mode = (hdr >> 5) & 3;
    const u32 total = hdr >> 7;
    const int q = lane & 15, g = lane >> 4;
    char* const wp = img + (g & 1) * 512 + (g >> 1) * 256 + q * 16;
    if (mode == MODE_RAW) {
        const u32x4 v3 = __builtin_amdgcn_raw_buffer_load_b128(rs, slot + 128 + 3072 + lane * 16, 0, AUX_SYS);
        *(u32x4*)(wp + 0) = cw0;
        *(u32x4*)(wp + 1024) = cw1;
        *(u32x4*)(wp + 2048) = cw2;
        *(u32x4*)(wp + 3072) = v3;
        return;
    }
    const u32 B4 = ((hdr & 31) << 2) * 0x01010101u;
    if (maybe5 && mode == MODE_W5) {
        ((u32*)img)[lane] = vv.x;
        ((u32*)img)[64 + lane] = vv.y;
        const u32 c[10] = {cw0.x, cw0.y, cw0.z, cw0.w, cw1.x, cw1.y, cw1.z, cw1.w, t2.x, t2.y};
        u32 o[16], n = 0;
#pragma unroll
        for (int h = 0; h < 2; ++h) {
            const u32* cc = c + 5 * h;
            u32* oo = o + 8 * h;
#pragma unroll
            for (int i = 0; i < 5; ++i) oo[i] = cc[i] & 0x8F8F8F8Fu;
            oo[5] = ((cc[0] >> 4) & 0x07070707u) | ((cc[1] >> 1) & 0x08080808u) | ((cc[1] << 2) & 0x80808080u);
            oo[6] = ((cc[2] >> 4) & 0x07070707u) | ((cc[3] >> 1) & 0x08080808u) | ((cc[1] << 1) & 0x80808080u);
            oo[7] = ((cc[4] >> 4) & 0x07070707u) | ((cc[3] >> 2) & 0x08080808u) | ((cc[3] << 1) & 0x80808080u);
        }
        u32 fm[16], m4[16], t3[16];
#pragma unroll
        for (int i = 0; i < 16; ++i) {
            m4[i] = o[i] & (o[i] >> 1) & 0x04040404u;
            fm[i] = shl<5>(m4[i]) - m4[i];
            t3[i] = __builtin_popcount(fm[i] & 0x1C1C1C1Cu);
            n += t3[i];
        }
        const u32 bit0 = wave_incl_scan(n) - n;
        u32 tlo, thi;
        w5_table(hdr & 31, tlo, thi);
        asm volatile("" ::: "memory");
        const u32* const sw = (const u32*)img + (bit0 >> 5);
        const u32 w0 = sw[0], w1 = sw[1], w2 = sw[2], w3 = sw[3];
        u32 v0 = __builtin_amdgcn_alignbit(w1, w0, bit0), v1 = __builtin_amdgcn_alignbit(w2, w1, bit0),
            v2 = __builtin_amdgcn_alignbit(w3, w2, bit0);
#pragma unroll
        for (int i = 0; i < 16; ++i) {
            const u32 t6 = (v0 & 0x3Fu) | (shl<10>(v0) & 0x003F0000u);
            const u32 kx = (t6 | shl<5>(t6)) & 0x07070707u;
            v0 = __builtin_amdgcn_alignbit(v1, v0, t3[i]);
            v1 = __builtin_amdgcn_alignbit(v2, v1, t3[i]);
            v2 >>= t3[i];
            u32 rk;
            asm("v_mul_u32_u24 %0, %1, %2" : "=v"(rk) : "v"(m4[i]), "s"(0x404040u));
            const u32 eb = perm(thi, tlo, perm(0u, kx, rk));
            o[i] = (fm[i] & eb) | (~fm[i] & (o[i] + B4));
        }
        asm volatile("" ::: "memory");
#pragma unroll
        for (int f = 0; f < 4; ++f) *(u32x4*)(wp + f * 1024) = u32x4{o[4 * f], o[4 * f + 1], o[4 * f + 2], o[4 * f + 3]};
    } else {
        const u32 c[12] = {cw0.x, cw0.y, cw0.z, cw0.w, cw1.x, cw1.y, cw1.z, cw1.w, cw2.x, cw2.y, cw2.z, cw2.w};
#pragma unroll
        for (int h = 0; h < 2; ++h) {
            const u32* cc = c + 6 * h;
            u32x4 a, bq;
            a.x = (cc[0] & 0x9F9F9F9Fu) + B4;
            a.y = (cc[1] & 0x9F9F9F9Fu) + B4;
            a.z = (cc[2] & 0x9F9F9F9Fu) + B4;
            a.w = (cc[3] & 0x9F9F9F9Fu) + B4;
            bq.x = (cc[4] & 0x9F9F9F9Fu) + B4;
            bq.y = (cc[5] & 0x9F9F9F9Fu) + B4;
            bq.z = (((cc[0] >> 5) & 0x03030303u) | ((cc[1] >> 3) & 0x0C0C0C0Cu) |
                    ((cc[2] >> 1) & 0x10101010u) | ((cc[2] << 1) & 0x80808080u)) + B4;
            bq.w = (((cc[3] >> 5) & 0x03030303u) | ((cc[4] >> 3) & 0x0C0C0C0Cu) |
                    ((cc[5] >> 1) & 0x10101010u) | ((cc[5] << 1) & 0x80808080u)) + B4;
            *(u32x4*)(wp + (2 * h) * 1024) = a;
            *(u32x4*)(wp + (2 * h + 1) * 1024) = bq;
        }
    }
    {
        const u32 e = (lane < 2 || (u32)lane >= total + 2) ? 0u : (ent & 0xFFFFu);
        const u32 pos = e >> 4;
        __hip_atomic_fetch_xor((u32*)(img + (pos & ~3u)), (e & 15) << (3 + 8 * (pos & 3)), __ATOMIC_RELAXED,
                               __HIP_MEMORY_SCOPE_WORKGROUP);
    }
#pragma clang loop unroll(disable)
    for (u32 i0 = NINL; i0 < total; i0 += 64) {
        const u32 i = i0 + lane;
        u32 e = __builtin_amdgcn_raw_buffer_load_b16(rs, slot + 128 + 3072 + (i - NINL) * 2, 0, AUX_SYS);
        e = (i < total) ? (e & 0xFFFFu) : 0u;
        const u32 pos = e >> 4;
        __hip_atomic_fetch_xor((u32*)(img + (pos & ~3u)), (e & 15) << (3 + 8 * (pos & 3)), __ATOMIC_RELAXED,
                               __HIP_MEMORY_SCOPE_WORKGROUP);
    }
}

__device__ __forceinline__ void send_x(const Params& p, u32 call, size_t par, int h, int lane) {
    const int dest = __builtin_amdgcn_readfirstlane((p.rank + 1 + h + (h >> 3)) & (NR - 1));
    const int rf = __builtin_amdgcn_readfirstlane((h >> 3) & 31);
    const unsigned char* src = p.x + (size_t)(dest * MS + rf * 16 + (lane & 15)) * KS
                             + ((lane >> 4) & 1) * 32 + (lane >> 5) * 16;
    u32x4 v[8];
#pragma unroll
    for (int j = 0; j < 8; ++j) v[j] = *(const u32x4*)(src + j * 64);
    char* const base = uni(p.ws[dest] + par + (size_t)p.rank * BLK + X_OFF + rf * 8192);
    char* const xf = uni(p.ws[dest] + par + (size_t)p.rank * BLK + XF_OFF + rf * 128);
#pragma unroll
    for (int j = 0; j < 8; ++j) asm volatile("" :: "v"(v[j]));
#pragma unroll
    for (int j = 0; j < 8; j += 4) {
        st16_sys<0>(base + j * 1024, lane * 16, v[j]);
        st16_sys<1024>(base + j * 1024, lane * 16, v[j + 1]);
        st16_sys<2048>(base + j * 1024, lane * 16, v[j + 2]);
        st16_sys<3072>(base + j * 1024, lane * 16, v[j + 3]);
    }
    wait_vm0();
    if (lane == 0) __hip_atomic_store((gu32*)(u64)xf, call, __ATOMIC_RELAXED, __HIP_MEMORY_SCOPE_SYSTEM);
}

template <int NF>
__device__ __forceinline__ void consumer_wave(const Params& p, char* lds, int c, int wave, int rf0, int lf0, u32 call, size_t par) {
    const int lane = threadIdx.x & 63;
    char* const myws = uni(p.ws[p.rank]);
    gu32* const tmo = (gu32*)(u64)(myws + TMO_OFF);
    const float scale = p.sx[0] * p.sw[0];
    const __amdgpu_buffer_rsrc_t rs =
        __builtin_amdgcn_make_buffer_rsrc(uni(myws + par), 0, (int)PAR_BYTES, 0x00020000);
    u32* const badw = (u32*)(lds + 65536);
    u32* const rdy = (u32*)(lds + 65536 + 32);
    const __amdgpu_buffer_rsrc_t ro = __builtin_amdgcn_make_buffer_rsrc(uni((char*)p.out), 0, MS * NN * 4, 0x00020000);

    f32x4 acc[NF][4];

    const u32 xoff = (u32)((lane >> 5) * BLK + X_OFF + rf0 * 8192 + (lane & 31) * 16);
    const int L16 = lane & 15, G = lane >> 4;
    const u32 bread = (G >> 1) * 4096 + (G & 1) * 256 + L16 * 16;

    u32x4 A[4 * NF * 2];
    auto load_a = [&](int P, int j) {
#pragma unroll
        for (int a = 0; a < NF; ++a)
#pragma unroll
            for (int i = 0; i < 2; ++i)
                A[(P * NF + a) * 2 + i] = __builtin_amdgcn_raw_buffer_load_b128(
                    rs, xoff + j * 1024 + i * 512, P * 2 * BLK + a * 8192, AUX_SYS);
    };
    char* const a7 = lds + A7_OFF + lf0 * 8192;
    auto dma_a7 = [&]() {
        const char* src = myws + par + xoff + (NJ - 1) * 1024;
        asm volatile("" : "+v"(src));
        const u32 dst = __builtin_amdgcn_readfirstlane(lds_addr(a7));
#pragma unroll
        for (int a = 0; a < NF; ++a)
#pragma unroll
            for (int P = 0; P < 4; ++P) {
                u32 keep;
                const char* s0 = src + a * 8192 + (size_t)P * 2 * BLK;
                asm volatile("s_mov_b32 %0, m0\n\ts_mov_b32 m0, %2\n\ts_nop 0\n\tglobal_load_lds_dwordx4 %1, off sc0 sc1\n\t"
                             "s_add_u32 m0, m0, 0x400\n\ts_nop 0\n\tglobal_load_lds_dwordx4 %3, off sc0 sc1\n\ts_mov_b32 m0, %0"
                             : "=&s"(keep) : "v"(s0), "s"(dst + a * 8192 + P * 2048), "v"(s0 + 512) : "memory", "scc");
            }
    };
    auto mfma = [&](int P, int f, const u32x4& b0, const u32x4& b1) {
        const i32x8 B = {(int)b0.x, (int)b0.y, (int)b0.z, (int)b0.w, (int)b1.x, (int)b1.y, (int)b1.z, (int)b1.w};
#pragma unroll
        for (int a = 0; a < NF; ++a) {
            const u32x4 a0 = A[(P * NF + a) * 2], a1 = A[(P * NF + a) * 2 + 1];
            const i32x8 Av = {(int)a0.x, (int)a0.y, (int)a0.z, (int)a0.w, (int)a1.x, (int)a1.y, (int)a1.z, (int)a1.w};
            acc[a][f] = __builtin_amdgcn_mfma_scale_f32_16x16x128_f8f6f4(Av, B, acc[a][f], 0, 1, 0, 0, 0, 0);
        }
    };
    u32x4 cw0, cw1, cw2;
    u32x2 t2 = {0, 0};
    u32x2 vv = {0, 0};
    u32 ent;
    auto unit_slot = [&](int j) { return slot_off(wave, j * NC + c); };
    auto issue = [&](int j, u32 oob) {
        const u32 slot = unit_slot(j) | oob;
        ent = __builtin_amdgcn_raw_buffer_load_b16(rs, slot + lane * 2, 0, AUX_SYS);
        cw0 = __builtin_amdgcn_raw_buffer_load_b128(rs, slot + 128 + lane * 16, 0, AUX_SYS);
        cw1 = __builtin_amdgcn_raw_buffer_load_b128(rs, slot + 128 + 1024 + lane * 16, 0, AUX_SYS);
        cw2 = __builtin_amdgcn_raw_buffer_load_b128(rs, slot + 128 + 2048 + lane * 16, 0, AUX_SYS);
        if (w5_round(j)) {
            t2 = __builtin_amdgcn_raw_buffer_load_b64(rs, slot + 128 + 2048 + lane * 8, 0, AUX_SYS);
            vv.x = __builtin_amdgcn_raw_buffer_load_b32(rs, slot + 128 + 2560 + lane * 4, 0, AUX_SYS);
            vv.y = __builtin_amdgcn_raw_buffer_load_b32(rs, slot + 128 + 2560 + 256 + lane * 4, 0, AUX_SYS);
        }
    };
    auto decode = [&](int j, u32 oob) {
        decode_unit(lds + (j & 1) * 32768 + wave * 4096, rs, unit_slot(j) | oob, lane, cw0, cw1, cw2, ent, t2, vv,
                    w5_round(j));
    };
    auto wflag = [&](int j) { return (const gu32*)(u64)(myws + par + (size_t)wave * BLK + WF_OFF + (j * NC + c) * 128); };
    bool bad = false;

    bool have = false;
    auto round = [&](int j) {
        if (!have) {
            bad |= wait_flags(wflag(j), lane == 0, call, tmo);
            issue(j, 0);
        }
        decode(j, 0);
        __syncthreads();
        if (j == NJ - 3) {
            const u32 dummy = __builtin_amdgcn_raw_buffer_load_b32(ro, (u32)(((rf0 * 16 + G * 4) * NN + c * 64 + L16 * 4) * 4), 0, 16);
            asm volatile("" :: "v"(dummy));
        }
        u32 pv = __hip_atomic_load(wflag(j + 1), __ATOMIC_RELAXED, __HIP_MEMORY_SCOPE_SYSTEM);
        const char* const buf = lds + (j & 1) * 32768;
        u32x4 Bf[2][2][2];
#pragma unroll
        for (int f = 0; f < 2; ++f) {
            Bf[0][f][0] = *(const u32x4*)(buf + f * 1024 + bread);
            Bf[0][f][1] = *(const u32x4*)(buf + f * 1024 + bread + 512);
        }
#pragma unroll
        for (int st = 0; st < 8; ++st) {
            if (st < 7) {
                const int Pn = (st + 1) >> 1, fn = ((st + 1) & 1) * 2;
#pragma unroll
                for (int f = 0; f < 2; ++f) {
                    Bf[(st + 1) & 1][f][0] = *(const u32x4*)(buf + Pn * 8192 + (fn + f) * 1024 + bread);
                    Bf[(st + 1) & 1][f][1] = *(const u32x4*)(buf + Pn * 8192 + (fn + f) * 1024 + bread + 512);
                }
            }
            __builtin_amdgcn_sched_barrier(0);
#pragma unroll
            for (int f = 0; f < 2; ++f) mfma(st >> 1, (st & 1) * 2 + f, Bf[st & 1][f][0], Bf[st & 1][f][1]);
            if ((st & 1) && j + 2 < NJ) load_a(st >> 1, j + 1);
            if (st == EARLY_ST) {
                have = __builtin_amdgcn_readfirstlane(pv) >= call;
                asm volatile("" ::: "memory");
                if (have) issue(j + 1, 0);
                else pv = __hip_atomic_load(wflag(j + 1), __ATOMIC_RELAXED, __HIP_MEMORY_SCOPE_SYSTEM);
            }
            if (st == EARLY_ST2 && !have) {
                have = __builtin_amdgcn_readfirstlane(pv) >= call;
                asm volatile("" ::: "memory");
                if (have) issue(j + 1, 0);
            }
            __builtin_amdgcn_sched_barrier(0);
        }
    };

#pragma clang loop unroll(disable)
    for (u32 pass = 0; pass < 2; ++pass) {
        asm volatile("" : "+s"(pass));
        const u32 oob = pass ? 0u : 0x80000000u;
#pragma unroll
        for (int a = 0; a < NF; ++a)
#pragma unroll
            for (int f = 0; f < 4; ++f) acc[a][f] = f32x4{0.f, 0.f, 0.f, 0.f};
        if (pass) {
            bad = wait_flags((const gu32*)(u64)(myws + par + (size_t)(lane & 7) * BLK + XF_OFF + (rf0 + ((lane >> 3) & (NF - 1))) * 128),
                             lane < NF * NR, call, tmo);
            dma_a7();
#pragma unroll
            for (int P = 0; P < 4; ++P) load_a(P, 0);
#pragma clang loop unroll(disable)
            for (int j = 0; j < NJ - 1; ++j) round(j);
            asm volatile("s_waitcnt vmcnt(0)" ::: "memory");
        }

        {
            constexpr int j = NJ - 1;
            const char* const bbuf = lds + (j & 1) * 32768 + bread;
            const char* const abuf = a7 + lane * 16;
            const gu32* const fl = wflag(j);
            const u64 t0 = __builtin_amdgcn_s_memrealtime();
            asm volatile("" :: "s"(t0));
            bool own = false;
            u32 todo = 15, avail = 0, it = 0;
            auto pick = [&]() {
                if (!pass) { own = false; avail = 0; todo = 15; }
                while (!avail) {
                    u32 v = call;
                    if (!own && pass && !have) v = __hip_atomic_load(fl, __ATOMIC_RELAXED, __HIP_MEMORY_SCOPE_SYSTEM);
                    const u32x4 r0 = *(const u32x4*)rdy, r1 = *(const u32x4*)(rdy + 4);
                    const u32 m = (min(r0.x, r0.y) > pass ? 1u : 0u) | (min(r0.z, r0.w) > pass ? 2u : 0u) |
                                  (min(r1.x, r1.y) > pass ? 4u : 0u) | (min(r1.z, r1.w) > pass ? 8u : 0u);
                    avail = __builtin_amdgcn_readfirstlane(m & todo);
                    todo &= ~avail;
                    if (!own) {
                        bool go = __builtin_amdgcn_readfirstlane(v) >= call;
                        if (!go && (++it & 15) == 15 &&
                            (__hip_atomic_load(tmo, __ATOMIC_RELAXED, __HIP_MEMORY_SCOPE_AGENT) == call ||
                             __builtin_amdgcn_s_memrealtime() - t0 > DEADLINE)) {
                            __hip_atomic_store(tmo, call, __ATOMIC_RELAXED, __HIP_MEMORY_SCOPE_AGENT);
                            bad = true;
                            go = true;
                        }
                        if (go) {
                            asm volatile("" ::: "memory");
                            if (!have) issue(j, oob);
                            decode(j, oob);
                            if (lane == 0) {
                                badw[wave] = bad;
                                __hip_atomic_store(&rdy[wave], pass + 1, __ATOMIC_RELEASE, __HIP_MEMORY_SCOPE_WORKGROUP);
                            }
                            own = true;
                        }
                    } else if (!avail) {
                        __builtin_amdgcn_s_sleep(1);
                    }
                }
                asm volatile("" ::: "memory");
                const int P = __builtin_ctz(avail);
                avail &= avail - 1;
                return P;
            };
            struct Frags { u32x4 a[NF][2], b[4][2]; };
            auto rd_a = [&](Frags& F, int P, int a) {
                F.a[a][0] = *(const u32x4*)(abuf + a * 8192 + P * 2048);
                F.a[a][1] = *(const u32x4*)(abuf + a * 8192 + P * 2048 + 1024);
            };
            auto rd_b = [&](Frags& F, int P, int f) {
                F.b[f][0] = *(const u32x4*)(bbuf + P * 8192 + f * 1024);
                F.b[f][1] = *(const u32x4*)(bbuf + P * 8192 + f * 1024 + 512);
            };
            auto mm_f = [&](const Frags& F, int f) {
                const i32x8 B = {(int)F.b[f][0].x, (int)F.b[f][0].y, (int)F.b[f][0].z, (int)F.b[f][0].w,
                                 (int)F.b[f][1].x, (int)F.b[f][1].y, (int)F.b[f][1].z, (int)F.b[f][1].w};
#pragma unroll
                for (int a = 0; a < NF; ++a) {
                    const i32x8 Av = {(int)F.a[a][0].x, (int)F.a[a][0].y, (int)F.a[a][0].z, (int)F.a[a][0].w,
                                      (int)F.a[a][1].x, (int)F.a[a][1].y, (int)F.a[a][1].z, (int)F.a[a][1].w};
                    acc[a][f] = __builtin_amdgcn_mfma_scale_f32_16x16x128_f8f6f4(Av, B, acc[a][f], 0, 1, 0, 0, 0, 0);
                }
            };
            auto mmrd = [&](const Frags& Fc, Frags& Fn, int P) {
#pragma unroll
                for (int f = 0; f < 4; ++f) {
                    mm_f(Fc, f);
                    if (f < NF) rd_a(Fn, P, f);
                    rd_b(Fn, P, f);
                    __builtin_amdgcn_sched_barrier(0);
                }
            };
            Frags F0, F1;
            {
                const int P = pick();
#pragma unroll
                for (int a = 0; a < NF; ++a) rd_a(F0, P, a);
#pragma unroll
                for (int f = 0; f < 4; ++f) rd_b(F0, P, f);
            }
            mmrd(F0, F1, pick());
            mmrd(F1, F0, pick());
            mmrd(F0, F1, pick());
#pragma unroll
            for (int f = 0; f < 4; ++f) mm_f(F1, f);
        }

        const u32x4 b0 = *(const u32x4*)badw, b1 = *(const u32x4*)(badw + 4);
        const float s = (b0.x | b0.y | b0.z | b0.w | b1.x | b1.y | b1.z | b1.w) ? __builtin_nanf("") : scale;
        const u32 obase = (u32)(((rf0 * 16 + G * 4) * NN + c * 64 + L16 * 4) * 4) | oob;
#pragma unroll
        for (int a = 0; a < NF; ++a)
#pragma unroll
            for (int r = 0; r < 4; ++r) {
                const f32x4 v = {acc[a][0][r] * s, acc[a][1][r] * s, acc[a][2][r] * s, acc[a][3][r] * s};
                __builtin_amdgcn_raw_buffer_store_b128(__builtin_bit_cast(u32x4, v), ro, obase + (a * 16 + r) * NN * 4, 0, OUT_AUX);
            }
    }
}

__device__ __forceinline__ void consumer(const Params& p, char* lds) {
    const int lane = threadIdx.x & 63;
    const int wave = __builtin_amdgcn_readfirstlane(threadIdx.x >> 6);
    const int t = blockIdx.x - NS;
    const int x = t & 7, i = t >> 3;
    int c, f0, nfr;
    if (i < 12) { c = 4 * x + i / 3; f0 = (i % 3) * 11; nfr = (i % 3 == 2) ? 10 : 11; }
    else        { c = 32 + 4 * x + ((i - 12) >> 2); f0 = ((i - 12) & 3) * 8; nfr = 8; }
    const int n2 = nfr - 8;
    const int lf0 = wave < n2 ? 2 * wave : n2 + wave;
    char* const myws = p.ws[p.rank];
    u32* const cnt = (u32*)(myws + CNT_OFF + (size_t)(blockIdx.x * 8 + wave) * 16);
    const u32 call = __builtin_amdgcn_readfirstlane(*cnt + 1);
    const size_t par = DATA_OFF + (size_t)(call & 1) * PAR_BYTES;
    if (lane == 0) ((u32*)(lds + 65536 + 32))[wave] = 0;
    if (wave == (t & 7)) send_x(p, call, par, blockIdx.x, lane);
    if (wave < n2) consumer_wave<2>(p, lds, c, wave, f0 + lf0, lf0, call, par);
    else consumer_wave<1>(p, lds, c, wave, f0 + lf0, lf0, call, par);
    if (lane == 0) *cnt = call;
}

__global__ void __launch_bounds__(512, 2) gemm_rs_kernel(Params p) {
    __shared__ __attribute__((aligned(16))) char lds[LDS_BYTES];
    if (blockIdx.x < NS) {
        const int wave = __builtin_amdgcn_readfirstlane(threadIdx.x >> 6);
        if (threadIdx.x < 16) ((u32*)(lds + NE * SND_LDS))[threadIdx.x] = 0;
        __syncthreads();
        if (wave == 0) {
            storer(p, lds);
        } else if (wave <= NE) {
            encoder(p, lds, wave - 1);
        } else if (wave == 7) {
            const int lane = threadIdx.x & 63;
            u32* const cnt = (u32*)(p.ws[p.rank] + CNT_OFF + (size_t)(blockIdx.x * 8 + wave) * 16);
            const u32 call = __builtin_amdgcn_readfirstlane(*cnt + 1);
            send_x(p, call, DATA_OFF + (size_t)(call & 1) * PAR_BYTES, blockIdx.x, lane);
            if (lane == 0) *cnt = call;
        }
    } else {
        consumer(p, lds);
    }
}

}

extern "C" void kernel_launch(int rank, int nranks, ncclComm_t comm, hipStream_t stream,
                              void* const* d_in, const int* in_sizes, int n_in,
                              void* d_out, int out_size, const int* in_dtypes, int out_dtype,
                              void* const* d_out_all, void* const* d_ws_all, size_t ws_bytes) {
    (void)comm; (void)in_dtypes; (void)out_dtype; (void)d_out_all;
    if (nranks != NR || n_in != 4 || rank < 0 || rank >= NR || ws_bytes < WS_NEED ||
        in_sizes[0] != 4096 * KS || in_sizes[1] != KS * NN || out_size != MS * NN) {
        fprintf(stderr, "gemm_rs: unexpected problem shape\n");
        return;
    }
    Params p;
    p.x = (const unsigned char*)d_in[0];
    p.w = (const unsigned char*)d_in[1];
    p.sx = (const float*)d_in[2];
    p.sw = (const float*)d_in[3];
    p.out = (float*)d_out;
    for (int r = 0; r < NR; ++r) p.ws[r] = (char*)d_ws_all[r];
    p.rank = rank;
    p.pad = 0;
    hipLaunchKernelGGL(gemm_rs_kernel, dim3(GRID), dim3(512), 0, stream, p);
    HIP_CHECK(hipGetLastError());
}
```
